```python
import numpy as np

kernel_name = "gpu_a2a_gemm_m4096_k4096_n4096_bf16_silu"


ABSMAX_RTOL = 0.008

COMPUTE_DTYPE = "bfloat16"
EPILOGUE = "silu"
IN_QUANT = None
OUT_QUANT = None
IN_DTYPES = {"x": "bfloat16", "w_mat": "bfloat16"}
WORLD_SIZE = 8

def _to_bf16(x):
    u = np.ascontiguousarray(x, dtype=np.float32).view(np.uint32)
    return ((u + 0x7FFF + ((u >> 16) & 1)) & 0xFFFF0000).view(np.float32)

def setup_inputs():

    x = _to_bf16(np.random.randn(4096, 4096).astype(np.float32))
    w_mat = _to_bf16((np.random.randn(4096, 4096) / np.sqrt(4096)).astype(np.float32))
    return {"x": x, "w_mat": w_mat}

def _epilogue(y):
    return (y / (1.0 + np.exp(-np.clip(y, -60.0, 60.0)))).astype(np.float32)

def reference(x, w_mat):
    y = (x @ w_mat).astype(np.float32)
    return _epilogue(y)
```

```cpp
#include <hip/hip_runtime.h>
#include <rccl/rccl.h>
#include <cstdio>
#include <cstdint>

#define HIP_CHECK(expr) do { hipError_t _e = (expr); if (_e != hipSuccess) { \
    fprintf(stderr, "HIP error %s at %s:%d\n", hipGetErrorString(_e), __FILE__, __LINE__); \
    return; } } while (0)

namespace {

typedef __bf16 bf2 __attribute__((ext_vector_type(2)));
typedef float f4 __attribute__((ext_vector_type(4)));
typedef unsigned u4 __attribute__((ext_vector_type(4)));
typedef unsigned u8v __attribute__((ext_vector_type(8)));
typedef unsigned long long u64;

constexpr int NR = 8;
constexpr int ML = 512;
constexpr int NN = 4096;
constexpr int KS = 512;
constexpr int NWG = 256;
constexpr int NPUSH = 224;
constexpr int NA = 52;
constexpr size_t BUF_BYTES = 4u << 20;
constexpr size_t OFF_FLAGS = 2 * BUF_BYTES;
constexpr size_t OFF_CNT = OFF_FLAGS + 16384;
constexpr size_t OFF_TMO = OFF_CNT + NWG * 64;
constexpr size_t OFF_XF = OFF_TMO + 4096;
constexpr size_t OFF_X = 10u << 20;
constexpr size_t WS_NEED = OFF_X + (size_t)NWG * 32768;
constexpr unsigned long long DEADLINE_TICKS = 50000000ull;

constexpr int RING = 16;
constexpr int LDS_CTRL = RING * 8192;
constexpr int LDS_RDY = LDS_CTRL + 256;
constexpr int LDS_SEQ = LDS_CTRL + 512;
constexpr int LDS_CONS = LDS_CTRL + 768;
#define XH 0
#define XCL 0
#define XCD 18874368
constexpr int LDS_XCH = LDS_CTRL + 3072;
constexpr int LDS_POS = LDS_CONS + 2048;
constexpr int LDS_BYTES = XH == 1 ? LDS_XCH + 28672 : XH == 2 ? LDS_POS + 512 : LDS_CONS + 2048;
#define POLL_SLEEP 4
#define NO_PUSH 0
#define NO_POLL 0
#define NO_W 0
#define POLLM 0
#define POLLSC "sc0 sc1"
#define PWIN 7
#define PRIO 3
#define LPRIO 2
#define WFLY 3
#define WFLY2 WFLY
#define WPRE 1
#define ROT 1
#define T_FIRST 112
#define T_SLOT 35
#define T_FAST 31
#define FASTW 1
#define T_LEAD 250
#define SLOTTED
#define NEARLY 4
#define EBAR 1
#define LOOKA 12
#define WLEAD RING
#define SG 2
#define SPEC 3
#define OUTM 1
#define OUTFL "sc1"
#define PF 0
#define PFT 25
#define DYN 0
#define DYNR 5
#define DYNT 2
#define DLT 0
#define PFN 1
#define PFMAX 56
#define PFS 0
#define PFS0 8u
#define FAKE_R 124
#define FAKE_H 350
#define PFD 14
#define BSH 0
#define PFFL ""
#define NTHR ((PF || XH) ? 512 : 448)
#define HPRIO 1
#define XT 0
#define XTN 8
#define WHIT 0
#define AMASK 0xFFFFFFFFu
#define WMASK 0xFFFFFFFFu
#define XPAIR 1
#define EPI 1
#define XQ0 1
#define XQ1 2
#define HB 0
#define HM (1 << HB)
#define XX 0
#define XQS 0
#define PLUT 0x70123456
#define XMS 0
#define NB0 1
#define NB1 4
#define NM0 (1 << NB0)
#define NM1 (1 << NB1)
#define WRS (NN * 2)
#define WNS 256

struct Params {
    const char* x;
    const char* wm;
    char* out;
    char* myws;
    int rank, pad;
    char* peer[NR - 1];
};

#define DGAPN 0
#define DGAP ""
#define LAY 0
#define SUB 1
#define T0TOP 1
#define KALIGN 4096
#define ENDW() do { asm volatile("s_endpgm\n\t.subsection 0" ::: "memory"); return; } while (0)
#define EXP4(c) (c)
#define PFIRST 0
#define WSUB 3
#define POSUB 2
#define LATER(n, L) LATER_(n, L)
#define LATER_(n, L) "\n\ts_branch " L "\n\t.subsection " #n "\n" L ":"
#define CPF 1
#define CPN 8
#define CPS 6
#define XBM 255
#define WBV 0
#define WAITV(n) asm volatile("s_waitcnt vmcnt(" #n ")" ::: "memory")

__device__ __forceinline__ int get_lane() {
    int l;
    asm volatile("v_mbcnt_lo_u32_b32 %0, -1, 0\n\tv_mbcnt_hi_u32_b32 %0, -1, %0" : "=v"(l));
    return l;
}
template <int N> __device__ __forceinline__ void wait_vm() {
    asm volatile("s_waitcnt vmcnt(%0)" :: "i"(N) : "memory");
}
__device__ __forceinline__ void dma4(const unsigned* v, const char* base, unsigned lds_addr) {
    asm volatile("s_mov_b32 m0, %5\n\ts_nop 0\n\t"
                 "global_load_lds_dwordx4 %0, %4\n\t" DGAP "global_load_lds_dwordx4 %1, %4 offset:1024\n\t" DGAP
                 "global_load_lds_dwordx4 %2, %4 offset:2048\n\t" DGAP "global_load_lds_dwordx4 %3, %4 offset:3072\n\t" DGAP
                 :: "v"(v[0]), "v"(v[1]), "v"(v[2]), "v"(v[3]), "s"(base), "s"(lds_addr) : "memory");
}
__device__ __forceinline__ void store_sys4(char* p, unsigned v) {
    asm volatile("global_store_dword %0, %1, off sc0 sc1" :: "v"(p), "v"(v) : "memory");
}
__device__ __forceinline__ unsigned lds_read_wait(unsigned addr) {
    unsigned v;
    asm volatile("ds_read_b32 %0, %1\n\ts_waitcnt lgkmcnt(0)" : "=&v"(v) : "v"(addr) : "memory");
    return v;
}
__device__ __forceinline__ void lds_write(unsigned addr, unsigned v) {
    asm volatile("ds_write_b32 %0, %1" :: "v"(addr), "v"(v) : "memory");
}

struct Chunk { int src, round; };
__device__ __forceinline__ Chunk step_chunk(int s, int rank, int pos) {
    Chunk c;
    if (s < 8) { c.src = rank; c.round = (s + 2 * pos) & 7; }
    else {
        const int g = s - 8;
        c.round = g / 7;
        int q = g - 7 * c.round + ((pos * 7) >> 2);
        if (q >= 7) q -= 7;
        c.src = (rank + 1 + q) & 7;
    }
    return c;
}
__device__ __forceinline__ float silu(float v) {
    const float cl = fminf(fmaxf(v, -60.f), 60.f);
    return v * __builtin_amdgcn_rcpf(1.f + __builtin_amdgcn_exp2f(cl * -1.44269504088896340736f));
}
__device__ __forceinline__ unsigned pack2(float a, float b) {
    bf2 o = {(__bf16)a, (__bf16)b};
    return __builtin_bit_cast(unsigned, o);
}

constexpr u64 make_mc() {
    u64 t = 0;
    for (int d = 0; d < 7; ++d)
        for (int m = 0; m < 4; ++m) {
            int r = 0;
            for (int o = 0; o < 4; ++o) r += (13 - d - ((o * 7) >> 2)) % 7 < (13 - d - ((m * 7) >> 2)) % 7;
            t |= (u64)m << (2 * (d * 4 + r));
        }
    return t;
}
constexpr u64 MC_TAB = make_mc();

}

#define KA "{s[52:59]}"
#define KR "{s51}"
#define KCLOB "s51", "s52", "s53", "s54", "s55", "s56", "s57", "s58", "s59"
#define ARGS() struct { const char* x; const char* wm; char* out; char* myws; } p; \
    p.x = (const char*)(ka[0] | (u64)ka[1] << 32); p.wm = (const char*)(ka[2] | (u64)ka[3] << 32); \
    p.out = (char*)(ka[4] | (u64)ka[5] << 32); p.myws = (char*)(ka[6] | (u64)ka[7] << 32); char* myws = p.myws; (void)myws

__global__ void __attribute__((aligned(KALIGN))) __launch_bounds__(NTHR, 2) a2a_gemm_silu(Params args) {
    __shared__ __attribute__((aligned(16))) char smem[LDS_BYTES];
    u8v ka;
    int rank;
    unsigned ubid = blockIdx.x;
    asm volatile(".Lk_top:\n\ts_load_dwordx8 s[52:59], %1, 0x0\n\ts_load_dword s51, %1, 0x20"
                 "\n\ts_memrealtime s[66:67]"
                 : "+s"(ubid) : "s"(__builtin_amdgcn_kernarg_segment_ptr()) : KCLOB, "s66", "s67");
    const int wv = __builtin_amdgcn_readfirstlane(threadIdx.x >> 6);
    const int bid = ubid & 255;
    const unsigned lds0 = (unsigned)(size_t)smem;
    const int xcd = bid & 7, idx = bid >> 3;
    const int par = xcd & 1, mt = (idx >> XMS) & 3, n8 = (idx & ((1 << XMS) - 1)) | ((idx >> (XMS + 2)) << XMS), cpy = n8;
    int nt = n8;
    nt = (nt & ((1 << XQ0) - 1)) | ((nt >> XQ0) << (XQ0 + 1));
    nt = (nt & ((1 << XQ1) - 1)) | ((nt >> XQ1) << (XQ1 + 1));
    nt |= XQS ? (((xcd >> 1) & 1) << XQ1) | ((xcd >> 2) << XQ0) : (((xcd >> 1) & 1) << XQ0) | ((xcd >> 2) << XQ1);
    if (XX & 1) nt ^= ((nt >> 3) & 1) << 1;
    if (XX & 2) nt ^= ((nt >> 4) & 1) << 1;
    if (XX & 4) nt ^= ((nt >> 4) & 1) << 2;
    if (XX & 8) nt ^= ((nt >> 3) & 1) << 2;
    const int pos = mt;
    const unsigned cb = (unsigned)((nt & ((1 << HB) - 1)) | ((nt >> HB) << (HB + 1))) * 128;

    if (EXP4(wv == 4)) {
        if (!T0TOP) asm volatile("s_memrealtime s[66:67]" ::: "s66", "s67");
        const int pb = PLUT ? (((PLUT >> (4 * (idx & 7))) & 7) * 4 + (idx >> 3)) * 8 + xcd : bid;
        if (NO_PUSH || pb >= 7 * NEARLY) __builtin_amdgcn_s_barrier();
        __builtin_amdgcn_s_setprio(PRIO);
        const int lane = get_lane();
        const int slot = pb / 7, d = pb - 7 * slot;
        if (!NO_PUSH && pb < NPUSH) {
            const int round = slot >> 2;
            const int mc = (int)(MC_TAB >> (2 * (d * 4 + (slot & 3)))) & 3;
            const unsigned odst = (unsigned)(round * 64 + mc * 8 + 4) * 1024;
            int jj = 6 - d - ((mc * 7) >> 2);
            if (jj < 0) jj += 7;
            const unsigned vfl = (unsigned)OFF_FLAGS + mc * 2048 + (lane & 7) * 256 + (7 + 7 * round + jj) * 4;
            const unsigned ocnt = (unsigned)OFF_CNT + bid * 64;
            const unsigned vld = (mc * 128 + (lane & 15)) * (KS * 2) + round * 128 + (lane >> 4) * 16, vst = lane * 16;
            const int tsl = slot ? T_FIRST + slot * T_SLOT : 0;
            const int tsf = slot ? T_FIRST + slot * T_FAST : 0;
            int jh = d - (((slot >> 3) * 7) >> 2);
            if (jh < 0) jh += 7;
            const unsigned ohel = (unsigned)OFF_FLAGS + slot * 256 + (7 * FASTW + jh) * 4;
            asm volatile(
        "v_add_u32 v11, 0x4000, %0\n\t"
        "v_add_u32 v12, 0x8000, %0\n\t"
        "v_add_u32 v13, 0xc000, %0\n\t"
        "v_add_u32 v14, 0x10000, %0\n\t"
        "v_add_u32 v15, 0x14000, %0\n\t"
        "v_add_u32 v16, 0x18000, %0\n\t"
        "v_add_u32 v17, 0x1c000, %0\n\t"
        "s_lshl3_add_u32 s71, %3, 32\n\t"
        "v_mov_b32 v8, %5\n\t"
        "s_waitcnt lgkmcnt(0)\n\t"
        "s_add_u32 s64, s51, %3\n\t"
        "s_and_b32 s64, s64, 7\n\t"
        "s_lshl_b32 s64, s64, 19\n\t"
        "s_add_u32 s64, s52, s64\n\t"
        "s_addc_u32 s65, s53, 0\n\t"
        "s_cmp_ge_u32 %9, %11\n\t"
        "s_cbranch_scc1 .Llate_%=\n\t"
        "global_load_dwordx4 v[32:35], %0, s[64:65] offset:0\n\t"
        "global_load_dwordx4 v[36:39], %0, s[64:65] offset:64\n\t"
        "global_load_dwordx4 v[40:43], v11, s[64:65] offset:0\n\t"
        "global_load_dwordx4 v[44:47], v11, s[64:65] offset:64\n\t"
        "global_load_dwordx4 v[48:51], v12, s[64:65] offset:0\n\t"
        "global_load_dwordx4 v[52:55], v12, s[64:65] offset:64\n\t"
        "global_load_dwordx4 v[56:59], v13, s[64:65] offset:0\n\t"
        "global_load_dwordx4 v[60:63], v13, s[64:65] offset:64\n\t"
        "global_load_dwordx4 v[64:67], v14, s[64:65] offset:0\n\t"
        "global_load_dwordx4 v[68:71], v14, s[64:65] offset:64\n\t"
        "global_load_dwordx4 v[72:75], v15, s[64:65] offset:0\n\t"
        "global_load_dwordx4 v[76:79], v15, s[64:65] offset:64\n\t"
        "global_load_dwordx4 v[80:83], v16, s[64:65] offset:0\n\t"
        "global_load_dwordx4 v[84:87], v16, s[64:65] offset:64\n\t"
        "global_load_dwordx4 v[88:91], v17, s[64:65] offset:0\n\t"
        "global_load_dwordx4 v[92:95], v17, s[64:65] offset:64\n\t"
        "s_load_dword s60, s[58:59], %5 glc\n\t"
        "s_load_dword s61, s[58:59], %6 glc\n\t"
        "s_load_dwordx2 s[78:79], %10, s71\n\t"
        "s_waitcnt vmcnt(0)\n\t"
        "s_barrier\n\t"
        "s_branch .Lgo_%=\n\t"
        ".Llate_%=:\n\t"
        "s_load_dword s60, s[58:59], %5 glc\n\t"
        "s_load_dword s61, s[58:59], %6 glc\n\t"
        "s_load_dwordx2 s[78:79], %10, s71\n\t"
        "s_waitcnt lgkmcnt(0)\n\t"
        "s_cmp_gt_u32 s61, s60\n\t"
        "s_cselect_b32 s70, %8, %7\n\t"
        "s_sub_u32 s70, s70, %12\n\t"
        ".Lwl_%=:\n\t"
        "s_memrealtime s[68:69]\n\t"
        "s_waitcnt lgkmcnt(0)\n\t"
        "s_sub_u32 s68, s68, s66\n\t"
        "s_cmp_ge_i32 s68, s70\n\t"
        "s_cbranch_scc1 .Lwle_%=\n\t"
        "s_sleep 1\n\t"
        "s_branch .Lwl_%=\n\t"
        ".Lwle_%=:\n\t"
        "global_load_dwordx4 v[32:35], %0, s[64:65] offset:0\n\t"
        "global_load_dwordx4 v[36:39], %0, s[64:65] offset:64\n\t"
        "global_load_dwordx4 v[40:43], v11, s[64:65] offset:0\n\t"
        "global_load_dwordx4 v[44:47], v11, s[64:65] offset:64\n\t"
        "global_load_dwordx4 v[48:51], v12, s[64:65] offset:0\n\t"
        "global_load_dwordx4 v[52:55], v12, s[64:65] offset:64\n\t"
        "global_load_dwordx4 v[56:59], v13, s[64:65] offset:0\n\t"
        "global_load_dwordx4 v[60:63], v13, s[64:65] offset:64\n\t"
        "global_load_dwordx4 v[64:67], v14, s[64:65] offset:0\n\t"
        "global_load_dwordx4 v[68:71], v14, s[64:65] offset:64\n\t"
        "global_load_dwordx4 v[72:75], v15, s[64:65] offset:0\n\t"
        "global_load_dwordx4 v[76:79], v15, s[64:65] offset:64\n\t"
        "global_load_dwordx4 v[80:83], v16, s[64:65] offset:0\n\t"
        "global_load_dwordx4 v[84:87], v16, s[64:65] offset:64\n\t"
        "global_load_dwordx4 v[88:91], v17, s[64:65] offset:0\n\t"
        "global_load_dwordx4 v[92:95], v17, s[64:65] offset:64\n\t"
        ".Lgo_%=:\n\t"
        "s_waitcnt lgkmcnt(0)\n\t"
        "s_cmp_gt_u32 s61, s60\n\t"
        "s_cselect_b32 s70, %8, %7\n\t"
        "s_add_u32 s60, s60, 1\n\t"
        "v_mov_b32 v9, s60\n\t"
        "s_and_b32 s60, s60, 1\n\t"
        "s_lshl3_add_u32 s60, s60, s51\n\t"
        "s_lshl_b32 s60, s60, 19\n\t"
        "s_add_u32 s60, s60, %4\n\t"
        "s_add_u32 s62, s78, s60\n\t"
        "s_addc_u32 s63, s79, 0\n\t"
        "s_add_u32 s64, s62, 0x8000\n\t"
        "s_addc_u32 s65, s63, 0\n\t"
        "s_waitcnt vmcnt(0)\n\t"
        ".Lws_%=:\n\t"
        "s_memrealtime s[68:69]\n\t"
        "s_waitcnt lgkmcnt(0)\n\t"
        "s_sub_u32 s68, s68, s66\n\t"
        "s_cmp_ge_i32 s68, s70\n\t"
        "s_cbranch_scc1 .Lwse_%=\n\t"
        "s_sleep 1\n\t"
        "s_branch .Lws_%=\n\t"
        ".Lwse_%=:\n\t"
        "global_store_dwordx4 %1, v[32:35], s[62:63] offset:-4096 sc0 sc1\n\t"
        "global_store_dwordx4 %1, v[36:39], s[64:65] offset:-4096 sc0 sc1\n\t"
        "global_store_dwordx4 %1, v[40:43], s[62:63] offset:-3072 sc0 sc1\n\t"
        "global_store_dwordx4 %1, v[44:47], s[64:65] offset:-3072 sc0 sc1\n\t"
        "global_store_dwordx4 %1, v[48:51], s[62:63] offset:-2048 sc0 sc1\n\t"
        "global_store_dwordx4 %1, v[52:55], s[64:65] offset:-2048 sc0 sc1\n\t"
        "global_store_dwordx4 %1, v[56:59], s[62:63] offset:-1024 sc0 sc1\n\t"
        "global_store_dwordx4 %1, v[60:63], s[64:65] offset:-1024 sc0 sc1\n\t"
        "global_store_dwordx4 %1, v[64:67], s[62:63] offset:0 sc0 sc1\n\t"
        "global_store_dwordx4 %1, v[68:71], s[64:65] offset:0 sc0 sc1\n\t"
        "global_store_dwordx4 %1, v[72:75], s[62:63] offset:1024 sc0 sc1\n\t"
        "global_store_dwordx4 %1, v[76:79], s[64:65] offset:1024 sc0 sc1\n\t"
        "global_store_dwordx4 %1, v[80:83], s[62:63] offset:2048 sc0 sc1\n\t"
        "global_store_dwordx4 %1, v[84:87], s[64:65] offset:2048 sc0 sc1\n\t"
        "global_store_dwordx4 %1, v[88:91], s[62:63] offset:3072 sc0 sc1\n\t"
        "global_store_dwordx4 %1, v[92:95], s[64:65] offset:3072 sc0 sc1\n\t"
        ".Lsk_%=:\n\t"
        "s_waitcnt vmcnt(0)\n\t"
        "s_mov_b64 exec, 0xff\n\t"
        "global_store_dword %2, v9, s[78:79] sc0 sc1\n\t"
        "s_mov_b64 exec, 1\n\t"
        "global_store_dword v8, v9, s[58:59] sc1\n\t"
        "s_mov_b64 exec, -1\n\t"
            :: "v"(vld), "v"(vst), "v"(vfl), "s"(d + 1), "s"(odst), "s"(ocnt), "s"(ohel), "s"(tsl), "s"(tsf), "s"(pb),
               "s"(__builtin_amdgcn_kernarg_segment_ptr()), "n"(7 * NEARLY), "n"(T_LEAD)
            : KCLOB, "v8", "v9", "v10", "v11", "v12", "v13", "v14", "v15", "v16", "v17", "v18", "v19", "v20", "v21", "v22", "v23", "v24", "v25", "v26", "v27", "v28", "v29", "v30", "v31", "v32", "v33", "v34", "v35", "v36", "v37", "v38", "v39", "v40", "v41", "v42", "v43", "v44", "v45", "v46", "v47", "v48", "v49", "v50", "v51", "v52", "v53", "v54", "v55", "v56", "v57", "v58", "v59", "v60", "v61", "v62", "v63", "v64", "v65", "v66", "v67", "v68", "v69", "v70", "v71", "v72", "v73", "v74", "v75", "v76", "v77", "v78", "v79", "v80", "v81", "v82", "v83", "v84", "v85", "v86", "v87", "v88", "v89", "v90", "v91", "v92", "v93", "v94", "v95", "s60", "s61", "s62", "s63", "s64", "s65", "s66", "s67", "s68", "s69", "s70", "s71", "s72", "s73", "s74", "s75", "s76", "s77", "s78", "s79", "scc", "memory");
        } else if (!NO_PUSH && pb < NPUSH + 7) {
            int jj = 6 - d - ((((lane >> 3) & 3) * 7) >> 2);
            if (jj < 0) jj += 7;
            const unsigned vfl = (unsigned)OFF_FLAGS + lane * 256 + jj * 4;
            asm volatile("s_load_dwordx2 s[62:63], %2, %3\n\ts_waitcnt lgkmcnt(0)\n\t"
                         "s_load_dword s60, s[58:59], %1 glc\n\tv_mov_b32 v8, %1\n\ts_waitcnt lgkmcnt(0)\n\t"
                         "s_add_u32 s60, s60, 1\n\tv_mov_b32 v9, s60\n\ts_mov_b32 exec_hi, 0\n\t"
                         "global_store_dword %0, v9, s[62:63] sc0 sc1\n\ts_mov_b32 exec_lo, 1\n\t"
                         "global_store_dword v8, v9, s[58:59] sc1\n\ts_mov_b64 exec, -1"
                         :: "v"(vfl), "s"((unsigned)OFF_CNT + bid * 64), "s"(__builtin_amdgcn_kernarg_segment_ptr()), "s"(40 + 8 * d)
                         : KCLOB, "s66", "s67", "v8", "v9", "s60", "s62", "s63", "scc", "memory");
        } else {
            asm volatile("s_waitcnt lgkmcnt(0)" ::: "memory");
        }
        ENDW();
    }
    if (wv == 5) {
        if (PFIRST & 1) asm volatile("" LATER(1, ".Lk_po0") ::: "memory");
        const int lane = get_lane();
        lds_write(lds0 + LDS_CTRL + lane * 4, lane == 0 ? 8u : 0u);
        lds_write(lds0 + LDS_CONS + (lane & 3) * 256, 0u);
        lds_write(lds0 + LDS_RDY + lane * 4, 0u);
        __builtin_amdgcn_s_setprio(PRIO);
        unsigned cnt;
        asm volatile("s_waitcnt lgkmcnt(0)\n\ts_load_dword %0, s[58:59], %3 glc\n\ts_barrier" LATER(POSUB, ".Lk_poll") "\n\ts_waitcnt lgkmcnt(0)"
                     : "=&s"(cnt), "=&" KA(ka), "=&" KR(rank) : "s"((unsigned)OFF_CNT + bid * 64 + 4) : "memory");
        ARGS();
        char* cntp = myws + OFF_CNT + bid * 64 + 4;
        const unsigned call = cnt + 1;
        if (lane == 0) lds_write(lds0 + LDS_CTRL + 12, call);
        const char* xp = myws + OFF_XF + (bid ^ (XPAIR ? 1 : 8)) * 128;
        const char* fl = myws + OFF_FLAGS + (mt * 8 + cpy) * 256;
        char* tmo = myws + OFF_TMO;
        unsigned xcc;
        asm volatile("s_getreg_b32 %0, hwreg(HW_REG_XCC_ID, 0, 4)" : "=s"(xcc));
        if (lane == 0) store_sys4(myws + OFF_XF + bid * 128 + 16, call * 16 + xcc);
        const Chunk cw = step_chunk(min(max(lane + 1, 8), 63), rank, pos);
        const unsigned vent = ((unsigned)(cw.src * 8 + cw.round) << 16) + ((call & 1) << 22) + 1;
        unsigned vpos = 0;
        constexpr u64 ALLW = 0x7FFFFFFFFFFFFFFFull, LATE = ALLW & ~((1ull << (NA - 1)) - 1), EARLY = ALLW & ~LATE & ~0x7Full;
        const u64 t0 = __builtin_amdgcn_s_memrealtime();
        bool bad = false;
        constexpr u64 FLEX = ALLW & (~0ull << (7 + 7 * DYNR));
        u64 late = DYN ? 0 : LATE;
        u64 arr = NO_POLL ? ~0ull : 1ull << 63, todo = DYN ? EARLY & ~FLEX : EARLY, wait = 0;
        unsigned nseq = 8, xpub = 0, npoll = 0;
        auto assign = [&](u64 c, unsigned room) {
            const unsigned q = __builtin_amdgcn_mbcnt_hi((unsigned)(c >> 32), __builtin_amdgcn_mbcnt_lo((unsigned)c, 0));
            const bool in = __builtin_amdgcn_inverse_ballot_w64(c) && q < room;
            c = __builtin_amdgcn_ballot_w64(in);
            if (in) { vpos = lds0 + LDS_RDY + (nseq + q) * 4; lds_write(vpos + (LDS_SEQ - LDS_RDY), vent);
                if (XH == 2) asm volatile("ds_write2_b32 %0, %1, %2 offset1:1" :: "v"(lds0 + LDS_POS + lane * 8), "v"(vpos), "v"(vent & AMASK) : "memory"); }
            nseq += __builtin_popcountll(c);
            todo &= ~c;
            wait |= c;
        };
        auto release = [&]() {
            if (arr & wait) {
                if (__builtin_amdgcn_inverse_ballot_w64(arr & wait)) lds_write(vpos, vent & AMASK);
                wait &= ~arr;
            }
        };
        for (;;) {
            unsigned xm, tag, wl;
            u64 dec;
#define CMP(r) "s_cmp_ge_u32 s" #r ", %[call]\n\ts_addc_u32 s26, s26, s26\n\t"
#define CMP4(r0, r1, r2, r3) CMP(r0) CMP(r1) CMP(r2) CMP(r3)
#define MERGE(w) "s_lshl_b64 s[26:27], s[26:27], " w "\n\ts_or_b64 %0, %0, s[26:27]\n\ts_mov_b64 s[26:27], 0\n\t"
#define NEXTW(wp, wn, regs, L) "s_add_u32 s35, " wp ", 16\n\ts_cmp_ge_u32 s35, 63\n\ts_cbranch_scc1 " L "\n\t" \
                         "s_lshr_b64 s[26:27], s[24:25], s35\n\ts_cbranch_scc0 " L "\n\t" \
                         "s_ff1_i32_b64 " wn ", s[26:27]\n\ts_add_u32 " wn ", " wn ", s35\n\ts_lshl_b32 s35, " wn ", 2\n\t" \
                         "s_load_dwordx16 " regs ", %[fl], s35 glc\n\t"
            asm volatile("s_not_b64 s[24:25], %0\n\ts_ff1_i32_b64 s84, s[24:25]\n\ts_and_b32 s84, s84, 63\n\ts_lshl_b32 s35, s84, 2\n\t"
                         "s_load_dwordx16 s[36:51], %[fl], s35 glc\n\t"
                         "s_load_dwordx4 s[28:31], %[xp], 0x0 glc\n\ts_load_dword %2, %[xp], 0x10 glc\n\t"
                         "s_lshl_b64 s[26:27], %0, 7\n\ts_and_b64 s[24:25], s[24:25], s[26:27]\n\t"
                         "s_mov_b32 s85, 0\n\ts_mov_b32 s86, 0\n\t"
                         NEXTW("s84", "s85", "s[52:67]", ".La_%=") NEXTW("s85", "s86", "s[68:83]", ".La_%=")
                         ".La_%=:\n\ts_mov_b64 s[26:27], 0\n\ts_waitcnt lgkmcnt(0)\n\tds_read_b32 %3, %[ctl] offset:4\n\t"
                         CMP4(51, 50, 49, 48) CMP4(47, 46, 45, 44) CMP4(43, 42, 41, 40) CMP4(39, 38, 37, 36) MERGE("s84")
                         "s_cmp_eq_u32 s85, 0\n\ts_cbranch_scc1 .Lb_%=\n\t"
                         CMP4(67, 66, 65, 64) CMP4(63, 62, 61, 60) CMP4(59, 58, 57, 56) CMP4(55, 54, 53, 52) MERGE("s85")
                         "s_cmp_eq_u32 s86, 0\n\ts_cbranch_scc1 .Lb_%=\n\t"
                         CMP4(83, 82, 81, 80) CMP4(79, 78, 77, 76) CMP4(75, 74, 73, 72) CMP4(71, 70, 69, 68) MERGE("s86")
                         ".Lb_%=:\n\ts_min_u32 s28, s28, s29\n\ts_min_u32 s30, s30, s31\n\ts_min_u32 %1, s28, s30\n\t"
                         "s_waitcnt lgkmcnt(0)"
                         : "+s"(arr), "=&s"(xm), "=&s"(tag), "=&v"(wl), [dec] "=&s"(dec)
                         : [call] "s"(call), [xp] "s"(xp), [fl] "s"(fl), [ctl] "v"(lds0 + LDS_CTRL)
                         : "s24", "s25", "s26", "s27", "s28", "s29", "s30", "s31", "s35", "s36", "s37", "s38", "s39", "s40", "s41", "s42", "s43", "s44", "s45", "s46", "s47", "s48", "s49", "s50", "s51", "s52", "s53", "s54", "s55", "s56", "s57", "s58", "s59", "s60", "s61", "s62", "s63", "s64", "s65", "s66", "s67", "s68", "s69", "s70", "s71", "s72", "s73", "s74", "s75", "s76", "s77", "s78", "s79", "s80", "s81", "s82", "s83", "s84", "s85", "s86", "scc", "memory");
            if (bad) arr = ~0ull;
            release();
            if (nseq < 64) {
                const unsigned n0 = nseq, lim = __builtin_amdgcn_readfirstlane(wl) + LOOKA;
                const u64 h = arr & 0x7F, a = arr & ALLW & ~0x7Full;
#pragma unroll
                for (int pass = 0; pass < 2; ++pass) {
                    if (arr & todo) assign(arr & todo, 64);
                    if (nseq < lim && ((a << 7) & todo)) assign((a << 7) & todo, lim - nseq);
                    if (nseq < lim && (((a << 14) | (h << 7)) & todo)) assign(((a << 14) | (h << 7)) & todo, lim - nseq);
                    if (SPEC > 2 && nseq < lim && (((a << 21) | (h << 14)) & todo)) assign(((a << 21) | (h << 14)) & todo, lim - nseq);
                    if (todo || nseq != NA) break;
                    todo = late;
                }
                if (nseq != n0 && lane == 0) lds_write(lds0 + LDS_CTRL, nseq);
                release();
            }
            unsigned xi = 0;
            if ((tag >> 4) == (call & 0x0FFFFFFFu)) {
                xi = (tag & 15) == xcc ? 2u : 1u;
                if (xm >= call) xi += 4u;
            }
            if (bad) xi = 5u;
            if (xi != xpub) {
                if (lane == 0) lds_write(lds0 + LDS_CTRL + 8, xi);
                xpub = xi;
            }
            if (nseq == 64 && !wait && xi >= 4) break;
            if ((++npoll & 1023) == 0) {
                const unsigned tv = __builtin_amdgcn_readfirstlane(__hip_atomic_load((unsigned*)tmo, __ATOMIC_RELAXED, __HIP_MEMORY_SCOPE_AGENT));
                if (tv == call || __builtin_amdgcn_s_memrealtime() - t0 > DEADLINE_TICKS) {
                    if (lane == 0) __hip_atomic_store((unsigned*)tmo, call, __ATOMIC_RELAXED, __HIP_MEMORY_SCOPE_AGENT);
                    if (lane == 0) lds_write(lds0 + LDS_CTRL + 16, 1u);
                    asm volatile("s_waitcnt lgkmcnt(0)" ::: "memory");
                    bad = true;
                }
            }
            __builtin_amdgcn_s_sleep(POLL_SLEEP);
        }
        if (lane == 0) asm volatile("global_store_dword %0, %1, off sc1" :: "v"(cntp), "v"(call) : "memory");
        ENDW();
    }
    if (wv == 6) {
        if (PFIRST & 2) asm volatile("" LATER(1, ".Lk_wl0") ::: "memory");
        const int lane = get_lane();
        asm volatile("s_getpc_b64 s[60:61]\n\ts_add_u32 s60, s60, .Lk_top@rel32@lo+4\n\ts_addc_u32 s61, s61, .Lk_top@rel32@hi+12\n\t"
                     "global_load_dword v200, %0, s[60:61]" :: "v"((lane + (idx & ((512 >> (CPS - 6)) / 64 - 1)) * 64) << CPS) : "s60", "s61", "scc", "v200", "memory");
        __builtin_amdgcn_s_setprio(LPRIO);
#define STP
#define STQ
#define STCL
        unsigned va[8], vb[8];
#pragma unroll
        for (int j = 0; j < 8; ++j) {
            const int row = 4 * j + (lane >> 4);
            const int su = ((lane & 15) >> 1) ^ (((j >> 1) & 1) * 4 + (lane >> 4));
            va[j] = (unsigned)row * WRS + (cb ^ (((su >> 2) ^ par) * (HM * 128))) + (su & 3) * 32 + (lane & 1) * 16 - (j & 3) * 1024 + 4096;
        }
        asm volatile("s_mov_b32 m0, %11\n\ts_waitcnt lgkmcnt(0)\n\t" STP
                     "s_lshl4_add_u32 s60, s51, %10\n\ts_lshl_b32 s60, s60, 18\n\ts_add_u32 s60, s54, s60\n\ts_addc_u32 s61, s55, 0\n\t"
#define TILE0 "global_load_lds_dwordx4 %2, s[60:61]\n\tglobal_load_lds_dwordx4 %3, s[60:61] offset:1024\n\t" \
                     "global_load_lds_dwordx4 %4, s[60:61] offset:2048\n\tglobal_load_lds_dwordx4 %5, s[60:61] offset:3072\n\t" \
                     "s_add_u32 m0, m0, 0x1000\n\ts_nop 0\n\t" \
                     "global_load_lds_dwordx4 %6, s[60:61]\n\tglobal_load_lds_dwordx4 %7, s[60:61] offset:1024\n\t" \
                     "global_load_lds_dwordx4 %8, s[60:61] offset:2048\n\tglobal_load_lds_dwordx4 %9, s[60:61] offset:3072\n\t"
#define TILE1 "s_add_u32 m0, m0, 0x1000\n\ts_add_u32 s60, s60, 0x80000\n\ts_addc_u32 s61, s61, 0\n\t" TILE0
                     TILE0
                     "s_barrier" STQ LATER(WSUB, ".Lk_wld")
                     : "=&" KA(ka), "=&" KR(rank)
                     : "v"(va[0]), "v"(va[1]), "v"(va[2]), "v"(va[3]), "v"(va[4]), "v"(va[5]), "v"(va[6]), "v"(va[7]),
                       "s"(((2 * pos) & 7) * 2 + par), "s"(lds0)
                     : STCL "s60", "s61", "scc", "memory");
        ARGS();
        const Chunk ch = step_chunk(lane & 7, rank, pos);
        const unsigned vpar = lane < NA ? par : 0;
        unsigned vTab = ch.src * 16 + ch.round * 2 + par;
        const char* wb = p.wm;
        const unsigned caddr = lane < 4 ? lds0 + LDS_CONS + lane * 256 : lds0 + LDS_CTRL, saddr = lds0 + LDS_SEQ + lane * 4;
        const bool slow = ((((unsigned)(size_t)wb + cb) >> 8) & 3) == 1;
        const int fly = slow ? WFLY2 : WFLY;
        unsigned pub = 0, freed = 8;
        const Chunk cp = step_chunk(max(lane, 8), rank, pos);
        const unsigned vPf = cp.src * 16 + cp.round * 2 + vpar;
        unsigned vpo = (lane >> 1) * WRS + cb + (lane & 1) * 128 + 4096;
        u64 pfm = __builtin_amdgcn_ballot_w64(lane >= 8 && (PFN || lane < NA) && ((cp.src + cp.round) & 3) == mt);
        unsigned pft = 0;
        auto tile = [&](int t, const unsigned* v) {
            if ((unsigned)t >= freed) {
                for (;;) {
                    unsigned c, e;
                    asm volatile("ds_read_b32 %0, %2\n\tds_read_b32 %1, %3\n\ts_waitcnt lgkmcnt(0)" : "=&v"(c), "=&v"(e) : "v"(caddr), "v"(saddr) : "memory");
                    const unsigned m = min(min(__builtin_amdgcn_readlane(c, 0), __builtin_amdgcn_readlane(c, 1)),
                                           min(__builtin_amdgcn_readlane(c, 2), __builtin_amdgcn_readlane(c, 3)));
                    freed = min(m + WLEAD, (unsigned)__builtin_amdgcn_readlane(c, 4));
                    vTab = ((e >> 15) & 126) + vpar;
                    if ((unsigned)t < freed) break;
                    if (pub < (unsigned)t) {
                        WAITV(0);
                        pub = t;
                        if (lane == 0) lds_write(lds0 + LDS_CTRL + 4, pub);
                    } else {
                        const unsigned now = (unsigned)__builtin_amdgcn_s_memrealtime();
                        pfm &= ~0ull << min(max((unsigned)__builtin_amdgcn_readlane(c, 4), PFS0) + PFS, 63u);
                        if (pfm && now - pft >= PFT) {
                            const int n = __builtin_ctzll(pfm);
                            if (n <= t + PFMAX) {
                                if (PFN && n >= NA) vpo = lane * WRS + (cb ^ (par * 128)) + 4096;
                                const char* b = wb + ((size_t)__builtin_amdgcn_readlane(vPf, n) << 18);
                                asm volatile("global_load_dword v255, %0, %1 " PFFL :: "v"(vpo), "s"(b) : "v255", "memory");
                                pfm &= pfm - 1;
                                pft = now;
                            }
                        }
                        __builtin_amdgcn_s_sleep(1);
                    }
                }
            }
            const char* b = wb + ((size_t)(__builtin_amdgcn_readlane(vTab, t) & WMASK) << 18);
            const unsigned slot = lds0 + (t & (RING - 1)) * 8192;
            dma4(v, b, slot);
            dma4(v + 4, b, slot + 4096);
            if (t + 2 > fly + (int)pub) {
                if (WFLY2 != WFLY && slow) wait_vm<8 * (WFLY2 - 1)>(); else wait_vm<8 * (WFLY - 1)>();
                pub = t + 2 - fly;
                if (lane == 0) lds_write(lds0 + LDS_CTRL + 4, pub);
            }
        };
        for (int t = WPRE > 1 ? 2 : 1; t < NA; ++t) tile(t, va);
#pragma unroll
        for (int j = 0; j < 8; ++j) {
            const int row = 4 * j + (lane >> 4);
            const int su = ((lane & 15) >> 1) ^ (((j >> 1) & 1) * 4 + (lane >> 4));
            vb[j] = (unsigned)row * WRS + (cb ^ (par * (HM * 128))) + (su >> 2) * (32 * NN * 2) + (su & 3) * 32 + (lane & 1) * 16 - (j & 3) * 1024 + 4096;
        }
        for (int t = NA; t < 64; ++t) tile(t, vb);
        WAITV(0);
        if (lane == 0) lds_write(lds0 + LDS_CTRL + 4, 64u);
        ENDW();
    }

    {
        if (PFIRST & 4) asm volatile("" LATER(1, ".Lk_ge0") ::: "memory");
        const int lane = get_lane();
        const int g = lane >> 4, q = (lane & 15) >> 2, pp = lane & 3;
        const unsigned vTr = lds0 + (unsigned)((8 * g + q) * 256 + (((g & 1) * 4 + q) * 32) + pp * 8);
        const unsigned voffA = lane * 16;
        const unsigned vRow = (mt * 128 + wv * 32 + (lane & 15)) * (KS * 2) + par * 64 + g * 16;
        const unsigned vCons = lds0 + LDS_CONS + wv * 256 + lane * 4;
        const unsigned vOut = (mt * 128 + wv * 32 + (lane & 15)) * (NN * 2) + (cb ^ (par * (HM * 128))) + (g & 1) * 32 + (g >> 1) * 16;
        asm volatile(
        "s_add_u32 s60, %10, 0\n\t"
        "s_and_b32 s60, s60, 3\n\t"
        "s_lshl_b32 s60, s60, 8\n\t"
        "v_add_u32 v116, s60, %3\n\t"
        "s_add_u32 s60, %10, 1\n\t"
        "s_and_b32 s60, s60, 3\n\t"
        "s_lshl_b32 s60, s60, 8\n\t"
        "v_add_u32 v117, s60, %3\n\t"
        "s_add_u32 s60, %10, 2\n\t"
        "s_and_b32 s60, s60, 3\n\t"
        "s_lshl_b32 s60, s60, 8\n\t"
        "v_add_u32 v118, s60, %3\n\t"
        "s_add_u32 s60, %10, 3\n\t"
        "s_and_b32 s60, s60, 3\n\t"
        "s_lshl_b32 s60, s60, 8\n\t"
        "v_add_u32 v119, s60, %3\n\t"
        "v_mov_b32 v8, %1\n\t"
        "v_xor_b32 v9, 32, v8\n\t"
        "v_xor_b32 v10, 64, v8\n\t"
        "v_xor_b32 v11, 96, v8\n\t"
        "v_xor_b32 v12, 128, v8\n\t"
        "v_xor_b32 v13, 160, v8\n\t"
        "v_xor_b32 v14, 192, v8\n\t"
        "v_xor_b32 v15, 224, v8\n\t"
        "v_add_u32 v16, 0x10000, v8\n\t"
        "v_add_u32 v17, 0x10000, v9\n\t"
        "v_add_u32 v18, 0x10000, v10\n\t"
        "v_add_u32 v19, 0x10000, v11\n\t"
        "v_add_u32 v20, 0x10000, v12\n\t"
        "v_add_u32 v21, 0x10000, v13\n\t"
        "v_add_u32 v22, 0x10000, v14\n\t"
        "v_add_u32 v23, 0x10000, v15\n\t"
        "v_mov_b32 v31, 1\n\t"
        "v_mov_b32 v127, %9\n\t"
        "s_waitcnt lgkmcnt(0)\n\t"
        "s_lshl_b32 s60, s51, 19\n\t"
        "s_add_u32 s62, s52, s60\n\t"
        "s_addc_u32 s63, s53, 0\n\t"
        "s_add_u32 s72, s62, 0x4000\n\t"
        "s_addc_u32 s73, s63, 0\n\t"
        "global_load_dwordx4 v[32:35], v116, s[62:63] offset:0\n\t"
        "global_load_dwordx4 v[36:39], v116, s[72:73] offset:0\n\t"
        "s_barrier\n\t"
        "global_load_dwordx4 v[40:43], v116, s[62:63] offset:128\n\t"
        "global_load_dwordx4 v[44:47], v116, s[72:73] offset:128\n\t"
        "global_load_dwordx4 v[48:51], v117, s[62:63] offset:0\n\t"
        "global_load_dwordx4 v[52:55], v117, s[72:73] offset:0\n\t"
        "global_load_dwordx4 v[56:59], v117, s[62:63] offset:128\n\t"
        "global_load_dwordx4 v[60:63], v117, s[72:73] offset:128\n\t"
        "s_branch .Lbody_%=\n\t"
        ".subsection 6\n\t"
        ".Lbody_%=:\n\t"
        "s_add_u32 s66, s58, %6\n\t"
        "s_addc_u32 s67, s59, 0\n\t"
        "s_sub_u32 s66, s66, 1\n\t"
        "s_subb_u32 s67, s67, 0\n\t"
        "s_add_u32 s64, s66, 0x8000\n\t"
        "s_addc_u32 s65, s67, 0\n\t"
        "s_add_u32 s68, s66, %11\n\t"
        "s_addc_u32 s69, s67, 0\n\t"
        "s_add_u32 s92, s58, %7\n\t"
        "s_addc_u32 s93, s59, 0\n\t"
        "s_xor_b32 s60, %7, 0x8000\n\t"
        "s_add_u32 s94, s58, s60\n\t"
        "s_addc_u32 s95, s59, 0\n\t"
        "s_add_u32 s96, s58, %8\n\t"
        "s_addc_u32 s97, s59, 0\n\t"
        "v_mov_b32 v192, 0\n\t"
        "v_mov_b32 v193, 0\n\t"
        "v_mov_b32 v194, 0\n\t"
        "v_mov_b32 v195, 0\n\t"
        "v_mov_b32 v196, 0\n\t"
        "v_mov_b32 v197, 0\n\t"
        "v_mov_b32 v198, 0\n\t"
        "v_mov_b32 v199, 0\n\t"
        "v_mov_b32 v200, 0\n\t"
        "v_mov_b32 v201, 0\n\t"
        "v_mov_b32 v202, 0\n\t"
        "v_mov_b32 v203, 0\n\t"
        "v_mov_b32 v204, 0\n\t"
        "v_mov_b32 v205, 0\n\t"
        "v_mov_b32 v206, 0\n\t"
        "v_mov_b32 v207, 0\n\t"
        "v_mov_b32 v208, 0\n\t"
        "v_mov_b32 v209, 0\n\t"
        "v_mov_b32 v210, 0\n\t"
        "v_mov_b32 v211, 0\n\t"
        "v_mov_b32 v212, 0\n\t"
        "v_mov_b32 v213, 0\n\t"
        "v_mov_b32 v214, 0\n\t"
        "v_mov_b32 v215, 0\n\t"
        "v_mov_b32 v216, 0\n\t"
        "v_mov_b32 v217, 0\n\t"
        "v_mov_b32 v218, 0\n\t"
        "v_mov_b32 v219, 0\n\t"
        "v_mov_b32 v220, 0\n\t"
        "v_mov_b32 v221, 0\n\t"
        "v_mov_b32 v222, 0\n\t"
        "v_mov_b32 v223, 0\n\t"
        "v_mov_b32 v224, 0\n\t"
        "v_mov_b32 v225, 0\n\t"
        "v_mov_b32 v226, 0\n\t"
        "v_mov_b32 v227, 0\n\t"
        "v_mov_b32 v228, 0\n\t"
        "v_mov_b32 v229, 0\n\t"
        "v_mov_b32 v230, 0\n\t"
        "v_mov_b32 v231, 0\n\t"
        "v_mov_b32 v232, 0\n\t"
        "v_mov_b32 v233, 0\n\t"
        "v_mov_b32 v234, 0\n\t"
        "v_mov_b32 v235, 0\n\t"
        "v_mov_b32 v236, 0\n\t"
        "v_mov_b32 v237, 0\n\t"
        "v_mov_b32 v238, 0\n\t"
        "v_mov_b32 v239, 0\n\t"
        "v_mov_b32 v240, 0\n\t"
        "v_mov_b32 v241, 0\n\t"
        "v_mov_b32 v242, 0\n\t"
        "v_mov_b32 v243, 0\n\t"
        "v_mov_b32 v244, 0\n\t"
        "v_mov_b32 v245, 0\n\t"
        "v_mov_b32 v246, 0\n\t"
        "v_mov_b32 v247, 0\n\t"
        "v_mov_b32 v248, 0\n\t"
        "v_mov_b32 v249, 0\n\t"
        "v_mov_b32 v250, 0\n\t"
        "v_mov_b32 v251, 0\n\t"
        "v_mov_b32 v252, 0\n\t"
        "v_mov_b32 v253, 0\n\t"
        "v_mov_b32 v254, 0\n\t"
        "v_mov_b32 v255, 0\n\t"
        "ds_read_b128 v[120:123], v127\n\t"
        "s_waitcnt lgkmcnt(0)\n\t"
        "s_movk_i32 s61, 0\n\t"
        "v_cmp_ge_u32 vcc, s61, v121\n\t"
        "s_getpc_b64 s[70:71]\n\t"
        "s_cbranch_vccnz .Lspin1_%=\n\t"
        "ds_read_b64_tr_b16 v[128:129], v8 offset:0\n\t"
        "ds_read_b64_tr_b16 v[130:131], v8 offset:1024\n\t"
        "ds_read_b64_tr_b16 v[132:133], v9 offset:0\n\t"
        "ds_read_b64_tr_b16 v[134:135], v9 offset:1024\n\t"
        "ds_read_b64_tr_b16 v[136:137], v10 offset:0\n\t"
        "ds_read_b64_tr_b16 v[138:139], v10 offset:1024\n\t"
        "ds_read_b64_tr_b16 v[140:141], v11 offset:0\n\t"
        "ds_read_b64_tr_b16 v[142:143], v11 offset:1024\n\t"
        "ds_read_b64_tr_b16 v[144:145], v12 offset:0\n\t"
        "ds_read_b64_tr_b16 v[146:147], v12 offset:1024\n\t"
        "ds_read_b64_tr_b16 v[148:149], v13 offset:0\n\t"
        "ds_read_b64_tr_b16 v[150:151], v13 offset:1024\n\t"
        "ds_read_b64_tr_b16 v[152:153], v14 offset:0\n\t"
        "ds_read_b64_tr_b16 v[154:155], v14 offset:1024\n\t"
        "ds_read_b64_tr_b16 v[156:157], v15 offset:0\n\t"
        "ds_read_b64_tr_b16 v[158:159], v15 offset:1024\n\t"
        "s_movk_i32 s61, 1\n\t"
        "v_cmp_ge_u32 vcc, s61, v121\n\t"
        "s_getpc_b64 s[70:71]\n\t"
        "s_cbranch_vccnz .Lspin1_%=\n\t"
        "ds_read_b32 v121, v127 offset:4\n\t"
        "s_waitcnt vmcnt(6) lgkmcnt(13)\n\t"
        "v_mfma_f32_16x16x32_bf16 v[192:195], v[128:131], v[32:35], v[192:195]\n\t"
        "ds_read_b64_tr_b16 v[160:161], v8 offset:8192\n\t"
        "v_mfma_f32_16x16x32_bf16 v[208:211], v[128:131], v[36:39], v[208:211]\n\t"
        "ds_read_b64_tr_b16 v[162:163], v8 offset:9216\n\t"
        "v_mfma_f32_16x16x32_bf16 v[196:199], v[132:135], v[32:35], v[196:199]\n\t"
        "ds_read_b64_tr_b16 v[164:165], v9 offset:8192\n\t"
        "v_mfma_f32_16x16x32_bf16 v[212:215], v[132:135], v[36:39], v[212:215]\n\t"
        "ds_read_b64_tr_b16 v[166:167], v9 offset:9216\n\t"
        "s_waitcnt lgkmcnt(4)\n\t"
        "v_mfma_f32_16x16x32_bf16 v[200:203], v[136:139], v[32:35], v[200:203]\n\t"
        "ds_read_b64_tr_b16 v[168:169], v10 offset:8192\n\t"
        "s_movk_i32 s61, 2\n\t"
        "v_mfma_f32_16x16x32_bf16 v[216:219], v[136:139], v[36:39], v[216:219]\n\t"
        "ds_read_b64_tr_b16 v[170:171], v10 offset:9216\n\t"
        "v_cmp_ge_u32 vcc, s61, v121\n\t"
        "v_mfma_f32_16x16x32_bf16 v[204:207], v[140:143], v[32:35], v[204:207]\n\t"
        "ds_read_b64_tr_b16 v[172:173], v11 offset:8192\n\t"
        "s_getpc_b64 s[70:71]\n\t"
        "s_cbranch_vccnz .Lspin1_%=\n\t"
        "v_mfma_f32_16x16x32_bf16 v[220:223], v[140:143], v[36:39], v[220:223]\n\t"
        "ds_read_b64_tr_b16 v[174:175], v11 offset:9216\n\t"
        "ds_read_b32 v121, v127 offset:4\n\t"
        "s_waitcnt lgkmcnt(14)\n\t"
        "v_mfma_f32_16x16x32_bf16 v[224:227], v[144:147], v[32:35], v[224:227]\n\t"
        "ds_read_b64_tr_b16 v[176:177], v12 offset:8192\n\t"
        "global_load_dwordx4 v[64:67], v118, s[62:63] offset:0\n\t"
        "v_mfma_f32_16x16x32_bf16 v[240:243], v[144:147], v[36:39], v[240:243]\n\t"
        "ds_read_b64_tr_b16 v[178:179], v12 offset:9216\n\t"
        "global_load_dwordx4 v[68:71], v118, s[72:73] offset:0\n\t"
        "v_mfma_f32_16x16x32_bf16 v[228:231], v[148:151], v[32:35], v[228:231]\n\t"
        "ds_read_b64_tr_b16 v[180:181], v13 offset:8192\n\t"
        "v_mfma_f32_16x16x32_bf16 v[244:247], v[148:151], v[36:39], v[244:247]\n\t"
        "ds_read_b64_tr_b16 v[182:183], v13 offset:9216\n\t"
        "s_waitcnt lgkmcnt(14)\n\t"
        "v_mfma_f32_16x16x32_bf16 v[232:235], v[152:155], v[32:35], v[232:235]\n\t"
        "ds_read_b64_tr_b16 v[184:185], v14 offset:8192\n\t"
        "v_mfma_f32_16x16x32_bf16 v[248:251], v[152:155], v[36:39], v[248:251]\n\t"
        "ds_read_b64_tr_b16 v[186:187], v14 offset:9216\n\t"
        "ds_add_u32 %4, v31\n\t"
        "v_mfma_f32_16x16x32_bf16 v[236:239], v[156:159], v[32:35], v[236:239]\n\t"
        "ds_read_b64_tr_b16 v[188:189], v15 offset:8192\n\t"
        "v_mfma_f32_16x16x32_bf16 v[252:255], v[156:159], v[36:39], v[252:255]\n\t"
        "ds_read_b64_tr_b16 v[190:191], v15 offset:9216\n\t"
        "s_waitcnt vmcnt(6) lgkmcnt(14)\n\t"
        "v_mfma_f32_16x16x32_bf16 v[192:195], v[160:163], v[40:43], v[192:195]\n\t"
        "ds_read_b64_tr_b16 v[128:129], v8 offset:16384\n\t"
        "v_mfma_f32_16x16x32_bf16 v[208:211], v[160:163], v[44:47], v[208:211]\n\t"
        "ds_read_b64_tr_b16 v[130:131], v8 offset:17408\n\t"
        "v_mfma_f32_16x16x32_bf16 v[196:199], v[164:167], v[40:43], v[196:199]\n\t"
        "ds_read_b64_tr_b16 v[132:133], v9 offset:16384\n\t"
        "v_mfma_f32_16x16x32_bf16 v[212:215], v[164:167], v[44:47], v[212:215]\n\t"
        "ds_read_b64_tr_b16 v[134:135], v9 offset:17408\n\t"
        "s_waitcnt lgkmcnt(13)\n\t"
        "v_mfma_f32_16x16x32_bf16 v[200:203], v[168:171], v[40:43], v[200:203]\n\t"
        "ds_read_b64_tr_b16 v[136:137], v10 offset:16384\n\t"
        "s_movk_i32 s61, 3\n\t"
        "v_mfma_f32_16x16x32_bf16 v[216:219], v[168:171], v[44:47], v[216:219]\n\t"
        "ds_read_b64_tr_b16 v[138:139], v10 offset:17408\n\t"
        "v_cmp_ge_u32 vcc, s61, v121\n\t"
        "v_mfma_f32_16x16x32_bf16 v[204:207], v[172:175], v[40:43], v[204:207]\n\t"
        "ds_read_b64_tr_b16 v[140:141], v11 offset:16384\n\t"
        "s_getpc_b64 s[70:71]\n\t"
        "s_cbranch_vccnz .Lspin1_%=\n\t"
        "v_mfma_f32_16x16x32_bf16 v[220:223], v[172:175], v[44:47], v[220:223]\n\t"
        "ds_read_b64_tr_b16 v[142:143], v11 offset:17408\n\t"
        "ds_read_b32 v121, v127 offset:4\n\t"
        "s_waitcnt lgkmcnt(14)\n\t"
        "v_mfma_f32_16x16x32_bf16 v[224:227], v[176:179], v[40:43], v[224:227]\n\t"
        "ds_read_b64_tr_b16 v[144:145], v12 offset:16384\n\t"
        "global_load_dwordx4 v[72:75], v118, s[62:63] offset:128\n\t"
        "v_mfma_f32_16x16x32_bf16 v[240:243], v[176:179], v[44:47], v[240:243]\n\t"
        "ds_read_b64_tr_b16 v[146:147], v12 offset:17408\n\t"
        "global_load_dwordx4 v[76:79], v118, s[72:73] offset:128\n\t"
        "v_mfma_f32_16x16x32_bf16 v[228:231], v[180:183], v[40:43], v[228:231]\n\t"
        "ds_read_b64_tr_b16 v[148:149], v13 offset:16384\n\t"
        "v_mfma_f32_16x16x32_bf16 v[244:247], v[180:183], v[44:47], v[244:247]\n\t"
        "ds_read_b64_tr_b16 v[150:151], v13 offset:17408\n\t"
        "s_waitcnt lgkmcnt(13)\n\t"
        "v_mfma_f32_16x16x32_bf16 v[232:235], v[184:187], v[40:43], v[232:235]\n\t"
        "ds_read_b64_tr_b16 v[152:153], v14 offset:16384\n\t"
        "v_mfma_f32_16x16x32_bf16 v[248:251], v[184:187], v[44:47], v[248:251]\n\t"
        "ds_read_b64_tr_b16 v[154:155], v14 offset:17408\n\t"
        "ds_add_u32 %4, v31\n\t"
        "v_mfma_f32_16x16x32_bf16 v[236:239], v[188:191], v[40:43], v[236:239]\n\t"
        "ds_read_b64_tr_b16 v[156:157], v15 offset:16384\n\t"
        "v_mfma_f32_16x16x32_bf16 v[252:255], v[188:191], v[44:47], v[252:255]\n\t"
        "ds_read_b64_tr_b16 v[158:159], v15 offset:17408\n\t"
        "s_waitcnt vmcnt(6) lgkmcnt(14)\n\t"
        "v_mfma_f32_16x16x32_bf16 v[192:195], v[128:131], v[48:51], v[192:195]\n\t"
        "ds_read_b64_tr_b16 v[160:161], v8 offset:24576\n\t"
        "v_mfma_f32_16x16x32_bf16 v[208:211], v[128:131], v[52:55], v[208:211]\n\t"
        "ds_read_b64_tr_b16 v[162:163], v8 offset:25600\n\t"
        "v_mfma_f32_16x16x32_bf16 v[196:199], v[132:135], v[48:51], v[196:199]\n\t"
        "ds_read_b64_tr_b16 v[164:165], v9 offset:24576\n\t"
        "v_mfma_f32_16x16x32_bf16 v[212:215], v[132:135], v[52:55], v[212:215]\n\t"
        "ds_read_b64_tr_b16 v[166:167], v9 offset:25600\n\t"
        "s_waitcnt lgkmcnt(13)\n\t"
        "v_mfma_f32_16x16x32_bf16 v[200:203], v[136:139], v[48:51], v[200:203]\n\t"
        "ds_read_b64_tr_b16 v[168:169], v10 offset:24576\n\t"
        "s_movk_i32 s61, 4\n\t"
        "v_mfma_f32_16x16x32_bf16 v[216:219], v[136:139], v[52:55], v[216:219]\n\t"
        "ds_read_b64_tr_b16 v[170:171], v10 offset:25600\n\t"
        "v_cmp_ge_u32 vcc, s61, v121\n\t"
        "v_mfma_f32_16x16x32_bf16 v[204:207], v[140:143], v[48:51], v[204:207]\n\t"
        "ds_read_b64_tr_b16 v[172:173], v11 offset:24576\n\t"
        "s_getpc_b64 s[70:71]\n\t"
        "s_cbranch_vccnz .Lspin1_%=\n\t"
        "v_mfma_f32_16x16x32_bf16 v[220:223], v[140:143], v[52:55], v[220:223]\n\t"
        "ds_read_b64_tr_b16 v[174:175], v11 offset:25600\n\t"
        "ds_read_b32 v121, v127 offset:4\n\t"
        "s_waitcnt lgkmcnt(14)\n\t"
        "v_mfma_f32_16x16x32_bf16 v[224:227], v[144:147], v[48:51], v[224:227]\n\t"
        "ds_read_b64_tr_b16 v[176:177], v12 offset:24576\n\t"
        "global_load_dwordx4 v[80:83], v119, s[62:63] offset:0\n\t"
        "v_mfma_f32_16x16x32_bf16 v[240:243], v[144:147], v[52:55], v[240:243]\n\t"
        "ds_read_b64_tr_b16 v[178:179], v12 offset:25600\n\t"
        "global_load_dwordx4 v[84:87], v119, s[72:73] offset:0\n\t"
        "v_mfma_f32_16x16x32_bf16 v[228:231], v[148:151], v[48:51], v[228:231]\n\t"
        "ds_read_b64_tr_b16 v[180:181], v13 offset:24576\n\t"
        "v_mfma_f32_16x16x32_bf16 v[244:247], v[148:151], v[52:55], v[244:247]\n\t"
        "ds_read_b64_tr_b16 v[182:183], v13 offset:25600\n\t"
        "s_waitcnt lgkmcnt(13)\n\t"
        "v_mfma_f32_16x16x32_bf16 v[232:235], v[152:155], v[48:51], v[232:235]\n\t"
        "ds_read_b64_tr_b16 v[184:185], v14 offset:24576\n\t"
        "v_mfma_f32_16x16x32_bf16 v[248:251], v[152:155], v[52:55], v[248:251]\n\t"
        "ds_read_b64_tr_b16 v[186:187], v14 offset:25600\n\t"
        "ds_add_u32 %4, v31\n\t"
        "v_mfma_f32_16x16x32_bf16 v[236:239], v[156:159], v[48:51], v[236:239]\n\t"
        "ds_read_b64_tr_b16 v[188:189], v15 offset:24576\n\t"
        "v_mfma_f32_16x16x32_bf16 v[252:255], v[156:159], v[52:55], v[252:255]\n\t"
        "ds_read_b64_tr_b16 v[190:191], v15 offset:25600\n\t"
        "s_waitcnt vmcnt(6) lgkmcnt(14)\n\t"
        "v_mfma_f32_16x16x32_bf16 v[192:195], v[160:163], v[56:59], v[192:195]\n\t"
        "ds_read_b64_tr_b16 v[128:129], v8 offset:32768\n\t"
        "v_mfma_f32_16x16x32_bf16 v[208:211], v[160:163], v[60:63], v[208:211]\n\t"
        "ds_read_b64_tr_b16 v[130:131], v8 offset:33792\n\t"
        "v_mfma_f32_16x16x32_bf16 v[196:199], v[164:167], v[56:59], v[196:199]\n\t"
        "ds_read_b64_tr_b16 v[132:133], v9 offset:32768\n\t"
        "v_mfma_f32_16x16x32_bf16 v[212:215], v[164:167], v[60:63], v[212:215]\n\t"
        "ds_read_b64_tr_b16 v[134:135], v9 offset:33792\n\t"
        "s_waitcnt lgkmcnt(13)\n\t"
        "v_mfma_f32_16x16x32_bf16 v[200:203], v[168:171], v[56:59], v[200:203]\n\t"
        "ds_read_b64_tr_b16 v[136:137], v10 offset:32768\n\t"
        "s_movk_i32 s61, 5\n\t"
        "v_mfma_f32_16x16x32_bf16 v[216:219], v[168:171], v[60:63], v[216:219]\n\t"
        "ds_read_b64_tr_b16 v[138:139], v10 offset:33792\n\t"
        "v_cmp_ge_u32 vcc, s61, v121\n\t"
        "v_mfma_f32_16x16x32_bf16 v[204:207], v[172:175], v[56:59], v[204:207]\n\t"
        "ds_read_b64_tr_b16 v[140:141], v11 offset:32768\n\t"
        "s_getpc_b64 s[70:71]\n\t"
        "s_cbranch_vccnz .Lspin1_%=\n\t"
        "v_mfma_f32_16x16x32_bf16 v[220:223], v[172:175], v[60:63], v[220:223]\n\t"
        "ds_read_b64_tr_b16 v[142:143], v11 offset:33792\n\t"
        "ds_read_b32 v121, v127 offset:4\n\t"
        "s_waitcnt lgkmcnt(14)\n\t"
        "v_mfma_f32_16x16x32_bf16 v[224:227], v[176:179], v[56:59], v[224:227]\n\t"
        "ds_read_b64_tr_b16 v[144:145], v12 offset:32768\n\t"
        "global_load_dwordx4 v[88:91], v119, s[62:63] offset:128\n\t"
        "v_mfma_f32_16x16x32_bf16 v[240:243], v[176:179], v[60:63], v[240:243]\n\t"
        "ds_read_b64_tr_b16 v[146:147], v12 offset:33792\n\t"
        "global_load_dwordx4 v[92:95], v119, s[72:73] offset:128\n\t"
        "v_mfma_f32_16x16x32_bf16 v[228:231], v[180:183], v[56:59], v[228:231]\n\t"
        "ds_read_b64_tr_b16 v[148:149], v13 offset:32768\n\t"
        "v_mfma_f32_16x16x32_bf16 v[244:247], v[180:183], v[60:63], v[244:247]\n\t"
        "ds_read_b64_tr_b16 v[150:151], v13 offset:33792\n\t"
        "s_waitcnt lgkmcnt(13)\n\t"
        "v_mfma_f32_16x16x32_bf16 v[232:235], v[184:187], v[56:59], v[232:235]\n\t"
        "ds_read_b64_tr_b16 v[152:153], v14 offset:32768\n\t"
        "v_mfma_f32_16x16x32_bf16 v[248:251], v[184:187], v[60:63], v[248:251]\n\t"
        "ds_read_b64_tr_b16 v[154:155], v14 offset:33792\n\t"
        "ds_add_u32 %4, v31\n\t"
        "v_mfma_f32_16x16x32_bf16 v[236:239], v[188:191], v[56:59], v[236:239]\n\t"
        "ds_read_b64_tr_b16 v[156:157], v15 offset:32768\n\t"
        "v_mfma_f32_16x16x32_bf16 v[252:255], v[188:191], v[60:63], v[252:255]\n\t"
        "ds_read_b64_tr_b16 v[158:159], v15 offset:33792\n\t"
        "s_waitcnt vmcnt(6) lgkmcnt(14)\n\t"
        "v_mfma_f32_16x16x32_bf16 v[192:195], v[128:131], v[64:67], v[192:195]\n\t"
        "ds_read_b64_tr_b16 v[160:161], v8 offset:40960\n\t"
        "v_mfma_f32_16x16x32_bf16 v[208:211], v[128:131], v[68:71], v[208:211]\n\t"
        "ds_read_b64_tr_b16 v[162:163], v8 offset:41984\n\t"
        "v_mfma_f32_16x16x32_bf16 v[196:199], v[132:135], v[64:67], v[196:199]\n\t"
        "ds_read_b64_tr_b16 v[164:165], v9 offset:40960\n\t"
        "v_mfma_f32_16x16x32_bf16 v[212:215], v[132:135], v[68:71], v[212:215]\n\t"
        "ds_read_b64_tr_b16 v[166:167], v9 offset:41984\n\t"
        "s_waitcnt lgkmcnt(13)\n\t"
        "v_mfma_f32_16x16x32_bf16 v[200:203], v[136:139], v[64:67], v[200:203]\n\t"
        "ds_read_b64_tr_b16 v[168:169], v10 offset:40960\n\t"
        "s_movk_i32 s61, 6\n\t"
        "v_mfma_f32_16x16x32_bf16 v[216:219], v[136:139], v[68:71], v[216:219]\n\t"
        "ds_read_b64_tr_b16 v[170:171], v10 offset:41984\n\t"
        "v_cmp_ge_u32 vcc, s61, v121\n\t"
        "v_mfma_f32_16x16x32_bf16 v[204:207], v[140:143], v[64:67], v[204:207]\n\t"
        "ds_read_b64_tr_b16 v[172:173], v11 offset:40960\n\t"
        "s_getpc_b64 s[70:71]\n\t"
        "s_cbranch_vccnz .Lspin1_%=\n\t"
        "v_mfma_f32_16x16x32_bf16 v[220:223], v[140:143], v[68:71], v[220:223]\n\t"
        "ds_read_b64_tr_b16 v[174:175], v11 offset:41984\n\t"
        "ds_read_b32 v121, v127 offset:4\n\t"
        "ds_read_b32 v24, v127 offset:288\n\t"
        "ds_read_b32 v25, v127 offset:292\n\t"
        "s_waitcnt lgkmcnt(15)\n\t"
        "v_mfma_f32_16x16x32_bf16 v[224:227], v[144:147], v[64:67], v[224:227]\n\t"
        "ds_read_b64_tr_b16 v[176:177], v12 offset:40960\n\t"
        "v_mfma_f32_16x16x32_bf16 v[240:243], v[144:147], v[68:71], v[240:243]\n\t"
        "ds_read_b64_tr_b16 v[178:179], v12 offset:41984\n\t"
        "v_mfma_f32_16x16x32_bf16 v[228:231], v[148:151], v[64:67], v[228:231]\n\t"
        "ds_read_b64_tr_b16 v[180:181], v13 offset:40960\n\t"
        "v_mfma_f32_16x16x32_bf16 v[244:247], v[148:151], v[68:71], v[244:247]\n\t"
        "ds_read_b64_tr_b16 v[182:183], v13 offset:41984\n\t"
        "s_waitcnt lgkmcnt(15)\n\t"
        "v_mfma_f32_16x16x32_bf16 v[232:235], v[152:155], v[64:67], v[232:235]\n\t"
        "ds_read_b64_tr_b16 v[184:185], v14 offset:40960\n\t"
        "v_mfma_f32_16x16x32_bf16 v[248:251], v[152:155], v[68:71], v[248:251]\n\t"
        "ds_read_b64_tr_b16 v[186:187], v14 offset:41984\n\t"
        "ds_add_u32 %4, v31\n\t"
        "v_mfma_f32_16x16x32_bf16 v[236:239], v[156:159], v[64:67], v[236:239]\n\t"
        "ds_read_b64_tr_b16 v[188:189], v15 offset:40960\n\t"
        "v_mfma_f32_16x16x32_bf16 v[252:255], v[156:159], v[68:71], v[252:255]\n\t"
        "ds_read_b64_tr_b16 v[190:191], v15 offset:41984\n\t"
        "s_waitcnt vmcnt(4) lgkmcnt(15)\n\t"
        "v_mfma_f32_16x16x32_bf16 v[192:195], v[160:163], v[72:75], v[192:195]\n\t"
        "ds_read_b64_tr_b16 v[128:129], v8 offset:49152\n\t"
        "v_mfma_f32_16x16x32_bf16 v[208:211], v[160:163], v[76:79], v[208:211]\n\t"
        "ds_read_b64_tr_b16 v[130:131], v8 offset:50176\n\t"
        "v_mfma_f32_16x16x32_bf16 v[196:199], v[164:167], v[72:75], v[196:199]\n\t"
        "ds_read_b64_tr_b16 v[132:133], v9 offset:49152\n\t"
        "v_mfma_f32_16x16x32_bf16 v[212:215], v[164:167], v[76:79], v[212:215]\n\t"
        "ds_read_b64_tr_b16 v[134:135], v9 offset:50176\n\t"
        "s_waitcnt lgkmcnt(13)\n\t"
        "v_mfma_f32_16x16x32_bf16 v[200:203], v[168:171], v[72:75], v[200:203]\n\t"
        "ds_read_b64_tr_b16 v[136:137], v10 offset:49152\n\t"
        "s_movk_i32 s61, 7\n\t"
        "v_cmp_ge_u32 vcc, s61, v121\n\t"
        "v_mfma_f32_16x16x32_bf16 v[216:219], v[168:171], v[76:79], v[216:219]\n\t"
        "ds_read_b64_tr_b16 v[138:139], v10 offset:50176\n\t"
        "s_getpc_b64 s[70:71]\n\t"
        "s_cbranch_vccnz .Lspin1_%=\n\t"
        "s_movk_i32 s61, 8\n\t"
        "v_mfma_f32_16x16x32_bf16 v[204:207], v[172:175], v[72:75], v[204:207]\n\t"
        "ds_read_b64_tr_b16 v[140:141], v11 offset:49152\n\t"
        "v_cmp_eq_u32 vcc, 0, v24\n\t"
        "s_getpc_b64 s[70:71]\n\t"
        "s_cbranch_vccnz .Lspin4_%=\n\t"
        "v_mfma_f32_16x16x32_bf16 v[220:223], v[172:175], v[76:79], v[220:223]\n\t"
        "ds_read_b64_tr_b16 v[142:143], v11 offset:50176\n\t"
        "v_add_u32 v24, v24, %2\n\t"
        "global_load_dwordx4 v[96:99], v24, s[68:69] offset:0\n\t"
        "s_waitcnt lgkmcnt(13)\n\t"
        "v_mfma_f32_16x16x32_bf16 v[224:227], v[176:179], v[72:75], v[224:227]\n\t"
        "ds_read_b64_tr_b16 v[144:145], v12 offset:49152\n\t"
        "global_load_dwordx4 v[100:103], v24, s[68:69] offset:1024\n\t"
        "v_mfma_f32_16x16x32_bf16 v[240:243], v[176:179], v[76:79], v[240:243]\n\t"
        "ds_read_b64_tr_b16 v[146:147], v12 offset:50176\n\t"
        "s_movk_i32 s61, 9\n\t"
        "v_mfma_f32_16x16x32_bf16 v[228:231], v[180:183], v[72:75], v[228:231]\n\t"
        "ds_read_b64_tr_b16 v[148:149], v13 offset:49152\n\t"
        "v_cmp_eq_u32 vcc, 0, v25\n\t"
        "v_mfma_f32_16x16x32_bf16 v[244:247], v[180:183], v[76:79], v[244:247]\n\t"
        "ds_read_b64_tr_b16 v[150:151], v13 offset:50176\n\t"
        "s_getpc_b64 s[70:71]\n\t"
        "s_cbranch_vccnz .Lspin5_%=\n\t"
        "s_waitcnt lgkmcnt(12)\n\t"
        "v_mfma_f32_16x16x32_bf16 v[232:235], v[184:187], v[72:75], v[232:235]\n\t"
        "ds_read_b64_tr_b16 v[152:153], v14 offset:49152\n\t"
        "ds_read_b32 v121, v127 offset:4\n\t"
        "ds_read_b32 v26, v127 offset:296\n\t"
        "v_mfma_f32_16x16x32_bf16 v[248:251], v[184:187], v[76:79], v[248:251]\n\t"
        "ds_read_b64_tr_b16 v[154:155], v14 offset:50176\n\t"
        "ds_add_u32 %4, v31\n\t"
        "v_add_u32 v25, v25, %2\n\t"
        "v_mfma_f32_16x16x32_bf16 v[236:239], v[188:191], v[72:75], v[236:239]\n\t"
        "ds_read_b64_tr_b16 v[156:157], v15 offset:49152\n\t"
        "global_load_dwordx4 v[104:107], v25, s[68:69] offset:0\n\t"
        "global_load_dwordx4 v[108:111], v25, s[68:69] offset:1024\n\t"
        "v_mfma_f32_16x16x32_bf16 v[252:255], v[188:191], v[76:79], v[252:255]\n\t"
        "ds_read_b64_tr_b16 v[158:159], v15 offset:50176\n\t"
        "s_waitcnt vmcnt(6) lgkmcnt(15)\n\t"
        "v_mfma_f32_16x16x32_bf16 v[192:195], v[128:131], v[80:83], v[192:195]\n\t"
        "ds_read_b64_tr_b16 v[160:161], v8 offset:57344\n\t"
        "v_mfma_f32_16x16x32_bf16 v[208:211], v[128:131], v[84:87], v[208:211]\n\t"
        "ds_read_b64_tr_b16 v[162:163], v8 offset:58368\n\t"
        "v_mfma_f32_16x16x32_bf16 v[196:199], v[132:135], v[80:83], v[196:199]\n\t"
        "ds_read_b64_tr_b16 v[164:165], v9 offset:57344\n\t"
        "v_mfma_f32_16x16x32_bf16 v[212:215], v[132:135], v[84:87], v[212:215]\n\t"
        "ds_read_b64_tr_b16 v[166:167], v9 offset:58368\n\t"
        "s_waitcnt lgkmcnt(8)\n\t"
        "v_mfma_f32_16x16x32_bf16 v[200:203], v[136:139], v[80:83], v[200:203]\n\t"
        "ds_read_b64_tr_b16 v[168:169], v10 offset:57344\n\t"
        "s_movk_i32 s61, 8\n\t"
        "v_mfma_f32_16x16x32_bf16 v[216:219], v[136:139], v[84:87], v[216:219]\n\t"
        "ds_read_b64_tr_b16 v[170:171], v10 offset:58368\n\t"
        "v_cmp_ge_u32 vcc, s61, v121\n\t"
        "v_mfma_f32_16x16x32_bf16 v[204:207], v[140:143], v[80:83], v[204:207]\n\t"
        "ds_read_b64_tr_b16 v[172:173], v11 offset:57344\n\t"
        "s_getpc_b64 s[70:71]\n\t"
        "s_cbranch_vccnz .Lspin1_%=\n\t"
        "v_mfma_f32_16x16x32_bf16 v[220:223], v[140:143], v[84:87], v[220:223]\n\t"
        "ds_read_b64_tr_b16 v[174:175], v11 offset:58368\n\t"
        "s_movk_i32 s61, 10\n\t"
        "s_waitcnt lgkmcnt(15)\n\t"
        "v_mfma_f32_16x16x32_bf16 v[224:227], v[144:147], v[80:83], v[224:227]\n\t"
        "ds_read_b64_tr_b16 v[176:177], v12 offset:57344\n\t"
        "v_cmp_eq_u32 vcc, 0, v26\n\t"
        "v_mfma_f32_16x16x32_bf16 v[240:243], v[144:147], v[84:87], v[240:243]\n\t"
        "ds_read_b64_tr_b16 v[178:179], v12 offset:58368\n\t"
        "s_getpc_b64 s[70:71]\n\t"
        "s_cbranch_vccnz .Lspin6_%=\n\t"
        "v_mfma_f32_16x16x32_bf16 v[228:231], v[148:151], v[80:83], v[228:231]\n\t"
        "ds_read_b64_tr_b16 v[180:181], v13 offset:57344\n\t"
        "ds_read_b32 v121, v127 offset:4\n\t"
        "ds_read_b32 v27, v127 offset:300\n\t"
        "v_mfma_f32_16x16x32_bf16 v[244:247], v[148:151], v[84:87], v[244:247]\n\t"
        "ds_read_b64_tr_b16 v[182:183], v13 offset:58368\n\t"
        "v_add_u32 v26, v26, %2\n\t"
        "s_waitcnt lgkmcnt(14)\n\t"
        "v_mfma_f32_16x16x32_bf16 v[232:235], v[152:155], v[80:83], v[232:235]\n\t"
        "ds_read_b64_tr_b16 v[184:185], v14 offset:57344\n\t"
        "global_load_dwordx4 v[112:115], v26, s[68:69] offset:0\n\t"
        "v_mfma_f32_16x16x32_bf16 v[248:251], v[152:155], v[84:87], v[248:251]\n\t"
        "ds_read_b64_tr_b16 v[186:187], v14 offset:58368\n\t"
        "ds_add_u32 %4, v31\n\t"
        "v_mfma_f32_16x16x32_bf16 v[236:239], v[156:159], v[80:83], v[236:239]\n\t"
        "ds_read_b64_tr_b16 v[188:189], v15 offset:57344\n\t"
        "global_load_dwordx4 v[116:119], v26, s[68:69] offset:1024\n\t"
        "v_mfma_f32_16x16x32_bf16 v[252:255], v[156:159], v[84:87], v[252:255]\n\t"
        "ds_read_b64_tr_b16 v[190:191], v15 offset:58368\n\t"
        "s_waitcnt vmcnt(6) lgkmcnt(15)\n\t"
        "v_mfma_f32_16x16x32_bf16 v[192:195], v[160:163], v[88:91], v[192:195]\n\t"
        "ds_read_b64_tr_b16 v[128:129], v16 offset:0\n\t"
        "v_mfma_f32_16x16x32_bf16 v[208:211], v[160:163], v[92:95], v[208:211]\n\t"
        "ds_read_b64_tr_b16 v[130:131], v16 offset:1024\n\t"
        "v_mfma_f32_16x16x32_bf16 v[196:199], v[164:167], v[88:91], v[196:199]\n\t"
        "ds_read_b64_tr_b16 v[132:133], v17 offset:0\n\t"
        "v_mfma_f32_16x16x32_bf16 v[212:215], v[164:167], v[92:95], v[212:215]\n\t"
        "ds_read_b64_tr_b16 v[134:135], v17 offset:1024\n\t"
        "s_waitcnt lgkmcnt(10)\n\t"
        "v_mfma_f32_16x16x32_bf16 v[200:203], v[168:171], v[88:91], v[200:203]\n\t"
        "ds_read_b64_tr_b16 v[136:137], v18 offset:0\n\t"
        "s_movk_i32 s61, 9\n\t"
        "v_mfma_f32_16x16x32_bf16 v[216:219], v[168:171], v[92:95], v[216:219]\n\t"
        "ds_read_b64_tr_b16 v[138:139], v18 offset:1024\n\t"
        "v_cmp_ge_u32 vcc, s61, v121\n\t"
        "v_mfma_f32_16x16x32_bf16 v[204:207], v[172:175], v[88:91], v[204:207]\n\t"
        "ds_read_b64_tr_b16 v[140:141], v19 offset:0\n\t"
        "s_getpc_b64 s[70:71]\n\t"
        "s_cbranch_vccnz .Lspin1_%=\n\t"
        "v_mfma_f32_16x16x32_bf16 v[220:223], v[172:175], v[92:95], v[220:223]\n\t"
        "ds_read_b64_tr_b16 v[142:143], v19 offset:1024\n\t"
        "s_movk_i32 s61, 11\n\t"
        "s_waitcnt lgkmcnt(13)\n\t"
        "v_mfma_f32_16x16x32_bf16 v[224:227], v[176:179], v[88:91], v[224:227]\n\t"
        "ds_read_b64_tr_b16 v[144:145], v20 offset:0\n\t"
        "v_cmp_eq_u32 vcc, 0, v27\n\t"
        "v_mfma_f32_16x16x32_bf16 v[240:243], v[176:179], v[92:95], v[240:243]\n\t"
        "ds_read_b64_tr_b16 v[146:147], v20 offset:1024\n\t"
        "s_getpc_b64 s[70:71]\n\t"
        "s_cbranch_vccnz .Lspin7_%=\n\t"
        "v_mfma_f32_16x16x32_bf16 v[228:231], v[180:183], v[88:91], v[228:231]\n\t"
        "ds_read_b64_tr_b16 v[148:149], v21 offset:0\n\t"
        "ds_read_b32 v121, v127 offset:4\n\t"
        "ds_read_b32 v24, v127 offset:304\n\t"
        "v_mfma_f32_16x16x32_bf16 v[244:247], v[180:183], v[92:95], v[244:247]\n\t"
        "ds_read_b64_tr_b16 v[150:151], v21 offset:1024\n\t"
        "v_add_u32 v27, v27, %2\n\t"
        "s_waitcnt lgkmcnt(14)\n\t"
        "v_mfma_f32_16x16x32_bf16 v[232:235], v[184:187], v[88:91], v[232:235]\n\t"
        "ds_read_b64_tr_b16 v[152:153], v22 offset:0\n\t"
        "global_load_dwordx4 v[32:35], v27, s[68:69] offset:0\n\t"
        "v_mfma_f32_16x16x32_bf16 v[248:251], v[184:187], v[92:95], v[248:251]\n\t"
        "ds_read_b64_tr_b16 v[154:155], v22 offset:1024\n\t"
        "ds_add_u32 %4, v31\n\t"
        "v_mfma_f32_16x16x32_bf16 v[236:239], v[188:191], v[88:91], v[236:239]\n\t"
        "ds_read_b64_tr_b16 v[156:157], v23 offset:0\n\t"
        "global_load_dwordx4 v[36:39], v27, s[68:69] offset:1024\n\t"
        "v_mfma_f32_16x16x32_bf16 v[252:255], v[188:191], v[92:95], v[252:255]\n\t"
        "ds_read_b64_tr_b16 v[158:159], v23 offset:1024\n\t"
        "s_waitcnt vmcnt(6) lgkmcnt(15)\n\t"
        "v_mfma_f32_16x16x32_bf16 v[192:195], v[128:131], v[96:99], v[192:195]\n\t"
        "ds_read_b64_tr_b16 v[160:161], v16 offset:8192\n\t"
        "v_mfma_f32_16x16x32_bf16 v[208:211], v[128:131], v[100:103], v[208:211]\n\t"
        "ds_read_b64_tr_b16 v[162:163], v16 offset:9216\n\t"
        "v_mfma_f32_16x16x32_bf16 v[196:199], v[132:135], v[96:99], v[196:199]\n\t"
        "ds_read_b64_tr_b16 v[164:165], v17 offset:8192\n\t"
        "v_mfma_f32_16x16x32_bf16 v[212:215], v[132:135], v[100:103], v[212:215]\n\t"
        "ds_read_b64_tr_b16 v[166:167], v17 offset:9216\n\t"
        "s_waitcnt lgkmcnt(10)\n\t"
        "v_mfma_f32_16x16x32_bf16 v[200:203], v[136:139], v[96:99], v[200:203]\n\t"
        "ds_read_b64_tr_b16 v[168:169], v18 offset:8192\n\t"
        "s_movk_i32 s61, 10\n\t"
        "v_mfma_f32_16x16x32_bf16 v[216:219], v[136:139], v[100:103], v[216:219]\n\t"
        "ds_read_b64_tr_b16 v[170:171], v18 offset:9216\n\t"
        "v_cmp_ge_u32 vcc, s61, v121\n\t"
        "v_mfma_f32_16x16x32_bf16 v[204:207], v[140:143], v[96:99], v[204:207]\n\t"
        "ds_read_b64_tr_b16 v[172:173], v19 offset:8192\n\t"
        "s_getpc_b64 s[70:71]\n\t"
        "s_cbranch_vccnz .Lspin1_%=\n\t"
        "v_mfma_f32_16x16x32_bf16 v[220:223], v[140:143], v[100:103], v[220:223]\n\t"
        "ds_read_b64_tr_b16 v[174:175], v19 offset:9216\n\t"
        "s_movk_i32 s61, 12\n\t"
        "s_waitcnt lgkmcnt(13)\n\t"
        "v_mfma_f32_16x16x32_bf16 v[224:227], v[144:147], v[96:99], v[224:227]\n\t"
        "ds_read_b64_tr_b16 v[176:177], v20 offset:8192\n\t"
        "v_cmp_eq_u32 vcc, 0, v24\n\t"
        "v_mfma_f32_16x16x32_bf16 v[240:243], v[144:147], v[100:103], v[240:243]\n\t"
        "ds_read_b64_tr_b16 v[178:179], v20 offset:9216\n\t"
        "s_getpc_b64 s[70:71]\n\t"
        "s_cbranch_vccnz .Lspin4_%=\n\t"
        "v_mfma_f32_16x16x32_bf16 v[228:231], v[148:151], v[96:99], v[228:231]\n\t"
        "ds_read_b64_tr_b16 v[180:181], v21 offset:8192\n\t"
        "ds_read_b32 v121, v127 offset:4\n\t"
        "ds_read_b32 v25, v127 offset:308\n\t"
        "v_mfma_f32_16x16x32_bf16 v[244:247], v[148:151], v[100:103], v[244:247]\n\t"
        "ds_read_b64_tr_b16 v[182:183], v21 offset:9216\n\t"
        "v_add_u32 v24, v24, %2\n\t"
        "s_waitcnt lgkmcnt(14)\n\t"
        "v_mfma_f32_16x16x32_bf16 v[232:235], v[152:155], v[96:99], v[232:235]\n\t"
        "ds_read_b64_tr_b16 v[184:185], v22 offset:8192\n\t"
        "global_load_dwordx4 v[40:43], v24, s[68:69] offset:0\n\t"
        "v_mfma_f32_16x16x32_bf16 v[248:251], v[152:155], v[100:103], v[248:251]\n\t"
        "ds_read_b64_tr_b16 v[186:187], v22 offset:9216\n\t"
        "ds_add_u32 %4, v31\n\t"
        "v_mfma_f32_16x16x32_bf16 v[236:239], v[156:159], v[96:99], v[236:239]\n\t"
        "ds_read_b64_tr_b16 v[188:189], v23 offset:8192\n\t"
        "global_load_dwordx4 v[44:47], v24, s[68:69] offset:1024\n\t"
        "v_mfma_f32_16x16x32_bf16 v[252:255], v[156:159], v[100:103], v[252:255]\n\t"
        "ds_read_b64_tr_b16 v[190:191], v23 offset:9216\n\t"
        "s_waitcnt vmcnt(6) lgkmcnt(15)\n\t"
        "v_mfma_f32_16x16x32_bf16 v[192:195], v[160:163], v[104:107], v[192:195]\n\t"
        "ds_read_b64_tr_b16 v[128:129], v16 offset:16384\n\t"
        "v_mfma_f32_16x16x32_bf16 v[208:211], v[160:163], v[108:111], v[208:211]\n\t"
        "ds_read_b64_tr_b16 v[130:131], v16 offset:17408\n\t"
        "v_mfma_f32_16x16x32_bf16 v[196:199], v[164:167], v[104:107], v[196:199]\n\t"
        "ds_read_b64_tr_b16 v[132:133], v17 offset:16384\n\t"
        "v_mfma_f32_16x16x32_bf16 v[212:215], v[164:167], v[108:111], v[212:215]\n\t"
        "ds_read_b64_tr_b16 v[134:135], v17 offset:17408\n\t"
        "s_waitcnt lgkmcnt(10)\n\t"
        "v_mfma_f32_16x16x32_bf16 v[200:203], v[168:171], v[104:107], v[200:203]\n\t"
        "ds_read_b64_tr_b16 v[136:137], v18 offset:16384\n\t"
        "s_movk_i32 s61, 11\n\t"
        "v_mfma_f32_16x16x32_bf16 v[216:219], v[168:171], v[108:111], v[216:219]\n\t"
        "ds_read_b64_tr_b16 v[138:139], v18 offset:17408\n\t"
        "v_cmp_ge_u32 vcc, s61, v121\n\t"
        "v_mfma_f32_16x16x32_bf16 v[204:207], v[172:175], v[104:107], v[204:207]\n\t"
        "ds_read_b64_tr_b16 v[140:141], v19 offset:16384\n\t"
        "s_getpc_b64 s[70:71]\n\t"
        "s_cbranch_vccnz .Lspin1_%=\n\t"
        "v_mfma_f32_16x16x32_bf16 v[220:223], v[172:175], v[108:111], v[220:223]\n\t"
        "ds_read_b64_tr_b16 v[142:143], v19 offset:17408\n\t"
        "s_movk_i32 s61, 13\n\t"
        "s_waitcnt lgkmcnt(13)\n\t"
        "v_mfma_f32_16x16x32_bf16 v[224:227], v[176:179], v[104:107], v[224:227]\n\t"
        "ds_read_b64_tr_b16 v[144:145], v20 offset:16384\n\t"
        "v_cmp_eq_u32 vcc, 0, v25\n\t"
        "v_mfma_f32_16x16x32_bf16 v[240:243], v[176:179], v[108:111], v[240:243]\n\t"
        "ds_read_b64_tr_b16 v[146:147], v20 offset:17408\n\t"
        "s_getpc_b64 s[70:71]\n\t"
        "s_cbranch_vccnz .Lspin5_%=\n\t"
        "v_mfma_f32_16x16x32_bf16 v[228:231], v[180:183], v[104:107], v[228:231]\n\t"
        "ds_read_b64_tr_b16 v[148:149], v21 offset:16384\n\t"
        "ds_read_b32 v121, v127 offset:4\n\t"
        "ds_read_b32 v26, v127 offset:312\n\t"
        "v_mfma_f32_16x16x32_bf16 v[244:247], v[180:183], v[108:111], v[244:247]\n\t"
        "ds_read_b64_tr_b16 v[150:151], v21 offset:17408\n\t"
        "v_add_u32 v25, v25, %2\n\t"
        "s_waitcnt lgkmcnt(14)\n\t"
        "v_mfma_f32_16x16x32_bf16 v[232:235], v[184:187], v[104:107], v[232:235]\n\t"
        "ds_read_b64_tr_b16 v[152:153], v22 offset:16384\n\t"
        "global_load_dwordx4 v[48:51], v25, s[68:69] offset:0\n\t"
        "v_mfma_f32_16x16x32_bf16 v[248:251], v[184:187], v[108:111], v[248:251]\n\t"
        "ds_read_b64_tr_b16 v[154:155], v22 offset:17408\n\t"
        "ds_add_u32 %4, v31\n\t"
        "v_mfma_f32_16x16x32_bf16 v[236:239], v[188:191], v[104:107], v[236:239]\n\t"
        "ds_read_b64_tr_b16 v[156:157], v23 offset:16384\n\t"
        "global_load_dwordx4 v[52:55], v25, s[68:69] offset:1024\n\t"
        "v_mfma_f32_16x16x32_bf16 v[252:255], v[188:191], v[108:111], v[252:255]\n\t"
        "ds_read_b64_tr_b16 v[158:159], v23 offset:17408\n\t"
        "s_waitcnt vmcnt(6) lgkmcnt(15)\n\t"
        "v_mfma_f32_16x16x32_bf16 v[192:195], v[128:131], v[112:115], v[192:195]\n\t"
        "ds_read_b64_tr_b16 v[160:161], v16 offset:24576\n\t"
        "v_mfma_f32_16x16x32_bf16 v[208:211], v[128:131], v[116:119], v[208:211]\n\t"
        "ds_read_b64_tr_b16 v[162:163], v16 offset:25600\n\t"
        "v_mfma_f32_16x16x32_bf16 v[196:199], v[132:135], v[112:115], v[196:199]\n\t"
        "ds_read_b64_tr_b16 v[164:165], v17 offset:24576\n\t"
        "v_mfma_f32_16x16x32_bf16 v[212:215], v[132:135], v[116:119], v[212:215]\n\t"
        "ds_read_b64_tr_b16 v[166:167], v17 offset:25600\n\t"
        "s_waitcnt lgkmcnt(10)\n\t"
        "v_mfma_f32_16x16x32_bf16 v[200:203], v[136:139], v[112:115], v[200:203]\n\t"
        "ds_read_b64_tr_b16 v[168:169], v18 offset:24576\n\t"
        "s_movk_i32 s61, 12\n\t"
        "v_mfma_f32_16x16x32_bf16 v[216:219], v[136:139], v[116:119], v[216:219]\n\t"
        "ds_read_b64_tr_b16 v[170:171], v18 offset:25600\n\t"
        "v_cmp_ge_u32 vcc, s61, v121\n\t"
        "v_mfma_f32_16x16x32_bf16 v[204:207], v[140:143], v[112:115], v[204:207]\n\t"
        "ds_read_b64_tr_b16 v[172:173], v19 offset:24576\n\t"
        "s_getpc_b64 s[70:71]\n\t"
        "s_cbranch_vccnz .Lspin1_%=\n\t"
        "v_mfma_f32_16x16x32_bf16 v[220:223], v[140:143], v[116:119], v[220:223]\n\t"
        "ds_read_b64_tr_b16 v[174:175], v19 offset:25600\n\t"
        "s_movk_i32 s61, 14\n\t"
        "s_waitcnt lgkmcnt(13)\n\t"
        "v_mfma_f32_16x16x32_bf16 v[224:227], v[144:147], v[112:115], v[224:227]\n\t"
        "ds_read_b64_tr_b16 v[176:177], v20 offset:24576\n\t"
        "v_cmp_eq_u32 vcc, 0, v26\n\t"
        "v_mfma_f32_16x16x32_bf16 v[240:243], v[144:147], v[116:119], v[240:243]\n\t"
        "ds_read_b64_tr_b16 v[178:179], v20 offset:25600\n\t"
        "s_getpc_b64 s[70:71]\n\t"
        "s_cbranch_vccnz .Lspin6_%=\n\t"
        "v_mfma_f32_16x16x32_bf16 v[228:231], v[148:151], v[112:115], v[228:231]\n\t"
        "ds_read_b64_tr_b16 v[180:181], v21 offset:24576\n\t"
        "ds_read_b32 v121, v127 offset:4\n\t"
        "ds_read_b32 v27, v127 offset:316\n\t"
        "v_mfma_f32_16x16x32_bf16 v[244:247], v[148:151], v[116:119], v[244:247]\n\t"
        "ds_read_b64_tr_b16 v[182:183], v21 offset:25600\n\t"
        "v_add_u32 v26, v26, %2\n\t"
        "s_waitcnt lgkmcnt(14)\n\t"
        "v_mfma_f32_16x16x32_bf16 v[232:235], v[152:155], v[112:115], v[232:235]\n\t"
        "ds_read_b64_tr_b16 v[184:185], v22 offset:24576\n\t"
        "global_load_dwordx4 v[56:59], v26, s[68:69] offset:0\n\t"
        "v_mfma_f32_16x16x32_bf16 v[248:251], v[152:155], v[116:119], v[248:251]\n\t"
        "ds_read_b64_tr_b16 v[186:187], v22 offset:25600\n\t"
        "ds_add_u32 %4, v31\n\t"
        "v_mfma_f32_16x16x32_bf16 v[236:239], v[156:159], v[112:115], v[236:239]\n\t"
        "ds_read_b64_tr_b16 v[188:189], v23 offset:24576\n\t"
        "global_load_dwordx4 v[60:63], v26, s[68:69] offset:1024\n\t"
        "v_mfma_f32_16x16x32_bf16 v[252:255], v[156:159], v[116:119], v[252:255]\n\t"
        "ds_read_b64_tr_b16 v[190:191], v23 offset:25600\n\t"
        "s_waitcnt vmcnt(6) lgkmcnt(15)\n\t"
        "v_mfma_f32_16x16x32_bf16 v[192:195], v[160:163], v[32:35], v[192:195]\n\t"
        "ds_read_b64_tr_b16 v[128:129], v16 offset:32768\n\t"
        "v_mfma_f32_16x16x32_bf16 v[208:211], v[160:163], v[36:39], v[208:211]\n\t"
        "ds_read_b64_tr_b16 v[130:131], v16 offset:33792\n\t"
        "v_mfma_f32_16x16x32_bf16 v[196:199], v[164:167], v[32:35], v[196:199]\n\t"
        "ds_read_b64_tr_b16 v[132:133], v17 offset:32768\n\t"
        "v_mfma_f32_16x16x32_bf16 v[212:215], v[164:167], v[36:39], v[212:215]\n\t"
        "ds_read_b64_tr_b16 v[134:135], v17 offset:33792\n\t"
        "s_waitcnt lgkmcnt(10)\n\t"
        "v_mfma_f32_16x16x32_bf16 v[200:203], v[168:171], v[32:35], v[200:203]\n\t"
        "ds_read_b64_tr_b16 v[136:137], v18 offset:32768\n\t"
        "s_movk_i32 s61, 13\n\t"
        "v_mfma_f32_16x16x32_bf16 v[216:219], v[168:171], v[36:39], v[216:219]\n\t"
        "ds_read_b64_tr_b16 v[138:139], v18 offset:33792\n\t"
        "v_cmp_ge_u32 vcc, s61, v121\n\t"
        "v_mfma_f32_16x16x32_bf16 v[204:207], v[172:175], v[32:35], v[204:207]\n\t"
        "ds_read_b64_tr_b16 v[140:141], v19 offset:32768\n\t"
        "s_getpc_b64 s[70:71]\n\t"
        "s_cbranch_vccnz .Lspin1_%=\n\t"
        "v_mfma_f32_16x16x32_bf16 v[220:223], v[172:175], v[36:39], v[220:223]\n\t"
        "ds_read_b64_tr_b16 v[142:143], v19 offset:33792\n\t"
        "s_movk_i32 s61, 15\n\t"
        "s_waitcnt lgkmcnt(13)\n\t"
        "v_mfma_f32_16x16x32_bf16 v[224:227], v[176:179], v[32:35], v[224:227]\n\t"
        "ds_read_b64_tr_b16 v[144:145], v20 offset:32768\n\t"
        "v_cmp_eq_u32 vcc, 0, v27\n\t"
        "v_mfma_f32_16x16x32_bf16 v[240:243], v[176:179], v[36:39], v[240:243]\n\t"
        "ds_read_b64_tr_b16 v[146:147], v20 offset:33792\n\t"
        "s_getpc_b64 s[70:71]\n\t"
        "s_cbranch_vccnz .Lspin7_%=\n\t"
        "v_mfma_f32_16x16x32_bf16 v[228:231], v[180:183], v[32:35], v[228:231]\n\t"
        "ds_read_b64_tr_b16 v[148:149], v21 offset:32768\n\t"
        "ds_read_b32 v121, v127 offset:4\n\t"
        "ds_read_b32 v24, v127 offset:320\n\t"
        "v_mfma_f32_16x16x32_bf16 v[244:247], v[180:183], v[36:39], v[244:247]\n\t"
        "ds_read_b64_tr_b16 v[150:151], v21 offset:33792\n\t"
        "v_add_u32 v27, v27, %2\n\t"
        "s_waitcnt lgkmcnt(14)\n\t"
        "v_mfma_f32_16x16x32_bf16 v[232:235], v[184:187], v[32:35], v[232:235]\n\t"
        "ds_read_b64_tr_b16 v[152:153], v22 offset:32768\n\t"
        "global_load_dwordx4 v[64:67], v27, s[68:69] offset:0\n\t"
        "v_mfma_f32_16x16x32_bf16 v[248:251], v[184:187], v[36:39], v[248:251]\n\t"
        "ds_read_b64_tr_b16 v[154:155], v22 offset:33792\n\t"
        "ds_add_u32 %4, v31\n\t"
        "v_mfma_f32_16x16x32_bf16 v[236:239], v[188:191], v[32:35], v[236:239]\n\t"
        "ds_read_b64_tr_b16 v[156:157], v23 offset:32768\n\t"
        "global_load_dwordx4 v[68:71], v27, s[68:69] offset:1024\n\t"
        "v_mfma_f32_16x16x32_bf16 v[252:255], v[188:191], v[36:39], v[252:255]\n\t"
        "ds_read_b64_tr_b16 v[158:159], v23 offset:33792\n\t"
        "s_waitcnt vmcnt(6) lgkmcnt(15)\n\t"
        "v_mfma_f32_16x16x32_bf16 v[192:195], v[128:131], v[40:43], v[192:195]\n\t"
        "ds_read_b64_tr_b16 v[160:161], v16 offset:40960\n\t"
        "v_mfma_f32_16x16x32_bf16 v[208:211], v[128:131], v[44:47], v[208:211]\n\t"
        "ds_read_b64_tr_b16 v[162:163], v16 offset:41984\n\t"
        "v_mfma_f32_16x16x32_bf16 v[196:199], v[132:135], v[40:43], v[196:199]\n\t"
        "ds_read_b64_tr_b16 v[164:165], v17 offset:40960\n\t"
        "v_mfma_f32_16x16x32_bf16 v[212:215], v[132:135], v[44:47], v[212:215]\n\t"
        "ds_read_b64_tr_b16 v[166:167], v17 offset:41984\n\t"
        "s_waitcnt lgkmcnt(10)\n\t"
        "v_mfma_f32_16x16x32_bf16 v[200:203], v[136:139], v[40:43], v[200:203]\n\t"
        "ds_read_b64_tr_b16 v[168:169], v18 offset:40960\n\t"
        "s_movk_i32 s61, 14\n\t"
        "v_mfma_f32_16x16x32_bf16 v[216:219], v[136:139], v[44:47], v[216:219]\n\t"
        "ds_read_b64_tr_b16 v[170:171], v18 offset:41984\n\t"
        "v_cmp_ge_u32 vcc, s61, v121\n\t"
        "v_mfma_f32_16x16x32_bf16 v[204:207], v[140:143], v[40:43], v[204:207]\n\t"
        "ds_read_b64_tr_b16 v[172:173], v19 offset:40960\n\t"
        "s_getpc_b64 s[70:71]\n\t"
        "s_cbranch_vccnz .Lspin1_%=\n\t"
        "v_mfma_f32_16x16x32_bf16 v[220:223], v[140:143], v[44:47], v[220:223]\n\t"
        "ds_read_b64_tr_b16 v[174:175], v19 offset:41984\n\t"
        "s_movk_i32 s61, 16\n\t"
        "s_waitcnt lgkmcnt(13)\n\t"
        "v_mfma_f32_16x16x32_bf16 v[224:227], v[144:147], v[40:43], v[224:227]\n\t"
        "ds_read_b64_tr_b16 v[176:177], v20 offset:40960\n\t"
        "v_cmp_eq_u32 vcc, 0, v24\n\t"
        "v_mfma_f32_16x16x32_bf16 v[240:243], v[144:147], v[44:47], v[240:243]\n\t"
        "ds_read_b64_tr_b16 v[178:179], v20 offset:41984\n\t"
        "s_getpc_b64 s[70:71]\n\t"
        "s_cbranch_vccnz .Lspin4_%=\n\t"
        "v_mfma_f32_16x16x32_bf16 v[228:231], v[148:151], v[40:43], v[228:231]\n\t"
        "ds_read_b64_tr_b16 v[180:181], v21 offset:40960\n\t"
        "ds_read_b32 v121, v127 offset:4\n\t"
        "ds_read_b32 v25, v127 offset:324\n\t"
        "v_mfma_f32_16x16x32_bf16 v[244:247], v[148:151], v[44:47], v[244:247]\n\t"
        "ds_read_b64_tr_b16 v[182:183], v21 offset:41984\n\t"
        "v_add_u32 v24, v24, %2\n\t"
        "s_waitcnt lgkmcnt(14)\n\t"
        "v_mfma_f32_16x16x32_bf16 v[232:235], v[152:155], v[40:43], v[232:235]\n\t"
        "ds_read_b64_tr_b16 v[184:185], v22 offset:40960\n\t"
        "global_load_dwordx4 v[72:75], v24, s[68:69] offset:0\n\t"
        "v_mfma_f32_16x16x32_bf16 v[248:251], v[152:155], v[44:47], v[248:251]\n\t"
        "ds_read_b64_tr_b16 v[186:187], v22 offset:41984\n\t"
        "ds_add_u32 %4, v31\n\t"
        "v_mfma_f32_16x16x32_bf16 v[236:239], v[156:159], v[40:43], v[236:239]\n\t"
        "ds_read_b64_tr_b16 v[188:189], v23 offset:40960\n\t"
        "global_load_dwordx4 v[76:79], v24, s[68:69] offset:1024\n\t"
        "v_mfma_f32_16x16x32_bf16 v[252:255], v[156:159], v[44:47], v[252:255]\n\t"
        "ds_read_b64_tr_b16 v[190:191], v23 offset:41984\n\t"
        "s_waitcnt vmcnt(6) lgkmcnt(15)\n\t"
        "v_mfma_f32_16x16x32_bf16 v[192:195], v[160:163], v[48:51], v[192:195]\n\t"
        "ds_read_b64_tr_b16 v[128:129], v16 offset:49152\n\t"
        "v_mfma_f32_16x16x32_bf16 v[208:211], v[160:163], v[52:55], v[208:211]\n\t"
        "ds_read_b64_tr_b16 v[130:131], v16 offset:50176\n\t"
        "v_mfma_f32_16x16x32_bf16 v[196:199], v[164:167], v[48:51], v[196:199]\n\t"
        "ds_read_b64_tr_b16 v[132:133], v17 offset:49152\n\t"
        "v_mfma_f32_16x16x32_bf16 v[212:215], v[164:167], v[52:55], v[212:215]\n\t"
        "ds_read_b64_tr_b16 v[134:135], v17 offset:50176\n\t"
        "s_waitcnt lgkmcnt(10)\n\t"
        "v_mfma_f32_16x16x32_bf16 v[200:203], v[168:171], v[48:51], v[200:203]\n\t"
        "ds_read_b64_tr_b16 v[136:137], v18 offset:49152\n\t"
        "s_movk_i32 s61, 15\n\t"
        "v_mfma_f32_16x16x32_bf16 v[216:219], v[168:171], v[52:55], v[216:219]\n\t"
        "ds_read_b64_tr_b16 v[138:139], v18 offset:50176\n\t"
        "v_cmp_ge_u32 vcc, s61, v121\n\t"
        "v_mfma_f32_16x16x32_bf16 v[204:207], v[172:175], v[48:51], v[204:207]\n\t"
        "ds_read_b64_tr_b16 v[140:141], v19 offset:49152\n\t"
        "s_getpc_b64 s[70:71]\n\t"
        "s_cbranch_vccnz .Lspin1_%=\n\t"
        "v_mfma_f32_16x16x32_bf16 v[220:223], v[172:175], v[52:55], v[220:223]\n\t"
        "ds_read_b64_tr_b16 v[142:143], v19 offset:50176\n\t"
        "s_movk_i32 s61, 17\n\t"
        "s_waitcnt lgkmcnt(13)\n\t"
        "v_mfma_f32_16x16x32_bf16 v[224:227], v[176:179], v[48:51], v[224:227]\n\t"
        "ds_read_b64_tr_b16 v[144:145], v20 offset:49152\n\t"
        "v_cmp_eq_u32 vcc, 0, v25\n\t"
        "v_mfma_f32_16x16x32_bf16 v[240:243], v[176:179], v[52:55], v[240:243]\n\t"
        "ds_read_b64_tr_b16 v[146:147], v20 offset:50176\n\t"
        "s_getpc_b64 s[70:71]\n\t"
        "s_cbranch_vccnz .Lspin5_%=\n\t"
        "v_mfma_f32_16x16x32_bf16 v[228:231], v[180:183], v[48:51], v[228:231]\n\t"
        "ds_read_b64_tr_b16 v[148:149], v21 offset:49152\n\t"
        "ds_read_b32 v121, v127 offset:4\n\t"
        "ds_read_b32 v26, v127 offset:328\n\t"
        "v_mfma_f32_16x16x32_bf16 v[244:247], v[180:183], v[52:55], v[244:247]\n\t"
        "ds_read_b64_tr_b16 v[150:151], v21 offset:50176\n\t"
        "v_add_u32 v25, v25, %2\n\t"
        "s_waitcnt lgkmcnt(14)\n\t"
        "v_mfma_f32_16x16x32_bf16 v[232:235], v[184:187], v[48:51], v[232:235]\n\t"
        "ds_read_b64_tr_b16 v[152:153], v22 offset:49152\n\t"
        "global_load_dwordx4 v[80:83], v25, s[68:69] offset:0\n\t"
        "v_mfma_f32_16x16x32_bf16 v[248:251], v[184:187], v[52:55], v[248:251]\n\t"
        "ds_read_b64_tr_b16 v[154:155], v22 offset:50176\n\t"
        "ds_add_u32 %4, v31\n\t"
        "v_mfma_f32_16x16x32_bf16 v[236:239], v[188:191], v[48:51], v[236:239]\n\t"
        "ds_read_b64_tr_b16 v[156:157], v23 offset:49152\n\t"
        "global_load_dwordx4 v[84:87], v25, s[68:69] offset:1024\n\t"
        "v_mfma_f32_16x16x32_bf16 v[252:255], v[188:191], v[52:55], v[252:255]\n\t"
        "ds_read_b64_tr_b16 v[158:159], v23 offset:50176\n\t"
        "s_waitcnt vmcnt(6) lgkmcnt(15)\n\t"
        "v_mfma_f32_16x16x32_bf16 v[192:195], v[128:131], v[56:59], v[192:195]\n\t"
        "ds_read_b64_tr_b16 v[160:161], v16 offset:57344\n\t"
        "v_mfma_f32_16x16x32_bf16 v[208:211], v[128:131], v[60:63], v[208:211]\n\t"
        "ds_read_b64_tr_b16 v[162:163], v16 offset:58368\n\t"
        "v_mfma_f32_16x16x32_bf16 v[196:199], v[132:135], v[56:59], v[196:199]\n\t"
        "ds_read_b64_tr_b16 v[164:165], v17 offset:57344\n\t"
        "v_mfma_f32_16x16x32_bf16 v[212:215], v[132:135], v[60:63], v[212:215]\n\t"
        "ds_read_b64_tr_b16 v[166:167], v17 offset:58368\n\t"
        "s_waitcnt lgkmcnt(10)\n\t"
        "v_mfma_f32_16x16x32_bf16 v[200:203], v[136:139], v[56:59], v[200:203]\n\t"
        "ds_read_b64_tr_b16 v[168:169], v18 offset:57344\n\t"
        "s_movk_i32 s61, 16\n\t"
        "v_mfma_f32_16x16x32_bf16 v[216:219], v[136:139], v[60:63], v[216:219]\n\t"
        "ds_read_b64_tr_b16 v[170:171], v18 offset:58368\n\t"
        "v_cmp_ge_u32 vcc, s61, v121\n\t"
        "v_mfma_f32_16x16x32_bf16 v[204:207], v[140:143], v[56:59], v[204:207]\n\t"
        "ds_read_b64_tr_b16 v[172:173], v19 offset:57344\n\t"
        "s_getpc_b64 s[70:71]\n\t"
        "s_cbranch_vccnz .Lspin1_%=\n\t"
        "v_mfma_f32_16x16x32_bf16 v[220:223], v[140:143], v[60:63], v[220:223]\n\t"
        "ds_read_b64_tr_b16 v[174:175], v19 offset:58368\n\t"
        "s_movk_i32 s61, 18\n\t"
        "s_waitcnt lgkmcnt(13)\n\t"
        "v_mfma_f32_16x16x32_bf16 v[224:227], v[144:147], v[56:59], v[224:227]\n\t"
        "ds_read_b64_tr_b16 v[176:177], v20 offset:57344\n\t"
        "v_cmp_eq_u32 vcc, 0, v26\n\t"
        "v_mfma_f32_16x16x32_bf16 v[240:243], v[144:147], v[60:63], v[240:243]\n\t"
        "ds_read_b64_tr_b16 v[178:179], v20 offset:58368\n\t"
        "s_getpc_b64 s[70:71]\n\t"
        "s_cbranch_vccnz .Lspin6_%=\n\t"
        "v_mfma_f32_16x16x32_bf16 v[228:231], v[148:151], v[56:59], v[228:231]\n\t"
        "ds_read_b64_tr_b16 v[180:181], v21 offset:57344\n\t"
        "ds_read_b32 v121, v127 offset:4\n\t"
        "ds_read_b32 v27, v127 offset:332\n\t"
        "v_mfma_f32_16x16x32_bf16 v[244:247], v[148:151], v[60:63], v[244:247]\n\t"
        "ds_read_b64_tr_b16 v[182:183], v21 offset:58368\n\t"
        "v_add_u32 v26, v26, %2\n\t"
        "s_waitcnt lgkmcnt(14)\n\t"
        "v_mfma_f32_16x16x32_bf16 v[232:235], v[152:155], v[56:59], v[232:235]\n\t"
        "ds_read_b64_tr_b16 v[184:185], v22 offset:57344\n\t"
        "global_load_dwordx4 v[88:91], v26, s[68:69] offset:0\n\t"
        "v_mfma_f32_16x16x32_bf16 v[248:251], v[152:155], v[60:63], v[248:251]\n\t"
        "ds_read_b64_tr_b16 v[186:187], v22 offset:58368\n\t"
        "ds_add_u32 %4, v31\n\t"
        "v_mfma_f32_16x16x32_bf16 v[236:239], v[156:159], v[56:59], v[236:239]\n\t"
        "ds_read_b64_tr_b16 v[188:189], v23 offset:57344\n\t"
        "global_load_dwordx4 v[92:95], v26, s[68:69] offset:1024\n\t"
        "v_mfma_f32_16x16x32_bf16 v[252:255], v[156:159], v[60:63], v[252:255]\n\t"
        "ds_read_b64_tr_b16 v[190:191], v23 offset:58368\n\t"
        "s_waitcnt vmcnt(6) lgkmcnt(15)\n\t"
        "v_mfma_f32_16x16x32_bf16 v[192:195], v[160:163], v[64:67], v[192:195]\n\t"
        "ds_read_b64_tr_b16 v[128:129], v8 offset:0\n\t"
        "v_mfma_f32_16x16x32_bf16 v[208:211], v[160:163], v[68:71], v[208:211]\n\t"
        "ds_read_b64_tr_b16 v[130:131], v8 offset:1024\n\t"
        "v_mfma_f32_16x16x32_bf16 v[196:199], v[164:167], v[64:67], v[196:199]\n\t"
        "ds_read_b64_tr_b16 v[132:133], v9 offset:0\n\t"
        "v_mfma_f32_16x16x32_bf16 v[212:215], v[164:167], v[68:71], v[212:215]\n\t"
        "ds_read_b64_tr_b16 v[134:135], v9 offset:1024\n\t"
        "s_waitcnt lgkmcnt(10)\n\t"
        "v_mfma_f32_16x16x32_bf16 v[200:203], v[168:171], v[64:67], v[200:203]\n\t"
        "ds_read_b64_tr_b16 v[136:137], v10 offset:0\n\t"
        "s_movk_i32 s61, 17\n\t"
        "v_mfma_f32_16x16x32_bf16 v[216:219], v[168:171], v[68:71], v[216:219]\n\t"
        "ds_read_b64_tr_b16 v[138:139], v10 offset:1024\n\t"
        "v_cmp_ge_u32 vcc, s61, v121\n\t"
        "v_mfma_f32_16x16x32_bf16 v[204:207], v[172:175], v[64:67], v[204:207]\n\t"
        "ds_read_b64_tr_b16 v[140:141], v11 offset:0\n\t"
        "s_getpc_b64 s[70:71]\n\t"
        "s_cbranch_vccnz .Lspin1_%=\n\t"
        "v_mfma_f32_16x16x32_bf16 v[220:223], v[172:175], v[68:71], v[220:223]\n\t"
        "ds_read_b64_tr_b16 v[142:143], v11 offset:1024\n\t"
        "s_movk_i32 s61, 19\n\t"
        "s_waitcnt lgkmcnt(13)\n\t"
        "v_mfma_f32_16x16x32_bf16 v[224:227], v[176:179], v[64:67], v[224:227]\n\t"
        "ds_read_b64_tr_b16 v[144:145], v12 offset:0\n\t"
        "v_cmp_eq_u32 vcc, 0, v27\n\t"
        "v_mfma_f32_16x16x32_bf16 v[240:243], v[176:179], v[68:71], v[240:243]\n\t"
        "ds_read_b64_tr_b16 v[146:147], v12 offset:1024\n\t"
        "s_getpc_b64 s[70:71]\n\t"
        "s_cbranch_vccnz .Lspin7_%=\n\t"
        "v_mfma_f32_16x16x32_bf16 v[228:231], v[180:183], v[64:67], v[228:231]\n\t"
        "ds_read_b64_tr_b16 v[148:149], v13 offset:0\n\t"
        "ds_read_b32 v121, v127 offset:4\n\t"
        "ds_read_b32 v24, v127 offset:336\n\t"
        "v_mfma_f32_16x16x32_bf16 v[244:247], v[180:183], v[68:71], v[244:247]\n\t"
        "ds_read_b64_tr_b16 v[150:151], v13 offset:1024\n\t"
        "v_add_u32 v27, v27, %2\n\t"
        "s_waitcnt lgkmcnt(14)\n\t"
        "v_mfma_f32_16x16x32_bf16 v[232:235], v[184:187], v[64:67], v[232:235]\n\t"
        "ds_read_b64_tr_b16 v[152:153], v14 offset:0\n\t"
        "global_load_dwordx4 v[96:99], v27, s[68:69] offset:0\n\t"
        "v_mfma_f32_16x16x32_bf16 v[248:251], v[184:187], v[68:71], v[248:251]\n\t"
        "ds_read_b64_tr_b16 v[154:155], v14 offset:1024\n\t"
        "ds_add_u32 %4, v31\n\t"
        "v_mfma_f32_16x16x32_bf16 v[236:239], v[188:191], v[64:67], v[236:239]\n\t"
        "ds_read_b64_tr_b16 v[156:157], v15 offset:0\n\t"
        "global_load_dwordx4 v[100:103], v27, s[68:69] offset:1024\n\t"
        "v_mfma_f32_16x16x32_bf16 v[252:255], v[188:191], v[68:71], v[252:255]\n\t"
        "ds_read_b64_tr_b16 v[158:159], v15 offset:1024\n\t"
        "s_waitcnt vmcnt(6) lgkmcnt(15)\n\t"
        "v_mfma_f32_16x16x32_bf16 v[192:195], v[128:131], v[72:75], v[192:195]\n\t"
        "ds_read_b64_tr_b16 v[160:161], v8 offset:8192\n\t"
        "v_mfma_f32_16x16x32_bf16 v[208:211], v[128:131], v[76:79], v[208:211]\n\t"
        "ds_read_b64_tr_b16 v[162:163], v8 offset:9216\n\t"
        "v_mfma_f32_16x16x32_bf16 v[196:199], v[132:135], v[72:75], v[196:199]\n\t"
        "ds_read_b64_tr_b16 v[164:165], v9 offset:8192\n\t"
        "v_mfma_f32_16x16x32_bf16 v[212:215], v[132:135], v[76:79], v[212:215]\n\t"
        "ds_read_b64_tr_b16 v[166:167], v9 offset:9216\n\t"
        "s_waitcnt lgkmcnt(10)\n\t"
        "v_mfma_f32_16x16x32_bf16 v[200:203], v[136:139], v[72:75], v[200:203]\n\t"
        "ds_read_b64_tr_b16 v[168:169], v10 offset:8192\n\t"
        "s_movk_i32 s61, 18\n\t"
        "v_mfma_f32_16x16x32_bf16 v[216:219], v[136:139], v[76:79], v[216:219]\n\t"
        "ds_read_b64_tr_b16 v[170:171], v10 offset:9216\n\t"
        "v_cmp_ge_u32 vcc, s61, v121\n\t"
        "v_mfma_f32_16x16x32_bf16 v[204:207], v[140:143], v[72:75], v[204:207]\n\t"
        "ds_read_b64_tr_b16 v[172:173], v11 offset:8192\n\t"
        "s_getpc_b64 s[70:71]\n\t"
        "s_cbranch_vccnz .Lspin1_%=\n\t"
        "v_mfma_f32_16x16x32_bf16 v[220:223], v[140:143], v[76:79], v[220:223]\n\t"
        "ds_read_b64_tr_b16 v[174:175], v11 offset:9216\n\t"
        "s_movk_i32 s61, 20\n\t"
        "s_waitcnt lgkmcnt(13)\n\t"
        "v_mfma_f32_16x16x32_bf16 v[224:227], v[144:147], v[72:75], v[224:227]\n\t"
        "ds_read_b64_tr_b16 v[176:177], v12 offset:8192\n\t"
        "v_cmp_eq_u32 vcc, 0, v24\n\t"
        "v_mfma_f32_16x16x32_bf16 v[240:243], v[144:147], v[76:79], v[240:243]\n\t"
        "ds_read_b64_tr_b16 v[178:179], v12 offset:9216\n\t"
        "s_getpc_b64 s[70:71]\n\t"
        "s_cbranch_vccnz .Lspin4_%=\n\t"
        "v_mfma_f32_16x16x32_bf16 v[228:231], v[148:151], v[72:75], v[228:231]\n\t"
        "ds_read_b64_tr_b16 v[180:181], v13 offset:8192\n\t"
        "ds_read_b32 v121, v127 offset:4\n\t"
        "ds_read_b32 v25, v127 offset:340\n\t"
        "v_mfma_f32_16x16x32_bf16 v[244:247], v[148:151], v[76:79], v[244:247]\n\t"
        "ds_read_b64_tr_b16 v[182:183], v13 offset:9216\n\t"
        "v_add_u32 v24, v24, %2\n\t"
        "s_waitcnt lgkmcnt(14)\n\t"
        "v_mfma_f32_16x16x32_bf16 v[232:235], v[152:155], v[72:75], v[232:235]\n\t"
        "ds_read_b64_tr_b16 v[184:185], v14 offset:8192\n\t"
        "global_load_dwordx4 v[104:107], v24, s[68:69] offset:0\n\t"
        "v_mfma_f32_16x16x32_bf16 v[248:251], v[152:155], v[76:79], v[248:251]\n\t"
        "ds_read_b64_tr_b16 v[186:187], v14 offset:9216\n\t"
        "ds_add_u32 %4, v31\n\t"
        "v_mfma_f32_16x16x32_bf16 v[236:239], v[156:159], v[72:75], v[236:239]\n\t"
        "ds_read_b64_tr_b16 v[188:189], v15 offset:8192\n\t"
        "global_load_dwordx4 v[108:111], v24, s[68:69] offset:1024\n\t"
        "v_mfma_f32_16x16x32_bf16 v[252:255], v[156:159], v[76:79], v[252:255]\n\t"
        "ds_read_b64_tr_b16 v[190:191], v15 offset:9216\n\t"
        "s_waitcnt vmcnt(6) lgkmcnt(15)\n\t"
        "v_mfma_f32_16x16x32_bf16 v[192:195], v[160:163], v[80:83], v[192:195]\n\t"
        "ds_read_b64_tr_b16 v[128:129], v8 offset:16384\n\t"
        "v_mfma_f32_16x16x32_bf16 v[208:211], v[160:163], v[84:87], v[208:211]\n\t"
        "ds_read_b64_tr_b16 v[130:131], v8 offset:17408\n\t"
        "v_mfma_f32_16x16x32_bf16 v[196:199], v[164:167], v[80:83], v[196:199]\n\t"
        "ds_read_b64_tr_b16 v[132:133], v9 offset:16384\n\t"
        "v_mfma_f32_16x16x32_bf16 v[212:215], v[164:167], v[84:87], v[212:215]\n\t"
        "ds_read_b64_tr_b16 v[134:135], v9 offset:17408\n\t"
        "s_waitcnt lgkmcnt(10)\n\t"
        "v_mfma_f32_16x16x32_bf16 v[200:203], v[168:171], v[80:83], v[200:203]\n\t"
        "ds_read_b64_tr_b16 v[136:137], v10 offset:16384\n\t"
        "s_movk_i32 s61, 19\n\t"
        "v_mfma_f32_16x16x32_bf16 v[216:219], v[168:171], v[84:87], v[216:219]\n\t"
        "ds_read_b64_tr_b16 v[138:139], v10 offset:17408\n\t"
        "v_cmp_ge_u32 vcc, s61, v121\n\t"
        "v_mfma_f32_16x16x32_bf16 v[204:207], v[172:175], v[80:83], v[204:207]\n\t"
        "ds_read_b64_tr_b16 v[140:141], v11 offset:16384\n\t"
        "s_getpc_b64 s[70:71]\n\t"
        "s_cbranch_vccnz .Lspin1_%=\n\t"
        "v_mfma_f32_16x16x32_bf16 v[220:223], v[172:175], v[84:87], v[220:223]\n\t"
        "ds_read_b64_tr_b16 v[142:143], v11 offset:17408\n\t"
        "s_movk_i32 s61, 21\n\t"
        "s_waitcnt lgkmcnt(13)\n\t"
        "v_mfma_f32_16x16x32_bf16 v[224:227], v[176:179], v[80:83], v[224:227]\n\t"
        "ds_read_b64_tr_b16 v[144:145], v12 offset:16384\n\t"
        "v_cmp_eq_u32 vcc, 0, v25\n\t"
        "v_mfma_f32_16x16x32_bf16 v[240:243], v[176:179], v[84:87], v[240:243]\n\t"
        "ds_read_b64_tr_b16 v[146:147], v12 offset:17408\n\t"
        "s_getpc_b64 s[70:71]\n\t"
        "s_cbranch_vccnz .Lspin5_%=\n\t"
        "v_mfma_f32_16x16x32_bf16 v[228:231], v[180:183], v[80:83], v[228:231]\n\t"
        "ds_read_b64_tr_b16 v[148:149], v13 offset:16384\n\t"
        "ds_read_b32 v121, v127 offset:4\n\t"
        "ds_read_b32 v26, v127 offset:344\n\t"
        "v_mfma_f32_16x16x32_bf16 v[244:247], v[180:183], v[84:87], v[244:247]\n\t"
        "ds_read_b64_tr_b16 v[150:151], v13 offset:17408\n\t"
        "v_add_u32 v25, v25, %2\n\t"
        "s_waitcnt lgkmcnt(14)\n\t"
        "v_mfma_f32_16x16x32_bf16 v[232:235], v[184:187], v[80:83], v[232:235]\n\t"
        "ds_read_b64_tr_b16 v[152:153], v14 offset:16384\n\t"
        "global_load_dwordx4 v[112:115], v25, s[68:69] offset:0\n\t"
        "v_mfma_f32_16x16x32_bf16 v[248:251], v[184:187], v[84:87], v[248:251]\n\t"
        "ds_read_b64_tr_b16 v[154:155], v14 offset:17408\n\t"
        "ds_add_u32 %4, v31\n\t"
        "v_mfma_f32_16x16x32_bf16 v[236:239], v[188:191], v[80:83], v[236:239]\n\t"
        "ds_read_b64_tr_b16 v[156:157], v15 offset:16384\n\t"
        "global_load_dwordx4 v[116:119], v25, s[68:69] offset:1024\n\t"
        "v_mfma_f32_16x16x32_bf16 v[252:255], v[188:191], v[84:87], v[252:255]\n\t"
        "ds_read_b64_tr_b16 v[158:159], v15 offset:17408\n\t"
        "s_waitcnt vmcnt(6) lgkmcnt(15)\n\t"
        "v_mfma_f32_16x16x32_bf16 v[192:195], v[128:131], v[88:91], v[192:195]\n\t"
        "ds_read_b64_tr_b16 v[160:161], v8 offset:24576\n\t"
        "v_mfma_f32_16x16x32_bf16 v[208:211], v[128:131], v[92:95], v[208:211]\n\t"
        "ds_read_b64_tr_b16 v[162:163], v8 offset:25600\n\t"
        "v_mfma_f32_16x16x32_bf16 v[196:199], v[132:135], v[88:91], v[196:199]\n\t"
        "ds_read_b64_tr_b16 v[164:165], v9 offset:24576\n\t"
        "v_mfma_f32_16x16x32_bf16 v[212:215], v[132:135], v[92:95], v[212:215]\n\t"
        "ds_read_b64_tr_b16 v[166:167], v9 offset:25600\n\t"
        "s_waitcnt lgkmcnt(10)\n\t"
        "v_mfma_f32_16x16x32_bf16 v[200:203], v[136:139], v[88:91], v[200:203]\n\t"
        "ds_read_b64_tr_b16 v[168:169], v10 offset:24576\n\t"
        "s_movk_i32 s61, 20\n\t"
        "v_mfma_f32_16x16x32_bf16 v[216:219], v[136:139], v[92:95], v[216:219]\n\t"
        "ds_read_b64_tr_b16 v[170:171], v10 offset:25600\n\t"
        "v_cmp_ge_u32 vcc, s61, v121\n\t"
        "v_mfma_f32_16x16x32_bf16 v[204:207], v[140:143], v[88:91], v[204:207]\n\t"
        "ds_read_b64_tr_b16 v[172:173], v11 offset:24576\n\t"
        "s_getpc_b64 s[70:71]\n\t"
        "s_cbranch_vccnz .Lspin1_%=\n\t"
        "v_mfma_f32_16x16x32_bf16 v[220:223], v[140:143], v[92:95], v[220:223]\n\t"
        "ds_read_b64_tr_b16 v[174:175], v11 offset:25600\n\t"
        "s_movk_i32 s61, 22\n\t"
        "s_waitcnt lgkmcnt(13)\n\t"
        "v_mfma_f32_16x16x32_bf16 v[224:227], v[144:147], v[88:91], v[224:227]\n\t"
        "ds_read_b64_tr_b16 v[176:177], v12 offset:24576\n\t"
        "v_cmp_eq_u32 vcc, 0, v26\n\t"
        "v_mfma_f32_16x16x32_bf16 v[240:243], v[144:147], v[92:95], v[240:243]\n\t"
        "ds_read_b64_tr_b16 v[178:179], v12 offset:25600\n\t"
        "s_getpc_b64 s[70:71]\n\t"
        "s_cbranch_vccnz .Lspin6_%=\n\t"
        "v_mfma_f32_16x16x32_bf16 v[228:231], v[148:151], v[88:91], v[228:231]\n\t"
        "ds_read_b64_tr_b16 v[180:181], v13 offset:24576\n\t"
        "ds_read_b32 v121, v127 offset:4\n\t"
        "ds_read_b32 v27, v127 offset:348\n\t"
        "v_mfma_f32_16x16x32_bf16 v[244:247], v[148:151], v[92:95], v[244:247]\n\t"
        "ds_read_b64_tr_b16 v[182:183], v13 offset:25600\n\t"
        "v_add_u32 v26, v26, %2\n\t"
        "s_waitcnt lgkmcnt(14)\n\t"
        "v_mfma_f32_16x16x32_bf16 v[232:235], v[152:155], v[88:91], v[232:235]\n\t"
        "ds_read_b64_tr_b16 v[184:185], v14 offset:24576\n\t"
        "global_load_dwordx4 v[32:35], v26, s[68:69] offset:0\n\t"
        "v_mfma_f32_16x16x32_bf16 v[248:251], v[152:155], v[92:95], v[248:251]\n\t"
        "ds_read_b64_tr_b16 v[186:187], v14 offset:25600\n\t"
        "ds_add_u32 %4, v31\n\t"
        "v_mfma_f32_16x16x32_bf16 v[236:239], v[156:159], v[88:91], v[236:239]\n\t"
        "ds_read_b64_tr_b16 v[188:189], v15 offset:24576\n\t"
        "global_load_dwordx4 v[36:39], v26, s[68:69] offset:1024\n\t"
        "v_mfma_f32_16x16x32_bf16 v[252:255], v[156:159], v[92:95], v[252:255]\n\t"
        "ds_read_b64_tr_b16 v[190:191], v15 offset:25600\n\t"
        "s_waitcnt vmcnt(6) lgkmcnt(15)\n\t"
        "v_mfma_f32_16x16x32_bf16 v[192:195], v[160:163], v[96:99], v[192:195]\n\t"
        "ds_read_b64_tr_b16 v[128:129], v8 offset:32768\n\t"
        "v_mfma_f32_16x16x32_bf16 v[208:211], v[160:163], v[100:103], v[208:211]\n\t"
        "ds_read_b64_tr_b16 v[130:131], v8 offset:33792\n\t"
        "v_mfma_f32_16x16x32_bf16 v[196:199], v[164:167], v[96:99], v[196:199]\n\t"
        "ds_read_b64_tr_b16 v[132:133], v9 offset:32768\n\t"
        "v_mfma_f32_16x16x32_bf16 v[212:215], v[164:167], v[100:103], v[212:215]\n\t"
        "ds_read_b64_tr_b16 v[134:135], v9 offset:33792\n\t"
        "s_waitcnt lgkmcnt(10)\n\t"
        "v_mfma_f32_16x16x32_bf16 v[200:203], v[168:171], v[96:99], v[200:203]\n\t"
        "ds_read_b64_tr_b16 v[136:137], v10 offset:32768\n\t"
        "s_movk_i32 s61, 21\n\t"
        "v_mfma_f32_16x16x32_bf16 v[216:219], v[168:171], v[100:103], v[216:219]\n\t"
        "ds_read_b64_tr_b16 v[138:139], v10 offset:33792\n\t"
        "v_cmp_ge_u32 vcc, s61, v121\n\t"
        "v_mfma_f32_16x16x32_bf16 v[204:207], v[172:175], v[96:99], v[204:207]\n\t"
        "ds_read_b64_tr_b16 v[140:141], v11 offset:32768\n\t"
        "s_getpc_b64 s[70:71]\n\t"
        "s_cbranch_vccnz .Lspin1_%=\n\t"
        "v_mfma_f32_16x16x32_bf16 v[220:223], v[172:175], v[100:103], v[220:223]\n\t"
        "ds_read_b64_tr_b16 v[142:143], v11 offset:33792\n\t"
        "s_movk_i32 s61, 23\n\t"
        "s_waitcnt lgkmcnt(13)\n\t"
        "v_mfma_f32_16x16x32_bf16 v[224:227], v[176:179], v[96:99], v[224:227]\n\t"
        "ds_read_b64_tr_b16 v[144:145], v12 offset:32768\n\t"
        "v_cmp_eq_u32 vcc, 0, v27\n\t"
        "v_mfma_f32_16x16x32_bf16 v[240:243], v[176:179], v[100:103], v[240:243]\n\t"
        "ds_read_b64_tr_b16 v[146:147], v12 offset:33792\n\t"
        "s_getpc_b64 s[70:71]\n\t"
        "s_cbranch_vccnz .Lspin7_%=\n\t"
        "v_mfma_f32_16x16x32_bf16 v[228:231], v[180:183], v[96:99], v[228:231]\n\t"
        "ds_read_b64_tr_b16 v[148:149], v13 offset:32768\n\t"
        "ds_read_b32 v121, v127 offset:4\n\t"
        "ds_read_b32 v24, v127 offset:352\n\t"
        "v_mfma_f32_16x16x32_bf16 v[244:247], v[180:183], v[100:103], v[244:247]\n\t"
        "ds_read_b64_tr_b16 v[150:151], v13 offset:33792\n\t"
        "v_add_u32 v27, v27, %2\n\t"
        "s_waitcnt lgkmcnt(14)\n\t"
        "v_mfma_f32_16x16x32_bf16 v[232:235], v[184:187], v[96:99], v[232:235]\n\t"
        "ds_read_b64_tr_b16 v[152:153], v14 offset:32768\n\t"
        "global_load_dwordx4 v[40:43], v27, s[68:69] offset:0\n\t"
        "v_mfma_f32_16x16x32_bf16 v[248:251], v[184:187], v[100:103], v[248:251]\n\t"
        "ds_read_b64_tr_b16 v[154:155], v14 offset:33792\n\t"
        "ds_add_u32 %4, v31\n\t"
        "v_mfma_f32_16x16x32_bf16 v[236:239], v[188:191], v[96:99], v[236:239]\n\t"
        "ds_read_b64_tr_b16 v[156:157], v15 offset:32768\n\t"
        "global_load_dwordx4 v[44:47], v27, s[68:69] offset:1024\n\t"
        "v_mfma_f32_16x16x32_bf16 v[252:255], v[188:191], v[100:103], v[252:255]\n\t"
        "ds_read_b64_tr_b16 v[158:159], v15 offset:33792\n\t"
        "s_waitcnt vmcnt(6) lgkmcnt(15)\n\t"
        "v_mfma_f32_16x16x32_bf16 v[192:195], v[128:131], v[104:107], v[192:195]\n\t"
        "ds_read_b64_tr_b16 v[160:161], v8 offset:40960\n\t"
        "v_mfma_f32_16x16x32_bf16 v[208:211], v[128:131], v[108:111], v[208:211]\n\t"
        "ds_read_b64_tr_b16 v[162:163], v8 offset:41984\n\t"
        "v_mfma_f32_16x16x32_bf16 v[196:199], v[132:135], v[104:107], v[196:199]\n\t"
        "ds_read_b64_tr_b16 v[164:165], v9 offset:40960\n\t"
        "v_mfma_f32_16x16x32_bf16 v[212:215], v[132:135], v[108:111], v[212:215]\n\t"
        "ds_read_b64_tr_b16 v[166:167], v9 offset:41984\n\t"
        "s_waitcnt lgkmcnt(10)\n\t"
        "v_mfma_f32_16x16x32_bf16 v[200:203], v[136:139], v[104:107], v[200:203]\n\t"
        "ds_read_b64_tr_b16 v[168:169], v10 offset:40960\n\t"
        "s_movk_i32 s61, 22\n\t"
        "v_mfma_f32_16x16x32_bf16 v[216:219], v[136:139], v[108:111], v[216:219]\n\t"
        "ds_read_b64_tr_b16 v[170:171], v10 offset:41984\n\t"
        "v_cmp_ge_u32 vcc, s61, v121\n\t"
        "v_mfma_f32_16x16x32_bf16 v[204:207], v[140:143], v[104:107], v[204:207]\n\t"
        "ds_read_b64_tr_b16 v[172:173], v11 offset:40960\n\t"
        "s_getpc_b64 s[70:71]\n\t"
        "s_cbranch_vccnz .Lspin1_%=\n\t"
        "v_mfma_f32_16x16x32_bf16 v[220:223], v[140:143], v[108:111], v[220:223]\n\t"
        "ds_read_b64_tr_b16 v[174:175], v11 offset:41984\n\t"
        "s_movk_i32 s61, 24\n\t"
        "s_waitcnt lgkmcnt(13)\n\t"
        "v_mfma_f32_16x16x32_bf16 v[224:227], v[144:147], v[104:107], v[224:227]\n\t"
        "ds_read_b64_tr_b16 v[176:177], v12 offset:40960\n\t"
        "v_cmp_eq_u32 vcc, 0, v24\n\t"
        "v_mfma_f32_16x16x32_bf16 v[240:243], v[144:147], v[108:111], v[240:243]\n\t"
        "ds_read_b64_tr_b16 v[178:179], v12 offset:41984\n\t"
        "s_getpc_b64 s[70:71]\n\t"
        "s_cbranch_vccnz .Lspin4_%=\n\t"
        "v_mfma_f32_16x16x32_bf16 v[228:231], v[148:151], v[104:107], v[228:231]\n\t"
        "ds_read_b64_tr_b16 v[180:181], v13 offset:40960\n\t"
        "ds_read_b32 v121, v127 offset:4\n\t"
        "ds_read_b32 v25, v127 offset:356\n\t"
        "v_mfma_f32_16x16x32_bf16 v[244:247], v[148:151], v[108:111], v[244:247]\n\t"
        "ds_read_b64_tr_b16 v[182:183], v13 offset:41984\n\t"
        "v_add_u32 v24, v24, %2\n\t"
        "s_waitcnt lgkmcnt(14)\n\t"
        "v_mfma_f32_16x16x32_bf16 v[232:235], v[152:155], v[104:107], v[232:235]\n\t"
        "ds_read_b64_tr_b16 v[184:185], v14 offset:40960\n\t"
        "global_load_dwordx4 v[48:51], v24, s[68:69] offset:0\n\t"
        "v_mfma_f32_16x16x32_bf16 v[248:251], v[152:155], v[108:111], v[248:251]\n\t"
        "ds_read_b64_tr_b16 v[186:187], v14 offset:41984\n\t"
        "ds_add_u32 %4, v31\n\t"
        "v_mfma_f32_16x16x32_bf16 v[236:239], v[156:159], v[104:107], v[236:239]\n\t"
        "ds_read_b64_tr_b16 v[188:189], v15 offset:40960\n\t"
        "global_load_dwordx4 v[52:55], v24, s[68:69] offset:1024\n\t"
        "v_mfma_f32_16x16x32_bf16 v[252:255], v[156:159], v[108:111], v[252:255]\n\t"
        "ds_read_b64_tr_b16 v[190:191], v15 offset:41984\n\t"
        "s_waitcnt vmcnt(6) lgkmcnt(15)\n\t"
        "v_mfma_f32_16x16x32_bf16 v[192:195], v[160:163], v[112:115], v[192:195]\n\t"
        "ds_read_b64_tr_b16 v[128:129], v8 offset:49152\n\t"
        "v_mfma_f32_16x16x32_bf16 v[208:211], v[160:163], v[116:119], v[208:211]\n\t"
        "ds_read_b64_tr_b16 v[130:131], v8 offset:50176\n\t"
        "v_mfma_f32_16x16x32_bf16 v[196:199], v[164:167], v[112:115], v[196:199]\n\t"
        "ds_read_b64_tr_b16 v[132:133], v9 offset:49152\n\t"
        "v_mfma_f32_16x16x32_bf16 v[212:215], v[164:167], v[116:119], v[212:215]\n\t"
        "ds_read_b64_tr_b16 v[134:135], v9 offset:50176\n\t"
        "s_waitcnt lgkmcnt(10)\n\t"
        "v_mfma_f32_16x16x32_bf16 v[200:203], v[168:171], v[112:115], v[200:203]\n\t"
        "ds_read_b64_tr_b16 v[136:137], v10 offset:49152\n\t"
        "s_movk_i32 s61, 23\n\t"
        "v_mfma_f32_16x16x32_bf16 v[216:219], v[168:171], v[116:119], v[216:219]\n\t"
        "ds_read_b64_tr_b16 v[138:139], v10 offset:50176\n\t"
        "v_cmp_ge_u32 vcc, s61, v121\n\t"
        "v_mfma_f32_16x16x32_bf16 v[204:207], v[172:175], v[112:115], v[204:207]\n\t"
        "ds_read_b64_tr_b16 v[140:141], v11 offset:49152\n\t"
        "s_getpc_b64 s[70:71]\n\t"
        "s_cbranch_vccnz .Lspin1_%=\n\t"
        "v_mfma_f32_16x16x32_bf16 v[220:223], v[172:175], v[116:119], v[220:223]\n\t"
        "ds_read_b64_tr_b16 v[142:143], v11 offset:50176\n\t"
        "s_movk_i32 s61, 25\n\t"
        "s_waitcnt lgkmcnt(13)\n\t"
        "v_mfma_f32_16x16x32_bf16 v[224:227], v[176:179], v[112:115], v[224:227]\n\t"
        "ds_read_b64_tr_b16 v[144:145], v12 offset:49152\n\t"
        "v_cmp_eq_u32 vcc, 0, v25\n\t"
        "v_mfma_f32_16x16x32_bf16 v[240:243], v[176:179], v[116:119], v[240:243]\n\t"
        "ds_read_b64_tr_b16 v[146:147], v12 offset:50176\n\t"
        "s_getpc_b64 s[70:71]\n\t"
        "s_cbranch_vccnz .Lspin5_%=\n\t"
        "v_mfma_f32_16x16x32_bf16 v[228:231], v[180:183], v[112:115], v[228:231]\n\t"
        "ds_read_b64_tr_b16 v[148:149], v13 offset:49152\n\t"
        "ds_read_b32 v121, v127 offset:4\n\t"
        "ds_read_b32 v26, v127 offset:360\n\t"
        "v_mfma_f32_16x16x32_bf16 v[244:247], v[180:183], v[116:119], v[244:247]\n\t"
        "ds_read_b64_tr_b16 v[150:151], v13 offset:50176\n\t"
        "v_add_u32 v25, v25, %2\n\t"
        "s_waitcnt lgkmcnt(14)\n\t"
        "v_mfma_f32_16x16x32_bf16 v[232:235], v[184:187], v[112:115], v[232:235]\n\t"
        "ds_read_b64_tr_b16 v[152:153], v14 offset:49152\n\t"
        "global_load_dwordx4 v[56:59], v25, s[68:69] offset:0\n\t"
        "v_mfma_f32_16x16x32_bf16 v[248:251], v[184:187], v[116:119], v[248:251]\n\t"
        "ds_read_b64_tr_b16 v[154:155], v14 offset:50176\n\t"
        "ds_add_u32 %4, v31\n\t"
        "v_mfma_f32_16x16x32_bf16 v[236:239], v[188:191], v[112:115], v[236:239]\n\t"
        "ds_read_b64_tr_b16 v[156:157], v15 offset:49152\n\t"
        "global_load_dwordx4 v[60:63], v25, s[68:69] offset:1024\n\t"
        "v_mfma_f32_16x16x32_bf16 v[252:255], v[188:191], v[116:119], v[252:255]\n\t"
        "ds_read_b64_tr_b16 v[158:159], v15 offset:50176\n\t"
        "s_waitcnt vmcnt(6) lgkmcnt(15)\n\t"
        "v_mfma_f32_16x16x32_bf16 v[192:195], v[128:131], v[32:35], v[192:195]\n\t"
        "ds_read_b64_tr_b16 v[160:161], v8 offset:57344\n\t"
        "v_mfma_f32_16x16x32_bf16 v[208:211], v[128:131], v[36:39], v[208:211]\n\t"
        "ds_read_b64_tr_b16 v[162:163], v8 offset:58368\n\t"
        "v_mfma_f32_16x16x32_bf16 v[196:199], v[132:135], v[32:35], v[196:199]\n\t"
        "ds_read_b64_tr_b16 v[164:165], v9 offset:57344\n\t"
        "v_mfma_f32_16x16x32_bf16 v[212:215], v[132:135], v[36:39], v[212:215]\n\t"
        "ds_read_b64_tr_b16 v[166:167], v9 offset:58368\n\t"
        "s_waitcnt lgkmcnt(10)\n\t"
        "v_mfma_f32_16x16x32_bf16 v[200:203], v[136:139], v[32:35], v[200:203]\n\t"
        "ds_read_b64_tr_b16 v[168:169], v10 offset:57344\n\t"
        "s_movk_i32 s61, 24\n\t"
        "v_mfma_f32_16x16x32_bf16 v[216:219], v[136:139], v[36:39], v[216:219]\n\t"
        "ds_read_b64_tr_b16 v[170:171], v10 offset:58368\n\t"
        "v_cmp_ge_u32 vcc, s61, v121\n\t"
        "v_mfma_f32_16x16x32_bf16 v[204:207], v[140:143], v[32:35], v[204:207]\n\t"
        "ds_read_b64_tr_b16 v[172:173], v11 offset:57344\n\t"
        "s_getpc_b64 s[70:71]\n\t"
        "s_cbranch_vccnz .Lspin1_%=\n\t"
        "v_mfma_f32_16x16x32_bf16 v[220:223], v[140:143], v[36:39], v[220:223]\n\t"
        "ds_read_b64_tr_b16 v[174:175], v11 offset:58368\n\t"
        "s_movk_i32 s61, 26\n\t"
        "s_waitcnt lgkmcnt(13)\n\t"
        "v_mfma_f32_16x16x32_bf16 v[224:227], v[144:147], v[32:35], v[224:227]\n\t"
        "ds_read_b64_tr_b16 v[176:177], v12 offset:57344\n\t"
        "v_cmp_eq_u32 vcc, 0, v26\n\t"
        "v_mfma_f32_16x16x32_bf16 v[240:243], v[144:147], v[36:39], v[240:243]\n\t"
        "ds_read_b64_tr_b16 v[178:179], v12 offset:58368\n\t"
        "s_getpc_b64 s[70:71]\n\t"
        "s_cbranch_vccnz .Lspin6_%=\n\t"
        "v_mfma_f32_16x16x32_bf16 v[228:231], v[148:151], v[32:35], v[228:231]\n\t"
        "ds_read_b64_tr_b16 v[180:181], v13 offset:57344\n\t"
        "ds_read_b32 v121, v127 offset:4\n\t"
        "ds_read_b32 v27, v127 offset:364\n\t"
        "v_mfma_f32_16x16x32_bf16 v[244:247], v[148:151], v[36:39], v[244:247]\n\t"
        "ds_read_b64_tr_b16 v[182:183], v13 offset:58368\n\t"
        "v_add_u32 v26, v26, %2\n\t"
        "s_waitcnt lgkmcnt(14)\n\t"
        "v_mfma_f32_16x16x32_bf16 v[232:235], v[152:155], v[32:35], v[232:235]\n\t"
        "ds_read_b64_tr_b16 v[184:185], v14 offset:57344\n\t"
        "global_load_dwordx4 v[64:67], v26, s[68:69] offset:0\n\t"
        "v_mfma_f32_16x16x32_bf16 v[248:251], v[152:155], v[36:39], v[248:251]\n\t"
        "ds_read_b64_tr_b16 v[186:187], v14 offset:58368\n\t"
        "ds_add_u32 %4, v31\n\t"
        "v_mfma_f32_16x16x32_bf16 v[236:239], v[156:159], v[32:35], v[236:239]\n\t"
        "ds_read_b64_tr_b16 v[188:189], v15 offset:57344\n\t"
        "global_load_dwordx4 v[68:71], v26, s[68:69] offset:1024\n\t"
        "v_mfma_f32_16x16x32_bf16 v[252:255], v[156:159], v[36:39], v[252:255]\n\t"
        "ds_read_b64_tr_b16 v[190:191], v15 offset:58368\n\t"
        "s_waitcnt vmcnt(6) lgkmcnt(15)\n\t"
        "v_mfma_f32_16x16x32_bf16 v[192:195], v[160:163], v[40:43], v[192:195]\n\t"
        "ds_read_b64_tr_b16 v[128:129], v16 offset:0\n\t"
        "v_mfma_f32_16x16x32_bf16 v[208:211], v[160:163], v[44:47], v[208:211]\n\t"
        "ds_read_b64_tr_b16 v[130:131], v16 offset:1024\n\t"
        "v_mfma_f32_16x16x32_bf16 v[196:199], v[164:167], v[40:43], v[196:199]\n\t"
        "ds_read_b64_tr_b16 v[132:133], v17 offset:0\n\t"
        "v_mfma_f32_16x16x32_bf16 v[212:215], v[164:167], v[44:47], v[212:215]\n\t"
        "ds_read_b64_tr_b16 v[134:135], v17 offset:1024\n\t"
        "s_waitcnt lgkmcnt(10)\n\t"
        "v_mfma_f32_16x16x32_bf16 v[200:203], v[168:171], v[40:43], v[200:203]\n\t"
        "ds_read_b64_tr_b16 v[136:137], v18 offset:0\n\t"
        "s_movk_i32 s61, 25\n\t"
        "v_mfma_f32_16x16x32_bf16 v[216:219], v[168:171], v[44:47], v[216:219]\n\t"
        "ds_read_b64_tr_b16 v[138:139], v18 offset:1024\n\t"
        "v_cmp_ge_u32 vcc, s61, v121\n\t"
        "v_mfma_f32_16x16x32_bf16 v[204:207], v[172:175], v[40:43], v[204:207]\n\t"
        "ds_read_b64_tr_b16 v[140:141], v19 offset:0\n\t"
        "s_getpc_b64 s[70:71]\n\t"
        "s_cbranch_vccnz .Lspin1_%=\n\t"
        "v_mfma_f32_16x16x32_bf16 v[220:223], v[172:175], v[44:47], v[220:223]\n\t"
        "ds_read_b64_tr_b16 v[142:143], v19 offset:1024\n\t"
        "s_movk_i32 s61, 27\n\t"
        "s_waitcnt lgkmcnt(13)\n\t"
        "v_mfma_f32_16x16x32_bf16 v[224:227], v[176:179], v[40:43], v[224:227]\n\t"
        "ds_read_b64_tr_b16 v[144:145], v20 offset:0\n\t"
        "v_cmp_eq_u32 vcc, 0, v27\n\t"
        "v_mfma_f32_16x16x32_bf16 v[240:243], v[176:179], v[44:47], v[240:243]\n\t"
        "ds_read_b64_tr_b16 v[146:147], v20 offset:1024\n\t"
        "s_getpc_b64 s[70:71]\n\t"
        "s_cbranch_vccnz .Lspin7_%=\n\t"
        "v_mfma_f32_16x16x32_bf16 v[228:231], v[180:183], v[40:43], v[228:231]\n\t"
        "ds_read_b64_tr_b16 v[148:149], v21 offset:0\n\t"
        "ds_read_b32 v121, v127 offset:4\n\t"
        "ds_read_b32 v24, v127 offset:368\n\t"
        "v_mfma_f32_16x16x32_bf16 v[244:247], v[180:183], v[44:47], v[244:247]\n\t"
        "ds_read_b64_tr_b16 v[150:151], v21 offset:1024\n\t"
        "v_add_u32 v27, v27, %2\n\t"
        "s_waitcnt lgkmcnt(14)\n\t"
        "v_mfma_f32_16x16x32_bf16 v[232:235], v[184:187], v[40:43], v[232:235]\n\t"
        "ds_read_b64_tr_b16 v[152:153], v22 offset:0\n\t"
        "global_load_dwordx4 v[72:75], v27, s[68:69] offset:0\n\t"
        "v_mfma_f32_16x16x32_bf16 v[248:251], v[184:187], v[44:47], v[248:251]\n\t"
        "ds_read_b64_tr_b16 v[154:155], v22 offset:1024\n\t"
        "ds_add_u32 %4, v31\n\t"
        "v_mfma_f32_16x16x32_bf16 v[236:239], v[188:191], v[40:43], v[236:239]\n\t"
        "ds_read_b64_tr_b16 v[156:157], v23 offset:0\n\t"
        "global_load_dwordx4 v[76:79], v27, s[68:69] offset:1024\n\t"
        "v_mfma_f32_16x16x32_bf16 v[252:255], v[188:191], v[44:47], v[252:255]\n\t"
        "ds_read_b64_tr_b16 v[158:159], v23 offset:1024\n\t"
        "s_waitcnt vmcnt(6) lgkmcnt(15)\n\t"
        "v_mfma_f32_16x16x32_bf16 v[192:195], v[128:131], v[48:51], v[192:195]\n\t"
        "ds_read_b64_tr_b16 v[160:161], v16 offset:8192\n\t"
        "v_mfma_f32_16x16x32_bf16 v[208:211], v[128:131], v[52:55], v[208:211]\n\t"
        "ds_read_b64_tr_b16 v[162:163], v16 offset:9216\n\t"
        "v_mfma_f32_16x16x32_bf16 v[196:199], v[132:135], v[48:51], v[196:199]\n\t"
        "ds_read_b64_tr_b16 v[164:165], v17 offset:8192\n\t"
        "v_mfma_f32_16x16x32_bf16 v[212:215], v[132:135], v[52:55], v[212:215]\n\t"
        "ds_read_b64_tr_b16 v[166:167], v17 offset:9216\n\t"
        "s_waitcnt lgkmcnt(10)\n\t"
        "v_mfma_f32_16x16x32_bf16 v[200:203], v[136:139], v[48:51], v[200:203]\n\t"
        "ds_read_b64_tr_b16 v[168:169], v18 offset:8192\n\t"
        "s_movk_i32 s61, 26\n\t"
        "v_mfma_f32_16x16x32_bf16 v[216:219], v[136:139], v[52:55], v[216:219]\n\t"
        "ds_read_b64_tr_b16 v[170:171], v18 offset:9216\n\t"
        "v_cmp_ge_u32 vcc, s61, v121\n\t"
        "v_mfma_f32_16x16x32_bf16 v[204:207], v[140:143], v[48:51], v[204:207]\n\t"
        "ds_read_b64_tr_b16 v[172:173], v19 offset:8192\n\t"
        "s_getpc_b64 s[70:71]\n\t"
        "s_cbranch_vccnz .Lspin1_%=\n\t"
        "v_mfma_f32_16x16x32_bf16 v[220:223], v[140:143], v[52:55], v[220:223]\n\t"
        "ds_read_b64_tr_b16 v[174:175], v19 offset:9216\n\t"
        "s_movk_i32 s61, 28\n\t"
        "s_waitcnt lgkmcnt(13)\n\t"
        "v_mfma_f32_16x16x32_bf16 v[224:227], v[144:147], v[48:51], v[224:227]\n\t"
        "ds_read_b64_tr_b16 v[176:177], v20 offset:8192\n\t"
        "v_cmp_eq_u32 vcc, 0, v24\n\t"
        "v_mfma_f32_16x16x32_bf16 v[240:243], v[144:147], v[52:55], v[240:243]\n\t"
        "ds_read_b64_tr_b16 v[178:179], v20 offset:9216\n\t"
        "s_getpc_b64 s[70:71]\n\t"
        "s_cbranch_vccnz .Lspin4_%=\n\t"
        "v_mfma_f32_16x16x32_bf16 v[228:231], v[148:151], v[48:51], v[228:231]\n\t"
        "ds_read_b64_tr_b16 v[180:181], v21 offset:8192\n\t"
        "ds_read_b32 v121, v127 offset:4\n\t"
        "ds_read_b32 v25, v127 offset:372\n\t"
        "v_mfma_f32_16x16x32_bf16 v[244:247], v[148:151], v[52:55], v[244:247]\n\t"
        "ds_read_b64_tr_b16 v[182:183], v21 offset:9216\n\t"
        "v_add_u32 v24, v24, %2\n\t"
        "s_waitcnt lgkmcnt(14)\n\t"
        "v_mfma_f32_16x16x32_bf16 v[232:235], v[152:155], v[48:51], v[232:235]\n\t"
        "ds_read_b64_tr_b16 v[184:185], v22 offset:8192\n\t"
        "global_load_dwordx4 v[80:83], v24, s[68:69] offset:0\n\t"
        "v_mfma_f32_16x16x32_bf16 v[248:251], v[152:155], v[52:55], v[248:251]\n\t"
        "ds_read_b64_tr_b16 v[186:187], v22 offset:9216\n\t"
        "ds_add_u32 %4, v31\n\t"
        "v_mfma_f32_16x16x32_bf16 v[236:239], v[156:159], v[48:51], v[236:239]\n\t"
        "ds_read_b64_tr_b16 v[188:189], v23 offset:8192\n\t"
        "global_load_dwordx4 v[84:87], v24, s[68:69] offset:1024\n\t"
        "v_mfma_f32_16x16x32_bf16 v[252:255], v[156:159], v[52:55], v[252:255]\n\t"
        "ds_read_b64_tr_b16 v[190:191], v23 offset:9216\n\t"
        "s_waitcnt vmcnt(6) lgkmcnt(15)\n\t"
        "v_mfma_f32_16x16x32_bf16 v[192:195], v[160:163], v[56:59], v[192:195]\n\t"
        "ds_read_b64_tr_b16 v[128:129], v16 offset:16384\n\t"
        "v_mfma_f32_16x16x32_bf16 v[208:211], v[160:163], v[60:63], v[208:211]\n\t"
        "ds_read_b64_tr_b16 v[130:131], v16 offset:17408\n\t"
        "v_mfma_f32_16x16x32_bf16 v[196:199], v[164:167], v[56:59], v[196:199]\n\t"
        "ds_read_b64_tr_b16 v[132:133], v17 offset:16384\n\t"
        "v_mfma_f32_16x16x32_bf16 v[212:215], v[164:167], v[60:63], v[212:215]\n\t"
        "ds_read_b64_tr_b16 v[134:135], v17 offset:17408\n\t"
        "s_waitcnt lgkmcnt(10)\n\t"
        "v_mfma_f32_16x16x32_bf16 v[200:203], v[168:171], v[56:59], v[200:203]\n\t"
        "ds_read_b64_tr_b16 v[136:137], v18 offset:16384\n\t"
        "s_movk_i32 s61, 27\n\t"
        "v_mfma_f32_16x16x32_bf16 v[216:219], v[168:171], v[60:63], v[216:219]\n\t"
        "ds_read_b64_tr_b16 v[138:139], v18 offset:17408\n\t"
        "v_cmp_ge_u32 vcc, s61, v121\n\t"
        "v_mfma_f32_16x16x32_bf16 v[204:207], v[172:175], v[56:59], v[204:207]\n\t"
        "ds_read_b64_tr_b16 v[140:141], v19 offset:16384\n\t"
        "s_getpc_b64 s[70:71]\n\t"
        "s_cbranch_vccnz .Lspin1_%=\n\t"
        "v_mfma_f32_16x16x32_bf16 v[220:223], v[172:175], v[60:63], v[220:223]\n\t"
        "ds_read_b64_tr_b16 v[142:143], v19 offset:17408\n\t"
        "s_movk_i32 s61, 29\n\t"
        "s_waitcnt lgkmcnt(13)\n\t"
        "v_mfma_f32_16x16x32_bf16 v[224:227], v[176:179], v[56:59], v[224:227]\n\t"
        "ds_read_b64_tr_b16 v[144:145], v20 offset:16384\n\t"
        "v_cmp_eq_u32 vcc, 0, v25\n\t"
        "v_mfma_f32_16x16x32_bf16 v[240:243], v[176:179], v[60:63], v[240:243]\n\t"
        "ds_read_b64_tr_b16 v[146:147], v20 offset:17408\n\t"
        "s_getpc_b64 s[70:71]\n\t"
        "s_cbranch_vccnz .Lspin5_%=\n\t"
        "v_mfma_f32_16x16x32_bf16 v[228:231], v[180:183], v[56:59], v[228:231]\n\t"
        "ds_read_b64_tr_b16 v[148:149], v21 offset:16384\n\t"
        "ds_read_b32 v121, v127 offset:4\n\t"
        "ds_read_b32 v26, v127 offset:376\n\t"
        "v_mfma_f32_16x16x32_bf16 v[244:247], v[180:183], v[60:63], v[244:247]\n\t"
        "ds_read_b64_tr_b16 v[150:151], v21 offset:17408\n\t"
        "v_add_u32 v25, v25, %2\n\t"
        "s_waitcnt lgkmcnt(14)\n\t"
        "v_mfma_f32_16x16x32_bf16 v[232:235], v[184:187], v[56:59], v[232:235]\n\t"
        "ds_read_b64_tr_b16 v[152:153], v22 offset:16384\n\t"
        "global_load_dwordx4 v[88:91], v25, s[68:69] offset:0\n\t"
        "v_mfma_f32_16x16x32_bf16 v[248:251], v[184:187], v[60:63], v[248:251]\n\t"
        "ds_read_b64_tr_b16 v[154:155], v22 offset:17408\n\t"
        "ds_add_u32 %4, v31\n\t"
        "v_mfma_f32_16x16x32_bf16 v[236:239], v[188:191], v[56:59], v[236:239]\n\t"
        "ds_read_b64_tr_b16 v[156:157], v23 offset:16384\n\t"
        "global_load_dwordx4 v[92:95], v25, s[68:69] offset:1024\n\t"
        "v_mfma_f32_16x16x32_bf16 v[252:255], v[188:191], v[60:63], v[252:255]\n\t"
        "ds_read_b64_tr_b16 v[158:159], v23 offset:17408\n\t"
        "s_waitcnt vmcnt(6) lgkmcnt(15)\n\t"
        "v_mfma_f32_16x16x32_bf16 v[192:195], v[128:131], v[64:67], v[192:195]\n\t"
        "ds_read_b64_tr_b16 v[160:161], v16 offset:24576\n\t"
        "v_mfma_f32_16x16x32_bf16 v[208:211], v[128:131], v[68:71], v[208:211]\n\t"
        "ds_read_b64_tr_b16 v[162:163], v16 offset:25600\n\t"
        "v_mfma_f32_16x16x32_bf16 v[196:199], v[132:135], v[64:67], v[196:199]\n\t"
        "ds_read_b64_tr_b16 v[164:165], v17 offset:24576\n\t"
        "v_mfma_f32_16x16x32_bf16 v[212:215], v[132:135], v[68:71], v[212:215]\n\t"
        "ds_read_b64_tr_b16 v[166:167], v17 offset:25600\n\t"
        "s_waitcnt lgkmcnt(10)\n\t"
        "v_mfma_f32_16x16x32_bf16 v[200:203], v[136:139], v[64:67], v[200:203]\n\t"
        "ds_read_b64_tr_b16 v[168:169], v18 offset:24576\n\t"
        "s_movk_i32 s61, 28\n\t"
        "v_mfma_f32_16x16x32_bf16 v[216:219], v[136:139], v[68:71], v[216:219]\n\t"
        "ds_read_b64_tr_b16 v[170:171], v18 offset:25600\n\t"
        "v_cmp_ge_u32 vcc, s61, v121\n\t"
        "v_mfma_f32_16x16x32_bf16 v[204:207], v[140:143], v[64:67], v[204:207]\n\t"
        "ds_read_b64_tr_b16 v[172:173], v19 offset:24576\n\t"
        "s_getpc_b64 s[70:71]\n\t"
        "s_cbranch_vccnz .Lspin1_%=\n\t"
        "v_mfma_f32_16x16x32_bf16 v[220:223], v[140:143], v[68:71], v[220:223]\n\t"
        "ds_read_b64_tr_b16 v[174:175], v19 offset:25600\n\t"
        "s_movk_i32 s61, 30\n\t"
        "s_waitcnt lgkmcnt(13)\n\t"
        "v_mfma_f32_16x16x32_bf16 v[224:227], v[144:147], v[64:67], v[224:227]\n\t"
        "ds_read_b64_tr_b16 v[176:177], v20 offset:24576\n\t"
        "v_cmp_eq_u32 vcc, 0, v26\n\t"
        "v_mfma_f32_16x16x32_bf16 v[240:243], v[144:147], v[68:71], v[240:243]\n\t"
        "ds_read_b64_tr_b16 v[178:179], v20 offset:25600\n\t"
        "s_getpc_b64 s[70:71]\n\t"
        "s_cbranch_vccnz .Lspin6_%=\n\t"
        "v_mfma_f32_16x16x32_bf16 v[228:231], v[148:151], v[64:67], v[228:231]\n\t"
        "ds_read_b64_tr_b16 v[180:181], v21 offset:24576\n\t"
        "ds_read_b32 v121, v127 offset:4\n\t"
        "ds_read_b32 v27, v127 offset:380\n\t"
        "v_mfma_f32_16x16x32_bf16 v[244:247], v[148:151], v[68:71], v[244:247]\n\t"
        "ds_read_b64_tr_b16 v[182:183], v21 offset:25600\n\t"
        "v_add_u32 v26, v26, %2\n\t"
        "s_waitcnt lgkmcnt(14)\n\t"
        "v_mfma_f32_16x16x32_bf16 v[232:235], v[152:155], v[64:67], v[232:235]\n\t"
        "ds_read_b64_tr_b16 v[184:185], v22 offset:24576\n\t"
        "global_load_dwordx4 v[96:99], v26, s[68:69] offset:0\n\t"
        "v_mfma_f32_16x16x32_bf16 v[248:251], v[152:155], v[68:71], v[248:251]\n\t"
        "ds_read_b64_tr_b16 v[186:187], v22 offset:25600\n\t"
        "ds_add_u32 %4, v31\n\t"
        "v_mfma_f32_16x16x32_bf16 v[236:239], v[156:159], v[64:67], v[236:239]\n\t"
        "ds_read_b64_tr_b16 v[188:189], v23 offset:24576\n\t"
        "global_load_dwordx4 v[100:103], v26, s[68:69] offset:1024\n\t"
        "v_mfma_f32_16x16x32_bf16 v[252:255], v[156:159], v[68:71], v[252:255]\n\t"
        "ds_read_b64_tr_b16 v[190:191], v23 offset:25600\n\t"
        "s_waitcnt vmcnt(6) lgkmcnt(15)\n\t"
        "v_mfma_f32_16x16x32_bf16 v[192:195], v[160:163], v[72:75], v[192:195]\n\t"
        "ds_read_b64_tr_b16 v[128:129], v16 offset:32768\n\t"
        "v_mfma_f32_16x16x32_bf16 v[208:211], v[160:163], v[76:79], v[208:211]\n\t"
        "ds_read_b64_tr_b16 v[130:131], v16 offset:33792\n\t"
        "v_mfma_f32_16x16x32_bf16 v[196:199], v[164:167], v[72:75], v[196:199]\n\t"
        "ds_read_b64_tr_b16 v[132:133], v17 offset:32768\n\t"
        "v_mfma_f32_16x16x32_bf16 v[212:215], v[164:167], v[76:79], v[212:215]\n\t"
        "ds_read_b64_tr_b16 v[134:135], v17 offset:33792\n\t"
        "s_waitcnt lgkmcnt(10)\n\t"
        "v_mfma_f32_16x16x32_bf16 v[200:203], v[168:171], v[72:75], v[200:203]\n\t"
        "ds_read_b64_tr_b16 v[136:137], v18 offset:32768\n\t"
        "s_movk_i32 s61, 29\n\t"
        "v_mfma_f32_16x16x32_bf16 v[216:219], v[168:171], v[76:79], v[216:219]\n\t"
        "ds_read_b64_tr_b16 v[138:139], v18 offset:33792\n\t"
        "v_cmp_ge_u32 vcc, s61, v121\n\t"
        "v_mfma_f32_16x16x32_bf16 v[204:207], v[172:175], v[72:75], v[204:207]\n\t"
        "ds_read_b64_tr_b16 v[140:141], v19 offset:32768\n\t"
        "s_getpc_b64 s[70:71]\n\t"
        "s_cbranch_vccnz .Lspin1_%=\n\t"
        "v_mfma_f32_16x16x32_bf16 v[220:223], v[172:175], v[76:79], v[220:223]\n\t"
        "ds_read_b64_tr_b16 v[142:143], v19 offset:33792\n\t"
        "s_movk_i32 s61, 31\n\t"
        "s_waitcnt lgkmcnt(13)\n\t"
        "v_mfma_f32_16x16x32_bf16 v[224:227], v[176:179], v[72:75], v[224:227]\n\t"
        "ds_read_b64_tr_b16 v[144:145], v20 offset:32768\n\t"
        "v_cmp_eq_u32 vcc, 0, v27\n\t"
        "v_mfma_f32_16x16x32_bf16 v[240:243], v[176:179], v[76:79], v[240:243]\n\t"
        "ds_read_b64_tr_b16 v[146:147], v20 offset:33792\n\t"
        "s_getpc_b64 s[70:71]\n\t"
        "s_cbranch_vccnz .Lspin7_%=\n\t"
        "v_mfma_f32_16x16x32_bf16 v[228:231], v[180:183], v[72:75], v[228:231]\n\t"
        "ds_read_b64_tr_b16 v[148:149], v21 offset:32768\n\t"
        "ds_read_b32 v121, v127 offset:4\n\t"
        "ds_read_b32 v24, v127 offset:384\n\t"
        "v_mfma_f32_16x16x32_bf16 v[244:247], v[180:183], v[76:79], v[244:247]\n\t"
        "ds_read_b64_tr_b16 v[150:151], v21 offset:33792\n\t"
        "v_add_u32 v27, v27, %2\n\t"
        "s_waitcnt lgkmcnt(14)\n\t"
        "v_mfma_f32_16x16x32_bf16 v[232:235], v[184:187], v[72:75], v[232:235]\n\t"
        "ds_read_b64_tr_b16 v[152:153], v22 offset:32768\n\t"
        "global_load_dwordx4 v[104:107], v27, s[68:69] offset:0\n\t"
        "v_mfma_f32_16x16x32_bf16 v[248:251], v[184:187], v[76:79], v[248:251]\n\t"
        "ds_read_b64_tr_b16 v[154:155], v22 offset:33792\n\t"
        "ds_add_u32 %4, v31\n\t"
        "v_mfma_f32_16x16x32_bf16 v[236:239], v[188:191], v[72:75], v[236:239]\n\t"
        "ds_read_b64_tr_b16 v[156:157], v23 offset:32768\n\t"
        "global_load_dwordx4 v[108:111], v27, s[68:69] offset:1024\n\t"
        "v_mfma_f32_16x16x32_bf16 v[252:255], v[188:191], v[76:79], v[252:255]\n\t"
        "ds_read_b64_tr_b16 v[158:159], v23 offset:33792\n\t"
        "s_waitcnt vmcnt(6) lgkmcnt(15)\n\t"
        "v_mfma_f32_16x16x32_bf16 v[192:195], v[128:131], v[80:83], v[192:195]\n\t"
        "ds_read_b64_tr_b16 v[160:161], v16 offset:40960\n\t"
        "v_mfma_f32_16x16x32_bf16 v[208:211], v[128:131], v[84:87], v[208:211]\n\t"
        "ds_read_b64_tr_b16 v[162:163], v16 offset:41984\n\t"
        "v_mfma_f32_16x16x32_bf16 v[196:199], v[132:135], v[80:83], v[196:199]\n\t"
        "ds_read_b64_tr_b16 v[164:165], v17 offset:40960\n\t"
        "v_mfma_f32_16x16x32_bf16 v[212:215], v[132:135], v[84:87], v[212:215]\n\t"
        "ds_read_b64_tr_b16 v[166:167], v17 offset:41984\n\t"
        "s_waitcnt lgkmcnt(10)\n\t"
        "v_mfma_f32_16x16x32_bf16 v[200:203], v[136:139], v[80:83], v[200:203]\n\t"
        "ds_read_b64_tr_b16 v[168:169], v18 offset:40960\n\t"
        "s_movk_i32 s61, 30\n\t"
        "v_mfma_f32_16x16x32_bf16 v[216:219], v[136:139], v[84:87], v[216:219]\n\t"
        "ds_read_b64_tr_b16 v[170:171], v18 offset:41984\n\t"
        "v_cmp_ge_u32 vcc, s61, v121\n\t"
        "v_mfma_f32_16x16x32_bf16 v[204:207], v[140:143], v[80:83], v[204:207]\n\t"
        "ds_read_b64_tr_b16 v[172:173], v19 offset:40960\n\t"
        "s_getpc_b64 s[70:71]\n\t"
        "s_cbranch_vccnz .Lspin1_%=\n\t"
        "v_mfma_f32_16x16x32_bf16 v[220:223], v[140:143], v[84:87], v[220:223]\n\t"
        "ds_read_b64_tr_b16 v[174:175], v19 offset:41984\n\t"
        "s_movk_i32 s61, 32\n\t"
        "s_waitcnt lgkmcnt(13)\n\t"
        "v_mfma_f32_16x16x32_bf16 v[224:227], v[144:147], v[80:83], v[224:227]\n\t"
        "ds_read_b64_tr_b16 v[176:177], v20 offset:40960\n\t"
        "v_cmp_eq_u32 vcc, 0, v24\n\t"
        "v_mfma_f32_16x16x32_bf16 v[240:243], v[144:147], v[84:87], v[240:243]\n\t"
        "ds_read_b64_tr_b16 v[178:179], v20 offset:41984\n\t"
        "s_getpc_b64 s[70:71]\n\t"
        "s_cbranch_vccnz .Lspin4_%=\n\t"
        "v_mfma_f32_16x16x32_bf16 v[228:231], v[148:151], v[80:83], v[228:231]\n\t"
        "ds_read_b64_tr_b16 v[180:181], v21 offset:40960\n\t"
        "ds_read_b32 v121, v127 offset:4\n\t"
        "ds_read_b32 v25, v127 offset:388\n\t"
        "v_mfma_f32_16x16x32_bf16 v[244:247], v[148:151], v[84:87], v[244:247]\n\t"
        "ds_read_b64_tr_b16 v[182:183], v21 offset:41984\n\t"
        "v_add_u32 v24, v24, %2\n\t"
        "s_waitcnt lgkmcnt(14)\n\t"
        "v_mfma_f32_16x16x32_bf16 v[232:235], v[152:155], v[80:83], v[232:235]\n\t"
        "ds_read_b64_tr_b16 v[184:185], v22 offset:40960\n\t"
        "global_load_dwordx4 v[112:115], v24, s[68:69] offset:0\n\t"
        "v_mfma_f32_16x16x32_bf16 v[248:251], v[152:155], v[84:87], v[248:251]\n\t"
        "ds_read_b64_tr_b16 v[186:187], v22 offset:41984\n\t"
        "ds_add_u32 %4, v31\n\t"
        "v_mfma_f32_16x16x32_bf16 v[236:239], v[156:159], v[80:83], v[236:239]\n\t"
        "ds_read_b64_tr_b16 v[188:189], v23 offset:40960\n\t"
        "global_load_dwordx4 v[116:119], v24, s[68:69] offset:1024\n\t"
        "v_mfma_f32_16x16x32_bf16 v[252:255], v[156:159], v[84:87], v[252:255]\n\t"
        "ds_read_b64_tr_b16 v[190:191], v23 offset:41984\n\t"
        "s_waitcnt vmcnt(6) lgkmcnt(15)\n\t"
        "v_mfma_f32_16x16x32_bf16 v[192:195], v[160:163], v[88:91], v[192:195]\n\t"
        "ds_read_b64_tr_b16 v[128:129], v16 offset:49152\n\t"
        "v_mfma_f32_16x16x32_bf16 v[208:211], v[160:163], v[92:95], v[208:211]\n\t"
        "ds_read_b64_tr_b16 v[130:131], v16 offset:50176\n\t"
        "v_mfma_f32_16x16x32_bf16 v[196:199], v[164:167], v[88:91], v[196:199]\n\t"
        "ds_read_b64_tr_b16 v[132:133], v17 offset:49152\n\t"
        "v_mfma_f32_16x16x32_bf16 v[212:215], v[164:167], v[92:95], v[212:215]\n\t"
        "ds_read_b64_tr_b16 v[134:135], v17 offset:50176\n\t"
        "s_waitcnt lgkmcnt(10)\n\t"
        "v_mfma_f32_16x16x32_bf16 v[200:203], v[168:171], v[88:91], v[200:203]\n\t"
        "ds_read_b64_tr_b16 v[136:137], v18 offset:49152\n\t"
        "s_movk_i32 s61, 31\n\t"
        "v_mfma_f32_16x16x32_bf16 v[216:219], v[168:171], v[92:95], v[216:219]\n\t"
        "ds_read_b64_tr_b16 v[138:139], v18 offset:50176\n\t"
        "v_cmp_ge_u32 vcc, s61, v121\n\t"
        "v_mfma_f32_16x16x32_bf16 v[204:207], v[172:175], v[88:91], v[204:207]\n\t"
        "ds_read_b64_tr_b16 v[140:141], v19 offset:49152\n\t"
        "s_getpc_b64 s[70:71]\n\t"
        "s_cbranch_vccnz .Lspin1_%=\n\t"
        "v_mfma_f32_16x16x32_bf16 v[220:223], v[172:175], v[92:95], v[220:223]\n\t"
        "ds_read_b64_tr_b16 v[142:143], v19 offset:50176\n\t"
        "s_movk_i32 s61, 33\n\t"
        "s_waitcnt lgkmcnt(13)\n\t"
        "v_mfma_f32_16x16x32_bf16 v[224:227], v[176:179], v[88:91], v[224:227]\n\t"
        "ds_read_b64_tr_b16 v[144:145], v20 offset:49152\n\t"
        "v_cmp_eq_u32 vcc, 0, v25\n\t"
        "v_mfma_f32_16x16x32_bf16 v[240:243], v[176:179], v[92:95], v[240:243]\n\t"
        "ds_read_b64_tr_b16 v[146:147], v20 offset:50176\n\t"
        "s_getpc_b64 s[70:71]\n\t"
        "s_cbranch_vccnz .Lspin5_%=\n\t"
        "v_mfma_f32_16x16x32_bf16 v[228:231], v[180:183], v[88:91], v[228:231]\n\t"
        "ds_read_b64_tr_b16 v[148:149], v21 offset:49152\n\t"
        "ds_read_b32 v121, v127 offset:4\n\t"
        "ds_read_b32 v26, v127 offset:392\n\t"
        "v_mfma_f32_16x16x32_bf16 v[244:247], v[180:183], v[92:95], v[244:247]\n\t"
        "ds_read_b64_tr_b16 v[150:151], v21 offset:50176\n\t"
        "v_add_u32 v25, v25, %2\n\t"
        "s_waitcnt lgkmcnt(14)\n\t"
        "v_mfma_f32_16x16x32_bf16 v[232:235], v[184:187], v[88:91], v[232:235]\n\t"
        "ds_read_b64_tr_b16 v[152:153], v22 offset:49152\n\t"
        "global_load_dwordx4 v[32:35], v25, s[68:69] offset:0\n\t"
        "v_mfma_f32_16x16x32_bf16 v[248:251], v[184:187], v[92:95], v[248:251]\n\t"
        "ds_read_b64_tr_b16 v[154:155], v22 offset:50176\n\t"
        "ds_add_u32 %4, v31\n\t"
        "v_mfma_f32_16x16x32_bf16 v[236:239], v[188:191], v[88:91], v[236:239]\n\t"
        "ds_read_b64_tr_b16 v[156:157], v23 offset:49152\n\t"
        "global_load_dwordx4 v[36:39], v25, s[68:69] offset:1024\n\t"
        "v_mfma_f32_16x16x32_bf16 v[252:255], v[188:191], v[92:95], v[252:255]\n\t"
        "ds_read_b64_tr_b16 v[158:159], v23 offset:50176\n\t"
        "s_waitcnt vmcnt(6) lgkmcnt(15)\n\t"
        "v_mfma_f32_16x16x32_bf16 v[192:195], v[128:131], v[96:99], v[192:195]\n\t"
        "ds_read_b64_tr_b16 v[160:161], v16 offset:57344\n\t"
        "v_mfma_f32_16x16x32_bf16 v[208:211], v[128:131], v[100:103], v[208:211]\n\t"
        "ds_read_b64_tr_b16 v[162:163], v16 offset:58368\n\t"
        "v_mfma_f32_16x16x32_bf16 v[196:199], v[132:135], v[96:99], v[196:199]\n\t"
        "ds_read_b64_tr_b16 v[164:165], v17 offset:57344\n\t"
        "v_mfma_f32_16x16x32_bf16 v[212:215], v[132:135], v[100:103], v[212:215]\n\t"
        "ds_read_b64_tr_b16 v[166:167], v17 offset:58368\n\t"
        "s_waitcnt lgkmcnt(10)\n\t"
        "v_mfma_f32_16x16x32_bf16 v[200:203], v[136:139], v[96:99], v[200:203]\n\t"
        "ds_read_b64_tr_b16 v[168:169], v18 offset:57344\n\t"
        "s_movk_i32 s61, 32\n\t"
        "v_mfma_f32_16x16x32_bf16 v[216:219], v[136:139], v[100:103], v[216:219]\n\t"
        "ds_read_b64_tr_b16 v[170:171], v18 offset:58368\n\t"
        "v_cmp_ge_u32 vcc, s61, v121\n\t"
        "v_mfma_f32_16x16x32_bf16 v[204:207], v[140:143], v[96:99], v[204:207]\n\t"
        "ds_read_b64_tr_b16 v[172:173], v19 offset:57344\n\t"
        "s_getpc_b64 s[70:71]\n\t"
        "s_cbranch_vccnz .Lspin1_%=\n\t"
        "v_mfma_f32_16x16x32_bf16 v[220:223], v[140:143], v[100:103], v[220:223]\n\t"
        "ds_read_b64_tr_b16 v[174:175], v19 offset:58368\n\t"
        "s_movk_i32 s61, 34\n\t"
        "s_waitcnt lgkmcnt(13)\n\t"
        "v_mfma_f32_16x16x32_bf16 v[224:227], v[144:147], v[96:99], v[224:227]\n\t"
        "ds_read_b64_tr_b16 v[176:177], v20 offset:57344\n\t"
        "v_cmp_eq_u32 vcc, 0, v26\n\t"
        "v_mfma_f32_16x16x32_bf16 v[240:243], v[144:147], v[100:103], v[240:243]\n\t"
        "ds_read_b64_tr_b16 v[178:179], v20 offset:58368\n\t"
        "s_getpc_b64 s[70:71]\n\t"
        "s_cbranch_vccnz .Lspin6_%=\n\t"
        "v_mfma_f32_16x16x32_bf16 v[228:231], v[148:151], v[96:99], v[228:231]\n\t"
        "ds_read_b64_tr_b16 v[180:181], v21 offset:57344\n\t"
        "ds_read_b32 v121, v127 offset:4\n\t"
        "ds_read_b32 v27, v127 offset:396\n\t"
        "v_mfma_f32_16x16x32_bf16 v[244:247], v[148:151], v[100:103], v[244:247]\n\t"
        "ds_read_b64_tr_b16 v[182:183], v21 offset:58368\n\t"
        "v_add_u32 v26, v26, %2\n\t"
        "s_waitcnt lgkmcnt(14)\n\t"
        "v_mfma_f32_16x16x32_bf16 v[232:235], v[152:155], v[96:99], v[232:235]\n\t"
        "ds_read_b64_tr_b16 v[184:185], v22 offset:57344\n\t"
        "global_load_dwordx4 v[40:43], v26, s[68:69] offset:0\n\t"
        "v_mfma_f32_16x16x32_bf16 v[248:251], v[152:155], v[100:103], v[248:251]\n\t"
        "ds_read_b64_tr_b16 v[186:187], v22 offset:58368\n\t"
        "ds_add_u32 %4, v31\n\t"
        "v_mfma_f32_16x16x32_bf16 v[236:239], v[156:159], v[96:99], v[236:239]\n\t"
        "ds_read_b64_tr_b16 v[188:189], v23 offset:57344\n\t"
        "global_load_dwordx4 v[44:47], v26, s[68:69] offset:1024\n\t"
        "v_mfma_f32_16x16x32_bf16 v[252:255], v[156:159], v[100:103], v[252:255]\n\t"
        "ds_read_b64_tr_b16 v[190:191], v23 offset:58368\n\t"
        "s_waitcnt vmcnt(6) lgkmcnt(15)\n\t"
        "v_mfma_f32_16x16x32_bf16 v[192:195], v[160:163], v[104:107], v[192:195]\n\t"
        "ds_read_b64_tr_b16 v[128:129], v8 offset:0\n\t"
        "v_mfma_f32_16x16x32_bf16 v[208:211], v[160:163], v[108:111], v[208:211]\n\t"
        "ds_read_b64_tr_b16 v[130:131], v8 offset:1024\n\t"
        "v_mfma_f32_16x16x32_bf16 v[196:199], v[164:167], v[104:107], v[196:199]\n\t"
        "ds_read_b64_tr_b16 v[132:133], v9 offset:0\n\t"
        "v_mfma_f32_16x16x32_bf16 v[212:215], v[164:167], v[108:111], v[212:215]\n\t"
        "ds_read_b64_tr_b16 v[134:135], v9 offset:1024\n\t"
        "s_waitcnt lgkmcnt(10)\n\t"
        "v_mfma_f32_16x16x32_bf16 v[200:203], v[168:171], v[104:107], v[200:203]\n\t"
        "ds_read_b64_tr_b16 v[136:137], v10 offset:0\n\t"
        "s_movk_i32 s61, 33\n\t"
        "v_mfma_f32_16x16x32_bf16 v[216:219], v[168:171], v[108:111], v[216:219]\n\t"
        "ds_read_b64_tr_b16 v[138:139], v10 offset:1024\n\t"
        "v_cmp_ge_u32 vcc, s61, v121\n\t"
        "v_mfma_f32_16x16x32_bf16 v[204:207], v[172:175], v[104:107], v[204:207]\n\t"
        "ds_read_b64_tr_b16 v[140:141], v11 offset:0\n\t"
        "s_getpc_b64 s[70:71]\n\t"
        "s_cbranch_vccnz .Lspin1_%=\n\t"
        "v_mfma_f32_16x16x32_bf16 v[220:223], v[172:175], v[108:111], v[220:223]\n\t"
        "ds_read_b64_tr_b16 v[142:143], v11 offset:1024\n\t"
        "s_movk_i32 s61, 35\n\t"
        "s_waitcnt lgkmcnt(13)\n\t"
        "v_mfma_f32_16x16x32_bf16 v[224:227], v[176:179], v[104:107], v[224:227]\n\t"
        "ds_read_b64_tr_b16 v[144:145], v12 offset:0\n\t"
        "v_cmp_eq_u32 vcc, 0, v27\n\t"
        "v_mfma_f32_16x16x32_bf16 v[240:243], v[176:179], v[108:111], v[240:243]\n\t"
        "ds_read_b64_tr_b16 v[146:147], v12 offset:1024\n\t"
        "s_getpc_b64 s[70:71]\n\t"
        "s_cbranch_vccnz .Lspin7_%=\n\t"
        "v_mfma_f32_16x16x32_bf16 v[228:231], v[180:183], v[104:107], v[228:231]\n\t"
        "ds_read_b64_tr_b16 v[148:149], v13 offset:0\n\t"
        "ds_read_b32 v121, v127 offset:4\n\t"
        "ds_read_b32 v24, v127 offset:400\n\t"
        "v_mfma_f32_16x16x32_bf16 v[244:247], v[180:183], v[108:111], v[244:247]\n\t"
        "ds_read_b64_tr_b16 v[150:151], v13 offset:1024\n\t"
        "v_add_u32 v27, v27, %2\n\t"
        "s_waitcnt lgkmcnt(14)\n\t"
        "v_mfma_f32_16x16x32_bf16 v[232:235], v[184:187], v[104:107], v[232:235]\n\t"
        "ds_read_b64_tr_b16 v[152:153], v14 offset:0\n\t"
        "global_load_dwordx4 v[48:51], v27, s[68:69] offset:0\n\t"
        "v_mfma_f32_16x16x32_bf16 v[248:251], v[184:187], v[108:111], v[248:251]\n\t"
        "ds_read_b64_tr_b16 v[154:155], v14 offset:1024\n\t"
        "ds_add_u32 %4, v31\n\t"
        "v_mfma_f32_16x16x32_bf16 v[236:239], v[188:191], v[104:107], v[236:239]\n\t"
        "ds_read_b64_tr_b16 v[156:157], v15 offset:0\n\t"
        "global_load_dwordx4 v[52:55], v27, s[68:69] offset:1024\n\t"
        "v_mfma_f32_16x16x32_bf16 v[252:255], v[188:191], v[108:111], v[252:255]\n\t"
        "ds_read_b64_tr_b16 v[158:159], v15 offset:1024\n\t"
        "s_waitcnt vmcnt(6) lgkmcnt(15)\n\t"
        "v_mfma_f32_16x16x32_bf16 v[192:195], v[128:131], v[112:115], v[192:195]\n\t"
        "ds_read_b64_tr_b16 v[160:161], v8 offset:8192\n\t"
        "v_mfma_f32_16x16x32_bf16 v[208:211], v[128:131], v[116:119], v[208:211]\n\t"
        "ds_read_b64_tr_b16 v[162:163], v8 offset:9216\n\t"
        "v_mfma_f32_16x16x32_bf16 v[196:199], v[132:135], v[112:115], v[196:199]\n\t"
        "ds_read_b64_tr_b16 v[164:165], v9 offset:8192\n\t"
        "v_mfma_f32_16x16x32_bf16 v[212:215], v[132:135], v[116:119], v[212:215]\n\t"
        "ds_read_b64_tr_b16 v[166:167], v9 offset:9216\n\t"
        "s_waitcnt lgkmcnt(10)\n\t"
        "v_mfma_f32_16x16x32_bf16 v[200:203], v[136:139], v[112:115], v[200:203]\n\t"
        "ds_read_b64_tr_b16 v[168:169], v10 offset:8192\n\t"
        "s_movk_i32 s61, 34\n\t"
        "v_mfma_f32_16x16x32_bf16 v[216:219], v[136:139], v[116:119], v[216:219]\n\t"
        "ds_read_b64_tr_b16 v[170:171], v10 offset:9216\n\t"
        "v_cmp_ge_u32 vcc, s61, v121\n\t"
        "v_mfma_f32_16x16x32_bf16 v[204:207], v[140:143], v[112:115], v[204:207]\n\t"
        "ds_read_b64_tr_b16 v[172:173], v11 offset:8192\n\t"
        "s_getpc_b64 s[70:71]\n\t"
        "s_cbranch_vccnz .Lspin1_%=\n\t"
        "v_mfma_f32_16x16x32_bf16 v[220:223], v[140:143], v[116:119], v[220:223]\n\t"
        "ds_read_b64_tr_b16 v[174:175], v11 offset:9216\n\t"
        "s_movk_i32 s61, 36\n\t"
        "s_waitcnt lgkmcnt(13)\n\t"
        "v_mfma_f32_16x16x32_bf16 v[224:227], v[144:147], v[112:115], v[224:227]\n\t"
        "ds_read_b64_tr_b16 v[176:177], v12 offset:8192\n\t"
        "v_cmp_eq_u32 vcc, 0, v24\n\t"
        "v_mfma_f32_16x16x32_bf16 v[240:243], v[144:147], v[116:119], v[240:243]\n\t"
        "ds_read_b64_tr_b16 v[178:179], v12 offset:9216\n\t"
        "s_getpc_b64 s[70:71]\n\t"
        "s_cbranch_vccnz .Lspin4_%=\n\t"
        "v_mfma_f32_16x16x32_bf16 v[228:231], v[148:151], v[112:115], v[228:231]\n\t"
        "ds_read_b64_tr_b16 v[180:181], v13 offset:8192\n\t"
        "ds_read_b32 v121, v127 offset:4\n\t"
        "ds_read_b32 v25, v127 offset:404\n\t"
        "v_mfma_f32_16x16x32_bf16 v[244:247], v[148:151], v[116:119], v[244:247]\n\t"
        "ds_read_b64_tr_b16 v[182:183], v13 offset:9216\n\t"
        "v_add_u32 v24, v24, %2\n\t"
        "s_waitcnt lgkmcnt(14)\n\t"
        "v_mfma_f32_16x16x32_bf16 v[232:235], v[152:155], v[112:115], v[232:235]\n\t"
        "ds_read_b64_tr_b16 v[184:185], v14 offset:8192\n\t"
        "global_load_dwordx4 v[56:59], v24, s[68:69] offset:0\n\t"
        "v_mfma_f32_16x16x32_bf16 v[248:251], v[152:155], v[116:119], v[248:251]\n\t"
        "ds_read_b64_tr_b16 v[186:187], v14 offset:9216\n\t"
        "ds_add_u32 %4, v31\n\t"
        "v_mfma_f32_16x16x32_bf16 v[236:239], v[156:159], v[112:115], v[236:239]\n\t"
        "ds_read_b64_tr_b16 v[188:189], v15 offset:8192\n\t"
        "global_load_dwordx4 v[60:63], v24, s[68:69] offset:1024\n\t"
        "v_mfma_f32_16x16x32_bf16 v[252:255], v[156:159], v[116:119], v[252:255]\n\t"
        "ds_read_b64_tr_b16 v[190:191], v15 offset:9216\n\t"
        "s_waitcnt vmcnt(6) lgkmcnt(15)\n\t"
        "v_mfma_f32_16x16x32_bf16 v[192:195], v[160:163], v[32:35], v[192:195]\n\t"
        "ds_read_b64_tr_b16 v[128:129], v8 offset:16384\n\t"
        "v_mfma_f32_16x16x32_bf16 v[208:211], v[160:163], v[36:39], v[208:211]\n\t"
        "ds_read_b64_tr_b16 v[130:131], v8 offset:17408\n\t"
        "v_mfma_f32_16x16x32_bf16 v[196:199], v[164:167], v[32:35], v[196:199]\n\t"
        "ds_read_b64_tr_b16 v[132:133], v9 offset:16384\n\t"
        "v_mfma_f32_16x16x32_bf16 v[212:215], v[164:167], v[36:39], v[212:215]\n\t"
        "ds_read_b64_tr_b16 v[134:135], v9 offset:17408\n\t"
        "s_waitcnt lgkmcnt(10)\n\t"
        "v_mfma_f32_16x16x32_bf16 v[200:203], v[168:171], v[32:35], v[200:203]\n\t"
        "ds_read_b64_tr_b16 v[136:137], v10 offset:16384\n\t"
        "s_movk_i32 s61, 35\n\t"
        "v_mfma_f32_16x16x32_bf16 v[216:219], v[168:171], v[36:39], v[216:219]\n\t"
        "ds_read_b64_tr_b16 v[138:139], v10 offset:17408\n\t"
        "v_cmp_ge_u32 vcc, s61, v121\n\t"
        "v_mfma_f32_16x16x32_bf16 v[204:207], v[172:175], v[32:35], v[204:207]\n\t"
        "ds_read_b64_tr_b16 v[140:141], v11 offset:16384\n\t"
        "s_getpc_b64 s[70:71]\n\t"
        "s_cbranch_vccnz .Lspin1_%=\n\t"
        "v_mfma_f32_16x16x32_bf16 v[220:223], v[172:175], v[36:39], v[220:223]\n\t"
        "ds_read_b64_tr_b16 v[142:143], v11 offset:17408\n\t"
        "s_movk_i32 s61, 37\n\t"
        "s_waitcnt lgkmcnt(13)\n\t"
        "v_mfma_f32_16x16x32_bf16 v[224:227], v[176:179], v[32:35], v[224:227]\n\t"
        "ds_read_b64_tr_b16 v[144:145], v12 offset:16384\n\t"
        "v_cmp_eq_u32 vcc, 0, v25\n\t"
        "v_mfma_f32_16x16x32_bf16 v[240:243], v[176:179], v[36:39], v[240:243]\n\t"
        "ds_read_b64_tr_b16 v[146:147], v12 offset:17408\n\t"
        "s_getpc_b64 s[70:71]\n\t"
        "s_cbranch_vccnz .Lspin5_%=\n\t"
        "v_mfma_f32_16x16x32_bf16 v[228:231], v[180:183], v[32:35], v[228:231]\n\t"
        "ds_read_b64_tr_b16 v[148:149], v13 offset:16384\n\t"
        "ds_read_b32 v121, v127 offset:4\n\t"
        "ds_read_b32 v26, v127 offset:408\n\t"
        "v_mfma_f32_16x16x32_bf16 v[244:247], v[180:183], v[36:39], v[244:247]\n\t"
        "ds_read_b64_tr_b16 v[150:151], v13 offset:17408\n\t"
        "v_add_u32 v25, v25, %2\n\t"
        "s_waitcnt lgkmcnt(14)\n\t"
        "v_mfma_f32_16x16x32_bf16 v[232:235], v[184:187], v[32:35], v[232:235]\n\t"
        "ds_read_b64_tr_b16 v[152:153], v14 offset:16384\n\t"
        "global_load_dwordx4 v[64:67], v25, s[68:69] offset:0\n\t"
        "v_mfma_f32_16x16x32_bf16 v[248:251], v[184:187], v[36:39], v[248:251]\n\t"
        "ds_read_b64_tr_b16 v[154:155], v14 offset:17408\n\t"
        "ds_add_u32 %4, v31\n\t"
        "v_mfma_f32_16x16x32_bf16 v[236:239], v[188:191], v[32:35], v[236:239]\n\t"
        "ds_read_b64_tr_b16 v[156:157], v15 offset:16384\n\t"
        "global_load_dwordx4 v[68:71], v25, s[68:69] offset:1024\n\t"
        "v_mfma_f32_16x16x32_bf16 v[252:255], v[188:191], v[36:39], v[252:255]\n\t"
        "ds_read_b64_tr_b16 v[158:159], v15 offset:17408\n\t"
        "s_waitcnt vmcnt(6) lgkmcnt(15)\n\t"
        "v_mfma_f32_16x16x32_bf16 v[192:195], v[128:131], v[40:43], v[192:195]\n\t"
        "ds_read_b64_tr_b16 v[160:161], v8 offset:24576\n\t"
        "v_mfma_f32_16x16x32_bf16 v[208:211], v[128:131], v[44:47], v[208:211]\n\t"
        "ds_read_b64_tr_b16 v[162:163], v8 offset:25600\n\t"
        "v_mfma_f32_16x16x32_bf16 v[196:199], v[132:135], v[40:43], v[196:199]\n\t"
        "ds_read_b64_tr_b16 v[164:165], v9 offset:24576\n\t"
        "v_mfma_f32_16x16x32_bf16 v[212:215], v[132:135], v[44:47], v[212:215]\n\t"
        "ds_read_b64_tr_b16 v[166:167], v9 offset:25600\n\t"
        "s_waitcnt lgkmcnt(10)\n\t"
        "v_mfma_f32_16x16x32_bf16 v[200:203], v[136:139], v[40:43], v[200:203]\n\t"
        "ds_read_b64_tr_b16 v[168:169], v10 offset:24576\n\t"
        "s_movk_i32 s61, 36\n\t"
        "v_mfma_f32_16x16x32_bf16 v[216:219], v[136:139], v[44:47], v[216:219]\n\t"
        "ds_read_b64_tr_b16 v[170:171], v10 offset:25600\n\t"
        "v_cmp_ge_u32 vcc, s61, v121\n\t"
        "v_mfma_f32_16x16x32_bf16 v[204:207], v[140:143], v[40:43], v[204:207]\n\t"
        "ds_read_b64_tr_b16 v[172:173], v11 offset:24576\n\t"
        "s_getpc_b64 s[70:71]\n\t"
        "s_cbranch_vccnz .Lspin1_%=\n\t"
        "v_mfma_f32_16x16x32_bf16 v[220:223], v[140:143], v[44:47], v[220:223]\n\t"
        "ds_read_b64_tr_b16 v[174:175], v11 offset:25600\n\t"
        "s_movk_i32 s61, 38\n\t"
        "s_waitcnt lgkmcnt(13)\n\t"
        "v_mfma_f32_16x16x32_bf16 v[224:227], v[144:147], v[40:43], v[224:227]\n\t"
        "ds_read_b64_tr_b16 v[176:177], v12 offset:24576\n\t"
        "v_cmp_eq_u32 vcc, 0, v26\n\t"
        "v_mfma_f32_16x16x32_bf16 v[240:243], v[144:147], v[44:47], v[240:243]\n\t"
        "ds_read_b64_tr_b16 v[178:179], v12 offset:25600\n\t"
        "s_getpc_b64 s[70:71]\n\t"
        "s_cbranch_vccnz .Lspin6_%=\n\t"
        "v_mfma_f32_16x16x32_bf16 v[228:231], v[148:151], v[40:43], v[228:231]\n\t"
        "ds_read_b64_tr_b16 v[180:181], v13 offset:24576\n\t"
        "ds_read_b32 v121, v127 offset:4\n\t"
        "ds_read_b32 v27, v127 offset:412\n\t"
        "v_mfma_f32_16x16x32_bf16 v[244:247], v[148:151], v[44:47], v[244:247]\n\t"
        "ds_read_b64_tr_b16 v[182:183], v13 offset:25600\n\t"
        "v_add_u32 v26, v26, %2\n\t"
        "s_waitcnt lgkmcnt(14)\n\t"
        "v_mfma_f32_16x16x32_bf16 v[232:235], v[152:155], v[40:43], v[232:235]\n\t"
        "ds_read_b64_tr_b16 v[184:185], v14 offset:24576\n\t"
        "global_load_dwordx4 v[72:75], v26, s[68:69] offset:0\n\t"
        "v_mfma_f32_16x16x32_bf16 v[248:251], v[152:155], v[44:47], v[248:251]\n\t"
        "ds_read_b64_tr_b16 v[186:187], v14 offset:25600\n\t"
        "ds_add_u32 %4, v31\n\t"
        "v_mfma_f32_16x16x32_bf16 v[236:239], v[156:159], v[40:43], v[236:239]\n\t"
        "ds_read_b64_tr_b16 v[188:189], v15 offset:24576\n\t"
        "global_load_dwordx4 v[76:79], v26, s[68:69] offset:1024\n\t"
        "v_mfma_f32_16x16x32_bf16 v[252:255], v[156:159], v[44:47], v[252:255]\n\t"
        "ds_read_b64_tr_b16 v[190:191], v15 offset:25600\n\t"
        "s_waitcnt vmcnt(6) lgkmcnt(15)\n\t"
        "v_mfma_f32_16x16x32_bf16 v[192:195], v[160:163], v[48:51], v[192:195]\n\t"
        "ds_read_b64_tr_b16 v[128:129], v8 offset:32768\n\t"
        "v_mfma_f32_16x16x32_bf16 v[208:211], v[160:163], v[52:55], v[208:211]\n\t"
        "ds_read_b64_tr_b16 v[130:131], v8 offset:33792\n\t"
        "v_mfma_f32_16x16x32_bf16 v[196:199], v[164:167], v[48:51], v[196:199]\n\t"
        "ds_read_b64_tr_b16 v[132:133], v9 offset:32768\n\t"
        "v_mfma_f32_16x16x32_bf16 v[212:215], v[164:167], v[52:55], v[212:215]\n\t"
        "ds_read_b64_tr_b16 v[134:135], v9 offset:33792\n\t"
        "s_waitcnt lgkmcnt(10)\n\t"
        "v_mfma_f32_16x16x32_bf16 v[200:203], v[168:171], v[48:51], v[200:203]\n\t"
        "ds_read_b64_tr_b16 v[136:137], v10 offset:32768\n\t"
        "s_movk_i32 s61, 37\n\t"
        "v_mfma_f32_16x16x32_bf16 v[216:219], v[168:171], v[52:55], v[216:219]\n\t"
        "ds_read_b64_tr_b16 v[138:139], v10 offset:33792\n\t"
        "v_cmp_ge_u32 vcc, s61, v121\n\t"
        "v_mfma_f32_16x16x32_bf16 v[204:207], v[172:175], v[48:51], v[204:207]\n\t"
        "ds_read_b64_tr_b16 v[140:141], v11 offset:32768\n\t"
        "s_getpc_b64 s[70:71]\n\t"
        "s_cbranch_vccnz .Lspin1_%=\n\t"
        "v_mfma_f32_16x16x32_bf16 v[220:223], v[172:175], v[52:55], v[220:223]\n\t"
        "ds_read_b64_tr_b16 v[142:143], v11 offset:33792\n\t"
        "s_movk_i32 s61, 39\n\t"
        "s_waitcnt lgkmcnt(13)\n\t"
        "v_mfma_f32_16x16x32_bf16 v[224:227], v[176:179], v[48:51], v[224:227]\n\t"
        "ds_read_b64_tr_b16 v[144:145], v12 offset:32768\n\t"
        "v_cmp_eq_u32 vcc, 0, v27\n\t"
        "v_mfma_f32_16x16x32_bf16 v[240:243], v[176:179], v[52:55], v[240:243]\n\t"
        "ds_read_b64_tr_b16 v[146:147], v12 offset:33792\n\t"
        "s_getpc_b64 s[70:71]\n\t"
        "s_cbranch_vccnz .Lspin7_%=\n\t"
        "v_mfma_f32_16x16x32_bf16 v[228:231], v[180:183], v[48:51], v[228:231]\n\t"
        "ds_read_b64_tr_b16 v[148:149], v13 offset:32768\n\t"
        "ds_read_b32 v121, v127 offset:4\n\t"
        "ds_read_b32 v24, v127 offset:416\n\t"
        "v_mfma_f32_16x16x32_bf16 v[244:247], v[180:183], v[52:55], v[244:247]\n\t"
        "ds_read_b64_tr_b16 v[150:151], v13 offset:33792\n\t"
        "v_add_u32 v27, v27, %2\n\t"
        "s_waitcnt lgkmcnt(14)\n\t"
        "v_mfma_f32_16x16x32_bf16 v[232:235], v[184:187], v[48:51], v[232:235]\n\t"
        "ds_read_b64_tr_b16 v[152:153], v14 offset:32768\n\t"
        "global_load_dwordx4 v[80:83], v27, s[68:69] offset:0\n\t"
        "v_mfma_f32_16x16x32_bf16 v[248:251], v[184:187], v[52:55], v[248:251]\n\t"
        "ds_read_b64_tr_b16 v[154:155], v14 offset:33792\n\t"
        "ds_add_u32 %4, v31\n\t"
        "v_mfma_f32_16x16x32_bf16 v[236:239], v[188:191], v[48:51], v[236:239]\n\t"
        "ds_read_b64_tr_b16 v[156:157], v15 offset:32768\n\t"
        "global_load_dwordx4 v[84:87], v27, s[68:69] offset:1024\n\t"
        "v_mfma_f32_16x16x32_bf16 v[252:255], v[188:191], v[52:55], v[252:255]\n\t"
        "ds_read_b64_tr_b16 v[158:159], v15 offset:33792\n\t"
        "s_waitcnt vmcnt(6) lgkmcnt(15)\n\t"
        "v_mfma_f32_16x16x32_bf16 v[192:195], v[128:131], v[56:59], v[192:195]\n\t"
        "ds_read_b64_tr_b16 v[160:161], v8 offset:40960\n\t"
        "v_mfma_f32_16x16x32_bf16 v[208:211], v[128:131], v[60:63], v[208:211]\n\t"
        "ds_read_b64_tr_b16 v[162:163], v8 offset:41984\n\t"
        "v_mfma_f32_16x16x32_bf16 v[196:199], v[132:135], v[56:59], v[196:199]\n\t"
        "ds_read_b64_tr_b16 v[164:165], v9 offset:40960\n\t"
        "v_mfma_f32_16x16x32_bf16 v[212:215], v[132:135], v[60:63], v[212:215]\n\t"
        "ds_read_b64_tr_b16 v[166:167], v9 offset:41984\n\t"
        "s_waitcnt lgkmcnt(10)\n\t"
        "v_mfma_f32_16x16x32_bf16 v[200:203], v[136:139], v[56:59], v[200:203]\n\t"
        "ds_read_b64_tr_b16 v[168:169], v10 offset:40960\n\t"
        "s_movk_i32 s61, 38\n\t"
        "v_mfma_f32_16x16x32_bf16 v[216:219], v[136:139], v[60:63], v[216:219]\n\t"
        "ds_read_b64_tr_b16 v[170:171], v10 offset:41984\n\t"
        "v_cmp_ge_u32 vcc, s61, v121\n\t"
        "v_mfma_f32_16x16x32_bf16 v[204:207], v[140:143], v[56:59], v[204:207]\n\t"
        "ds_read_b64_tr_b16 v[172:173], v11 offset:40960\n\t"
        "s_getpc_b64 s[70:71]\n\t"
        "s_cbranch_vccnz .Lspin1_%=\n\t"
        "v_mfma_f32_16x16x32_bf16 v[220:223], v[140:143], v[60:63], v[220:223]\n\t"
        "ds_read_b64_tr_b16 v[174:175], v11 offset:41984\n\t"
        "s_movk_i32 s61, 40\n\t"
        "s_waitcnt lgkmcnt(13)\n\t"
        "v_mfma_f32_16x16x32_bf16 v[224:227], v[144:147], v[56:59], v[224:227]\n\t"
        "ds_read_b64_tr_b16 v[176:177], v12 offset:40960\n\t"
        "v_cmp_eq_u32 vcc, 0, v24\n\t"
        "v_mfma_f32_16x16x32_bf16 v[240:243], v[144:147], v[60:63], v[240:243]\n\t"
        "ds_read_b64_tr_b16 v[178:179], v12 offset:41984\n\t"
        "s_getpc_b64 s[70:71]\n\t"
        "s_cbranch_vccnz .Lspin4_%=\n\t"
        "v_mfma_f32_16x16x32_bf16 v[228:231], v[148:151], v[56:59], v[228:231]\n\t"
        "ds_read_b64_tr_b16 v[180:181], v13 offset:40960\n\t"
        "ds_read_b32 v121, v127 offset:4\n\t"
        "ds_read_b32 v25, v127 offset:420\n\t"
        "v_mfma_f32_16x16x32_bf16 v[244:247], v[148:151], v[60:63], v[244:247]\n\t"
        "ds_read_b64_tr_b16 v[182:183], v13 offset:41984\n\t"
        "v_add_u32 v24, v24, %2\n\t"
        "s_waitcnt lgkmcnt(14)\n\t"
        "v_mfma_f32_16x16x32_bf16 v[232:235], v[152:155], v[56:59], v[232:235]\n\t"
        "ds_read_b64_tr_b16 v[184:185], v14 offset:40960\n\t"
        "global_load_dwordx4 v[88:91], v24, s[68:69] offset:0\n\t"
        "v_mfma_f32_16x16x32_bf16 v[248:251], v[152:155], v[60:63], v[248:251]\n\t"
        "ds_read_b64_tr_b16 v[186:187], v14 offset:41984\n\t"
        "ds_add_u32 %4, v31\n\t"
        "v_mfma_f32_16x16x32_bf16 v[236:239], v[156:159], v[56:59], v[236:239]\n\t"
        "ds_read_b64_tr_b16 v[188:189], v15 offset:40960\n\t"
        "global_load_dwordx4 v[92:95], v24, s[68:69] offset:1024\n\t"
        "v_mfma_f32_16x16x32_bf16 v[252:255], v[156:159], v[60:63], v[252:255]\n\t"
        "ds_read_b64_tr_b16 v[190:191], v15 offset:41984\n\t"
        "s_waitcnt vmcnt(6) lgkmcnt(15)\n\t"
        "v_mfma_f32_16x16x32_bf16 v[192:195], v[160:163], v[64:67], v[192:195]\n\t"
        "ds_read_b64_tr_b16 v[128:129], v8 offset:49152\n\t"
        "v_mfma_f32_16x16x32_bf16 v[208:211], v[160:163], v[68:71], v[208:211]\n\t"
        "ds_read_b64_tr_b16 v[130:131], v8 offset:50176\n\t"
        "v_mfma_f32_16x16x32_bf16 v[196:199], v[164:167], v[64:67], v[196:199]\n\t"
        "ds_read_b64_tr_b16 v[132:133], v9 offset:49152\n\t"
        "v_mfma_f32_16x16x32_bf16 v[212:215], v[164:167], v[68:71], v[212:215]\n\t"
        "ds_read_b64_tr_b16 v[134:135], v9 offset:50176\n\t"
        "s_waitcnt lgkmcnt(10)\n\t"
        "v_mfma_f32_16x16x32_bf16 v[200:203], v[168:171], v[64:67], v[200:203]\n\t"
        "ds_read_b64_tr_b16 v[136:137], v10 offset:49152\n\t"
        "s_movk_i32 s61, 39\n\t"
        "v_mfma_f32_16x16x32_bf16 v[216:219], v[168:171], v[68:71], v[216:219]\n\t"
        "ds_read_b64_tr_b16 v[138:139], v10 offset:50176\n\t"
        "v_cmp_ge_u32 vcc, s61, v121\n\t"
        "v_mfma_f32_16x16x32_bf16 v[204:207], v[172:175], v[64:67], v[204:207]\n\t"
        "ds_read_b64_tr_b16 v[140:141], v11 offset:49152\n\t"
        "s_getpc_b64 s[70:71]\n\t"
        "s_cbranch_vccnz .Lspin1_%=\n\t"
        "v_mfma_f32_16x16x32_bf16 v[220:223], v[172:175], v[68:71], v[220:223]\n\t"
        "ds_read_b64_tr_b16 v[142:143], v11 offset:50176\n\t"
        "s_movk_i32 s61, 41\n\t"
        "s_waitcnt lgkmcnt(13)\n\t"
        "v_mfma_f32_16x16x32_bf16 v[224:227], v[176:179], v[64:67], v[224:227]\n\t"
        "ds_read_b64_tr_b16 v[144:145], v12 offset:49152\n\t"
        "v_cmp_eq_u32 vcc, 0, v25\n\t"
        "v_mfma_f32_16x16x32_bf16 v[240:243], v[176:179], v[68:71], v[240:243]\n\t"
        "ds_read_b64_tr_b16 v[146:147], v12 offset:50176\n\t"
        "s_getpc_b64 s[70:71]\n\t"
        "s_cbranch_vccnz .Lspin5_%=\n\t"
        "v_mfma_f32_16x16x32_bf16 v[228:231], v[180:183], v[64:67], v[228:231]\n\t"
        "ds_read_b64_tr_b16 v[148:149], v13 offset:49152\n\t"
        "ds_read_b32 v121, v127 offset:4\n\t"
        "ds_read_b32 v26, v127 offset:424\n\t"
        "v_mfma_f32_16x16x32_bf16 v[244:247], v[180:183], v[68:71], v[244:247]\n\t"
        "ds_read_b64_tr_b16 v[150:151], v13 offset:50176\n\t"
        "v_add_u32 v25, v25, %2\n\t"
        "s_waitcnt lgkmcnt(14)\n\t"
        "v_mfma_f32_16x16x32_bf16 v[232:235], v[184:187], v[64:67], v[232:235]\n\t"
        "ds_read_b64_tr_b16 v[152:153], v14 offset:49152\n\t"
        "global_load_dwordx4 v[96:99], v25, s[68:69] offset:0\n\t"
        "v_mfma_f32_16x16x32_bf16 v[248:251], v[184:187], v[68:71], v[248:251]\n\t"
        "ds_read_b64_tr_b16 v[154:155], v14 offset:50176\n\t"
        "ds_add_u32 %4, v31\n\t"
        "v_mfma_f32_16x16x32_bf16 v[236:239], v[188:191], v[64:67], v[236:239]\n\t"
        "ds_read_b64_tr_b16 v[156:157], v15 offset:49152\n\t"
        "global_load_dwordx4 v[100:103], v25, s[68:69] offset:1024\n\t"
        "v_mfma_f32_16x16x32_bf16 v[252:255], v[188:191], v[68:71], v[252:255]\n\t"
        "ds_read_b64_tr_b16 v[158:159], v15 offset:50176\n\t"
        "s_waitcnt vmcnt(6) lgkmcnt(15)\n\t"
        "v_mfma_f32_16x16x32_bf16 v[192:195], v[128:131], v[72:75], v[192:195]\n\t"
        "ds_read_b64_tr_b16 v[160:161], v8 offset:57344\n\t"
        "v_mfma_f32_16x16x32_bf16 v[208:211], v[128:131], v[76:79], v[208:211]\n\t"
        "ds_read_b64_tr_b16 v[162:163], v8 offset:58368\n\t"
        "v_mfma_f32_16x16x32_bf16 v[196:199], v[132:135], v[72:75], v[196:199]\n\t"
        "ds_read_b64_tr_b16 v[164:165], v9 offset:57344\n\t"
        "v_mfma_f32_16x16x32_bf16 v[212:215], v[132:135], v[76:79], v[212:215]\n\t"
        "ds_read_b64_tr_b16 v[166:167], v9 offset:58368\n\t"
        "s_waitcnt lgkmcnt(10)\n\t"
        "v_mfma_f32_16x16x32_bf16 v[200:203], v[136:139], v[72:75], v[200:203]\n\t"
        "ds_read_b64_tr_b16 v[168:169], v10 offset:57344\n\t"
        "s_movk_i32 s61, 40\n\t"
        "v_mfma_f32_16x16x32_bf16 v[216:219], v[136:139], v[76:79], v[216:219]\n\t"
        "ds_read_b64_tr_b16 v[170:171], v10 offset:58368\n\t"
        "v_cmp_ge_u32 vcc, s61, v121\n\t"
        "v_mfma_f32_16x16x32_bf16 v[204:207], v[140:143], v[72:75], v[204:207]\n\t"
        "ds_read_b64_tr_b16 v[172:173], v11 offset:57344\n\t"
        "s_getpc_b64 s[70:71]\n\t"
        "s_cbranch_vccnz .Lspin1_%=\n\t"
        "v_mfma_f32_16x16x32_bf16 v[220:223], v[140:143], v[76:79], v[220:223]\n\t"
        "ds_read_b64_tr_b16 v[174:175], v11 offset:58368\n\t"
        "s_movk_i32 s61, 42\n\t"
        "s_waitcnt lgkmcnt(13)\n\t"
        "v_mfma_f32_16x16x32_bf16 v[224:227], v[144:147], v[72:75], v[224:227]\n\t"
        "ds_read_b64_tr_b16 v[176:177], v12 offset:57344\n\t"
        "v_cmp_eq_u32 vcc, 0, v26\n\t"
        "v_mfma_f32_16x16x32_bf16 v[240:243], v[144:147], v[76:79], v[240:243]\n\t"
        "ds_read_b64_tr_b16 v[178:179], v12 offset:58368\n\t"
        "s_getpc_b64 s[70:71]\n\t"
        "s_cbranch_vccnz .Lspin6_%=\n\t"
        "v_mfma_f32_16x16x32_bf16 v[228:231], v[148:151], v[72:75], v[228:231]\n\t"
        "ds_read_b64_tr_b16 v[180:181], v13 offset:57344\n\t"
        "ds_read_b32 v121, v127 offset:4\n\t"
        "ds_read_b32 v27, v127 offset:428\n\t"
        "v_mfma_f32_16x16x32_bf16 v[244:247], v[148:151], v[76:79], v[244:247]\n\t"
        "ds_read_b64_tr_b16 v[182:183], v13 offset:58368\n\t"
        "v_add_u32 v26, v26, %2\n\t"
        "s_waitcnt lgkmcnt(14)\n\t"
        "v_mfma_f32_16x16x32_bf16 v[232:235], v[152:155], v[72:75], v[232:235]\n\t"
        "ds_read_b64_tr_b16 v[184:185], v14 offset:57344\n\t"
        "global_load_dwordx4 v[104:107], v26, s[68:69] offset:0\n\t"
        "v_mfma_f32_16x16x32_bf16 v[248:251], v[152:155], v[76:79], v[248:251]\n\t"
        "ds_read_b64_tr_b16 v[186:187], v14 offset:58368\n\t"
        "ds_add_u32 %4, v31\n\t"
        "v_mfma_f32_16x16x32_bf16 v[236:239], v[156:159], v[72:75], v[236:239]\n\t"
        "ds_read_b64_tr_b16 v[188:189], v15 offset:57344\n\t"
        "global_load_dwordx4 v[108:111], v26, s[68:69] offset:1024\n\t"
        "v_mfma_f32_16x16x32_bf16 v[252:255], v[156:159], v[76:79], v[252:255]\n\t"
        "ds_read_b64_tr_b16 v[190:191], v15 offset:58368\n\t"
        "s_waitcnt vmcnt(6) lgkmcnt(15)\n\t"
        "v_mfma_f32_16x16x32_bf16 v[192:195], v[160:163], v[80:83], v[192:195]\n\t"
        "ds_read_b64_tr_b16 v[128:129], v16 offset:0\n\t"
        "v_mfma_f32_16x16x32_bf16 v[208:211], v[160:163], v[84:87], v[208:211]\n\t"
        "ds_read_b64_tr_b16 v[130:131], v16 offset:1024\n\t"
        "v_mfma_f32_16x16x32_bf16 v[196:199], v[164:167], v[80:83], v[196:199]\n\t"
        "ds_read_b64_tr_b16 v[132:133], v17 offset:0\n\t"
        "v_mfma_f32_16x16x32_bf16 v[212:215], v[164:167], v[84:87], v[212:215]\n\t"
        "ds_read_b64_tr_b16 v[134:135], v17 offset:1024\n\t"
        "s_waitcnt lgkmcnt(10)\n\t"
        "v_mfma_f32_16x16x32_bf16 v[200:203], v[168:171], v[80:83], v[200:203]\n\t"
        "ds_read_b64_tr_b16 v[136:137], v18 offset:0\n\t"
        "s_movk_i32 s61, 41\n\t"
        "v_mfma_f32_16x16x32_bf16 v[216:219], v[168:171], v[84:87], v[216:219]\n\t"
        "ds_read_b64_tr_b16 v[138:139], v18 offset:1024\n\t"
        "v_cmp_ge_u32 vcc, s61, v121\n\t"
        "v_mfma_f32_16x16x32_bf16 v[204:207], v[172:175], v[80:83], v[204:207]\n\t"
        "ds_read_b64_tr_b16 v[140:141], v19 offset:0\n\t"
        "s_getpc_b64 s[70:71]\n\t"
        "s_cbranch_vccnz .Lspin1_%=\n\t"
        "v_mfma_f32_16x16x32_bf16 v[220:223], v[172:175], v[84:87], v[220:223]\n\t"
        "ds_read_b64_tr_b16 v[142:143], v19 offset:1024\n\t"
        "s_movk_i32 s61, 43\n\t"
        "s_waitcnt lgkmcnt(13)\n\t"
        "v_mfma_f32_16x16x32_bf16 v[224:227], v[176:179], v[80:83], v[224:227]\n\t"
        "ds_read_b64_tr_b16 v[144:145], v20 offset:0\n\t"
        "v_cmp_eq_u32 vcc, 0, v27\n\t"
        "v_mfma_f32_16x16x32_bf16 v[240:243], v[176:179], v[84:87], v[240:243]\n\t"
        "ds_read_b64_tr_b16 v[146:147], v20 offset:1024\n\t"
        "s_getpc_b64 s[70:71]\n\t"
        "s_cbranch_vccnz .Lspin7_%=\n\t"
        "v_mfma_f32_16x16x32_bf16 v[228:231], v[180:183], v[80:83], v[228:231]\n\t"
        "ds_read_b64_tr_b16 v[148:149], v21 offset:0\n\t"
        "ds_read_b32 v121, v127 offset:4\n\t"
        "ds_read_b32 v24, v127 offset:432\n\t"
        "v_mfma_f32_16x16x32_bf16 v[244:247], v[180:183], v[84:87], v[244:247]\n\t"
        "ds_read_b64_tr_b16 v[150:151], v21 offset:1024\n\t"
        "v_add_u32 v27, v27, %2\n\t"
        "s_waitcnt lgkmcnt(14)\n\t"
        "v_mfma_f32_16x16x32_bf16 v[232:235], v[184:187], v[80:83], v[232:235]\n\t"
        "ds_read_b64_tr_b16 v[152:153], v22 offset:0\n\t"
        "global_load_dwordx4 v[112:115], v27, s[68:69] offset:0\n\t"
        "v_mfma_f32_16x16x32_bf16 v[248:251], v[184:187], v[84:87], v[248:251]\n\t"
        "ds_read_b64_tr_b16 v[154:155], v22 offset:1024\n\t"
        "ds_add_u32 %4, v31\n\t"
        "v_mfma_f32_16x16x32_bf16 v[236:239], v[188:191], v[80:83], v[236:239]\n\t"
        "ds_read_b64_tr_b16 v[156:157], v23 offset:0\n\t"
        "global_load_dwordx4 v[116:119], v27, s[68:69] offset:1024\n\t"
        "v_mfma_f32_16x16x32_bf16 v[252:255], v[188:191], v[84:87], v[252:255]\n\t"
        "ds_read_b64_tr_b16 v[158:159], v23 offset:1024\n\t"
        "s_waitcnt vmcnt(6) lgkmcnt(15)\n\t"
        "v_mfma_f32_16x16x32_bf16 v[192:195], v[128:131], v[88:91], v[192:195]\n\t"
        "ds_read_b64_tr_b16 v[160:161], v16 offset:8192\n\t"
        "v_mfma_f32_16x16x32_bf16 v[208:211], v[128:131], v[92:95], v[208:211]\n\t"
        "ds_read_b64_tr_b16 v[162:163], v16 offset:9216\n\t"
        "v_mfma_f32_16x16x32_bf16 v[196:199], v[132:135], v[88:91], v[196:199]\n\t"
        "ds_read_b64_tr_b16 v[164:165], v17 offset:8192\n\t"
        "v_mfma_f32_16x16x32_bf16 v[212:215], v[132:135], v[92:95], v[212:215]\n\t"
        "ds_read_b64_tr_b16 v[166:167], v17 offset:9216\n\t"
        "s_waitcnt lgkmcnt(10)\n\t"
        "v_mfma_f32_16x16x32_bf16 v[200:203], v[136:139], v[88:91], v[200:203]\n\t"
        "ds_read_b64_tr_b16 v[168:169], v18 offset:8192\n\t"
        "s_movk_i32 s61, 42\n\t"
        "v_mfma_f32_16x16x32_bf16 v[216:219], v[136:139], v[92:95], v[216:219]\n\t"
        "ds_read_b64_tr_b16 v[170:171], v18 offset:9216\n\t"
        "v_cmp_ge_u32 vcc, s61, v121\n\t"
        "v_mfma_f32_16x16x32_bf16 v[204:207], v[140:143], v[88:91], v[204:207]\n\t"
        "ds_read_b64_tr_b16 v[172:173], v19 offset:8192\n\t"
        "s_getpc_b64 s[70:71]\n\t"
        "s_cbranch_vccnz .Lspin1_%=\n\t"
        "v_mfma_f32_16x16x32_bf16 v[220:223], v[140:143], v[92:95], v[220:223]\n\t"
        "ds_read_b64_tr_b16 v[174:175], v19 offset:9216\n\t"
        "s_movk_i32 s61, 44\n\t"
        "s_waitcnt lgkmcnt(13)\n\t"
        "v_mfma_f32_16x16x32_bf16 v[224:227], v[144:147], v[88:91], v[224:227]\n\t"
        "ds_read_b64_tr_b16 v[176:177], v20 offset:8192\n\t"
        "v_cmp_eq_u32 vcc, 0, v24\n\t"
        "v_mfma_f32_16x16x32_bf16 v[240:243], v[144:147], v[92:95], v[240:243]\n\t"
        "ds_read_b64_tr_b16 v[178:179], v20 offset:9216\n\t"
        "s_getpc_b64 s[70:71]\n\t"
        "s_cbranch_vccnz .Lspin4_%=\n\t"
        "v_mfma_f32_16x16x32_bf16 v[228:231], v[148:151], v[88:91], v[228:231]\n\t"
        "ds_read_b64_tr_b16 v[180:181], v21 offset:8192\n\t"
        "ds_read_b32 v121, v127 offset:4\n\t"
        "ds_read_b32 v25, v127 offset:436\n\t"
        "v_mfma_f32_16x16x32_bf16 v[244:247], v[148:151], v[92:95], v[244:247]\n\t"
        "ds_read_b64_tr_b16 v[182:183], v21 offset:9216\n\t"
        "v_add_u32 v24, v24, %2\n\t"
        "s_waitcnt lgkmcnt(14)\n\t"
        "v_mfma_f32_16x16x32_bf16 v[232:235], v[152:155], v[88:91], v[232:235]\n\t"
        "ds_read_b64_tr_b16 v[184:185], v22 offset:8192\n\t"
        "global_load_dwordx4 v[32:35], v24, s[68:69] offset:0\n\t"
        "v_mfma_f32_16x16x32_bf16 v[248:251], v[152:155], v[92:95], v[248:251]\n\t"
        "ds_read_b64_tr_b16 v[186:187], v22 offset:9216\n\t"
        "ds_add_u32 %4, v31\n\t"
        "v_mfma_f32_16x16x32_bf16 v[236:239], v[156:159], v[88:91], v[236:239]\n\t"
        "ds_read_b64_tr_b16 v[188:189], v23 offset:8192\n\t"
        "global_load_dwordx4 v[36:39], v24, s[68:69] offset:1024\n\t"
        "v_mfma_f32_16x16x32_bf16 v[252:255], v[156:159], v[92:95], v[252:255]\n\t"
        "ds_read_b64_tr_b16 v[190:191], v23 offset:9216\n\t"
        "s_waitcnt vmcnt(6) lgkmcnt(15)\n\t"
        "v_mfma_f32_16x16x32_bf16 v[192:195], v[160:163], v[96:99], v[192:195]\n\t"
        "ds_read_b64_tr_b16 v[128:129], v16 offset:16384\n\t"
        "v_mfma_f32_16x16x32_bf16 v[208:211], v[160:163], v[100:103], v[208:211]\n\t"
        "ds_read_b64_tr_b16 v[130:131], v16 offset:17408\n\t"
        "v_mfma_f32_16x16x32_bf16 v[196:199], v[164:167], v[96:99], v[196:199]\n\t"
        "ds_read_b64_tr_b16 v[132:133], v17 offset:16384\n\t"
        "v_mfma_f32_16x16x32_bf16 v[212:215], v[164:167], v[100:103], v[212:215]\n\t"
        "ds_read_b64_tr_b16 v[134:135], v17 offset:17408\n\t"
        "s_waitcnt lgkmcnt(10)\n\t"
        "v_mfma_f32_16x16x32_bf16 v[200:203], v[168:171], v[96:99], v[200:203]\n\t"
        "ds_read_b64_tr_b16 v[136:137], v18 offset:16384\n\t"
        "s_movk_i32 s61, 43\n\t"
        "v_mfma_f32_16x16x32_bf16 v[216:219], v[168:171], v[100:103], v[216:219]\n\t"
        "ds_read_b64_tr_b16 v[138:139], v18 offset:17408\n\t"
        "v_cmp_ge_u32 vcc, s61, v121\n\t"
        "v_mfma_f32_16x16x32_bf16 v[204:207], v[172:175], v[96:99], v[204:207]\n\t"
        "ds_read_b64_tr_b16 v[140:141], v19 offset:16384\n\t"
        "s_getpc_b64 s[70:71]\n\t"
        "s_cbranch_vccnz .Lspin1_%=\n\t"
        "v_mfma_f32_16x16x32_bf16 v[220:223], v[172:175], v[100:103], v[220:223]\n\t"
        "ds_read_b64_tr_b16 v[142:143], v19 offset:17408\n\t"
        "s_movk_i32 s61, 45\n\t"
        "s_waitcnt lgkmcnt(13)\n\t"
        "v_mfma_f32_16x16x32_bf16 v[224:227], v[176:179], v[96:99], v[224:227]\n\t"
        "ds_read_b64_tr_b16 v[144:145], v20 offset:16384\n\t"
        "v_cmp_eq_u32 vcc, 0, v25\n\t"
        "v_mfma_f32_16x16x32_bf16 v[240:243], v[176:179], v[100:103], v[240:243]\n\t"
        "ds_read_b64_tr_b16 v[146:147], v20 offset:17408\n\t"
        "s_getpc_b64 s[70:71]\n\t"
        "s_cbranch_vccnz .Lspin5_%=\n\t"
        "v_mfma_f32_16x16x32_bf16 v[228:231], v[180:183], v[96:99], v[228:231]\n\t"
        "ds_read_b64_tr_b16 v[148:149], v21 offset:16384\n\t"
        "ds_read_b32 v121, v127 offset:4\n\t"
        "ds_read_b32 v26, v127 offset:440\n\t"
        "v_mfma_f32_16x16x32_bf16 v[244:247], v[180:183], v[100:103], v[244:247]\n\t"
        "ds_read_b64_tr_b16 v[150:151], v21 offset:17408\n\t"
        "v_add_u32 v25, v25, %2\n\t"
        "s_waitcnt lgkmcnt(14)\n\t"
        "v_mfma_f32_16x16x32_bf16 v[232:235], v[184:187], v[96:99], v[232:235]\n\t"
        "ds_read_b64_tr_b16 v[152:153], v22 offset:16384\n\t"
        "global_load_dwordx4 v[40:43], v25, s[68:69] offset:0\n\t"
        "v_mfma_f32_16x16x32_bf16 v[248:251], v[184:187], v[100:103], v[248:251]\n\t"
        "ds_read_b64_tr_b16 v[154:155], v22 offset:17408\n\t"
        "ds_add_u32 %4, v31\n\t"
        "v_mfma_f32_16x16x32_bf16 v[236:239], v[188:191], v[96:99], v[236:239]\n\t"
        "ds_read_b64_tr_b16 v[156:157], v23 offset:16384\n\t"
        "global_load_dwordx4 v[44:47], v25, s[68:69] offset:1024\n\t"
        "v_mfma_f32_16x16x32_bf16 v[252:255], v[188:191], v[100:103], v[252:255]\n\t"
        "ds_read_b64_tr_b16 v[158:159], v23 offset:17408\n\t"
        "s_waitcnt vmcnt(6) lgkmcnt(15)\n\t"
        "v_mfma_f32_16x16x32_bf16 v[192:195], v[128:131], v[104:107], v[192:195]\n\t"
        "ds_read_b64_tr_b16 v[160:161], v16 offset:24576\n\t"
        "v_mfma_f32_16x16x32_bf16 v[208:211], v[128:131], v[108:111], v[208:211]\n\t"
        "ds_read_b64_tr_b16 v[162:163], v16 offset:25600\n\t"
        "v_mfma_f32_16x16x32_bf16 v[196:199], v[132:135], v[104:107], v[196:199]\n\t"
        "ds_read_b64_tr_b16 v[164:165], v17 offset:24576\n\t"
        "v_mfma_f32_16x16x32_bf16 v[212:215], v[132:135], v[108:111], v[212:215]\n\t"
        "ds_read_b64_tr_b16 v[166:167], v17 offset:25600\n\t"
        "s_waitcnt lgkmcnt(10)\n\t"
        "v_mfma_f32_16x16x32_bf16 v[200:203], v[136:139], v[104:107], v[200:203]\n\t"
        "ds_read_b64_tr_b16 v[168:169], v18 offset:24576\n\t"
        "s_movk_i32 s61, 44\n\t"
        "v_mfma_f32_16x16x32_bf16 v[216:219], v[136:139], v[108:111], v[216:219]\n\t"
        "ds_read_b64_tr_b16 v[170:171], v18 offset:25600\n\t"
        "v_cmp_ge_u32 vcc, s61, v121\n\t"
        "v_mfma_f32_16x16x32_bf16 v[204:207], v[140:143], v[104:107], v[204:207]\n\t"
        "ds_read_b64_tr_b16 v[172:173], v19 offset:24576\n\t"
        "s_getpc_b64 s[70:71]\n\t"
        "s_cbranch_vccnz .Lspin1_%=\n\t"
        "v_mfma_f32_16x16x32_bf16 v[220:223], v[140:143], v[108:111], v[220:223]\n\t"
        "ds_read_b64_tr_b16 v[174:175], v19 offset:25600\n\t"
        "s_movk_i32 s61, 46\n\t"
        "s_waitcnt lgkmcnt(13)\n\t"
        "v_mfma_f32_16x16x32_bf16 v[224:227], v[144:147], v[104:107], v[224:227]\n\t"
        "ds_read_b64_tr_b16 v[176:177], v20 offset:24576\n\t"
        "v_cmp_eq_u32 vcc, 0, v26\n\t"
        "v_mfma_f32_16x16x32_bf16 v[240:243], v[144:147], v[108:111], v[240:243]\n\t"
        "ds_read_b64_tr_b16 v[178:179], v20 offset:25600\n\t"
        "s_getpc_b64 s[70:71]\n\t"
        "s_cbranch_vccnz .Lspin6_%=\n\t"
        "v_mfma_f32_16x16x32_bf16 v[228:231], v[148:151], v[104:107], v[228:231]\n\t"
        "ds_read_b64_tr_b16 v[180:181], v21 offset:24576\n\t"
        "ds_read_b32 v121, v127 offset:4\n\t"
        "ds_read_b32 v27, v127 offset:444\n\t"
        "v_mfma_f32_16x16x32_bf16 v[244:247], v[148:151], v[108:111], v[244:247]\n\t"
        "ds_read_b64_tr_b16 v[182:183], v21 offset:25600\n\t"
        "v_add_u32 v26, v26, %2\n\t"
        "s_waitcnt lgkmcnt(14)\n\t"
        "v_mfma_f32_16x16x32_bf16 v[232:235], v[152:155], v[104:107], v[232:235]\n\t"
        "ds_read_b64_tr_b16 v[184:185], v22 offset:24576\n\t"
        "global_load_dwordx4 v[48:51], v26, s[68:69] offset:0\n\t"
        "v_mfma_f32_16x16x32_bf16 v[248:251], v[152:155], v[108:111], v[248:251]\n\t"
        "ds_read_b64_tr_b16 v[186:187], v22 offset:25600\n\t"
        "ds_add_u32 %4, v31\n\t"
        "v_mfma_f32_16x16x32_bf16 v[236:239], v[156:159], v[104:107], v[236:239]\n\t"
        "ds_read_b64_tr_b16 v[188:189], v23 offset:24576\n\t"
        "global_load_dwordx4 v[52:55], v26, s[68:69] offset:1024\n\t"
        "v_mfma_f32_16x16x32_bf16 v[252:255], v[156:159], v[108:111], v[252:255]\n\t"
        "ds_read_b64_tr_b16 v[190:191], v23 offset:25600\n\t"
        "s_waitcnt vmcnt(6) lgkmcnt(15)\n\t"
        "v_mfma_f32_16x16x32_bf16 v[192:195], v[160:163], v[112:115], v[192:195]\n\t"
        "ds_read_b64_tr_b16 v[128:129], v16 offset:32768\n\t"
        "v_mfma_f32_16x16x32_bf16 v[208:211], v[160:163], v[116:119], v[208:211]\n\t"
        "ds_read_b64_tr_b16 v[130:131], v16 offset:33792\n\t"
        "v_mfma_f32_16x16x32_bf16 v[196:199], v[164:167], v[112:115], v[196:199]\n\t"
        "ds_read_b64_tr_b16 v[132:133], v17 offset:32768\n\t"
        "v_mfma_f32_16x16x32_bf16 v[212:215], v[164:167], v[116:119], v[212:215]\n\t"
        "ds_read_b64_tr_b16 v[134:135], v17 offset:33792\n\t"
        "s_waitcnt lgkmcnt(10)\n\t"
        "v_mfma_f32_16x16x32_bf16 v[200:203], v[168:171], v[112:115], v[200:203]\n\t"
        "ds_read_b64_tr_b16 v[136:137], v18 offset:32768\n\t"
        "s_movk_i32 s61, 45\n\t"
        "v_mfma_f32_16x16x32_bf16 v[216:219], v[168:171], v[116:119], v[216:219]\n\t"
        "ds_read_b64_tr_b16 v[138:139], v18 offset:33792\n\t"
        "v_cmp_ge_u32 vcc, s61, v121\n\t"
        "v_mfma_f32_16x16x32_bf16 v[204:207], v[172:175], v[112:115], v[204:207]\n\t"
        "ds_read_b64_tr_b16 v[140:141], v19 offset:32768\n\t"
        "s_getpc_b64 s[70:71]\n\t"
        "s_cbranch_vccnz .Lspin1_%=\n\t"
        "v_mfma_f32_16x16x32_bf16 v[220:223], v[172:175], v[116:119], v[220:223]\n\t"
        "ds_read_b64_tr_b16 v[142:143], v19 offset:33792\n\t"
        "s_movk_i32 s61, 47\n\t"
        "s_waitcnt lgkmcnt(13)\n\t"
        "v_mfma_f32_16x16x32_bf16 v[224:227], v[176:179], v[112:115], v[224:227]\n\t"
        "ds_read_b64_tr_b16 v[144:145], v20 offset:32768\n\t"
        "v_cmp_eq_u32 vcc, 0, v27\n\t"
        "v_mfma_f32_16x16x32_bf16 v[240:243], v[176:179], v[116:119], v[240:243]\n\t"
        "ds_read_b64_tr_b16 v[146:147], v20 offset:33792\n\t"
        "s_getpc_b64 s[70:71]\n\t"
        "s_cbranch_vccnz .Lspin7_%=\n\t"
        "v_mfma_f32_16x16x32_bf16 v[228:231], v[180:183], v[112:115], v[228:231]\n\t"
        "ds_read_b64_tr_b16 v[148:149], v21 offset:32768\n\t"
        "ds_read_b32 v121, v127 offset:4\n\t"
        "ds_read_b32 v24, v127 offset:448\n\t"
        "v_mfma_f32_16x16x32_bf16 v[244:247], v[180:183], v[116:119], v[244:247]\n\t"
        "ds_read_b64_tr_b16 v[150:151], v21 offset:33792\n\t"
        "v_add_u32 v27, v27, %2\n\t"
        "s_waitcnt lgkmcnt(14)\n\t"
        "v_mfma_f32_16x16x32_bf16 v[232:235], v[184:187], v[112:115], v[232:235]\n\t"
        "ds_read_b64_tr_b16 v[152:153], v22 offset:32768\n\t"
        "global_load_dwordx4 v[56:59], v27, s[68:69] offset:0\n\t"
        "v_mfma_f32_16x16x32_bf16 v[248:251], v[184:187], v[116:119], v[248:251]\n\t"
        "ds_read_b64_tr_b16 v[154:155], v22 offset:33792\n\t"
        "ds_add_u32 %4, v31\n\t"
        "v_mfma_f32_16x16x32_bf16 v[236:239], v[188:191], v[112:115], v[236:239]\n\t"
        "ds_read_b64_tr_b16 v[156:157], v23 offset:32768\n\t"
        "global_load_dwordx4 v[60:63], v27, s[68:69] offset:1024\n\t"
        "v_mfma_f32_16x16x32_bf16 v[252:255], v[188:191], v[116:119], v[252:255]\n\t"
        "ds_read_b64_tr_b16 v[158:159], v23 offset:33792\n\t"
        "s_waitcnt vmcnt(6) lgkmcnt(15)\n\t"
        "v_mfma_f32_16x16x32_bf16 v[192:195], v[128:131], v[32:35], v[192:195]\n\t"
        "ds_read_b64_tr_b16 v[160:161], v16 offset:40960\n\t"
        "v_mfma_f32_16x16x32_bf16 v[208:211], v[128:131], v[36:39], v[208:211]\n\t"
        "ds_read_b64_tr_b16 v[162:163], v16 offset:41984\n\t"
        "v_mfma_f32_16x16x32_bf16 v[196:199], v[132:135], v[32:35], v[196:199]\n\t"
        "ds_read_b64_tr_b16 v[164:165], v17 offset:40960\n\t"
        "v_mfma_f32_16x16x32_bf16 v[212:215], v[132:135], v[36:39], v[212:215]\n\t"
        "ds_read_b64_tr_b16 v[166:167], v17 offset:41984\n\t"
        "s_waitcnt lgkmcnt(10)\n\t"
        "v_mfma_f32_16x16x32_bf16 v[200:203], v[136:139], v[32:35], v[200:203]\n\t"
        "ds_read_b64_tr_b16 v[168:169], v18 offset:40960\n\t"
        "s_movk_i32 s61, 46\n\t"
        "v_mfma_f32_16x16x32_bf16 v[216:219], v[136:139], v[36:39], v[216:219]\n\t"
        "ds_read_b64_tr_b16 v[170:171], v18 offset:41984\n\t"
        "v_cmp_ge_u32 vcc, s61, v121\n\t"
        "v_mfma_f32_16x16x32_bf16 v[204:207], v[140:143], v[32:35], v[204:207]\n\t"
        "ds_read_b64_tr_b16 v[172:173], v19 offset:40960\n\t"
        "s_getpc_b64 s[70:71]\n\t"
        "s_cbranch_vccnz .Lspin1_%=\n\t"
        "v_mfma_f32_16x16x32_bf16 v[220:223], v[140:143], v[36:39], v[220:223]\n\t"
        "ds_read_b64_tr_b16 v[174:175], v19 offset:41984\n\t"
        "s_movk_i32 s61, 48\n\t"
        "s_waitcnt lgkmcnt(13)\n\t"
        "v_mfma_f32_16x16x32_bf16 v[224:227], v[144:147], v[32:35], v[224:227]\n\t"
        "ds_read_b64_tr_b16 v[176:177], v20 offset:40960\n\t"
        "v_cmp_eq_u32 vcc, 0, v24\n\t"
        "v_mfma_f32_16x16x32_bf16 v[240:243], v[144:147], v[36:39], v[240:243]\n\t"
        "ds_read_b64_tr_b16 v[178:179], v20 offset:41984\n\t"
        "s_getpc_b64 s[70:71]\n\t"
        "s_cbranch_vccnz .Lspin4_%=\n\t"
        "v_mfma_f32_16x16x32_bf16 v[228:231], v[148:151], v[32:35], v[228:231]\n\t"
        "ds_read_b64_tr_b16 v[180:181], v21 offset:40960\n\t"
        "ds_read_b32 v121, v127 offset:4\n\t"
        "ds_read_b32 v25, v127 offset:452\n\t"
        "v_mfma_f32_16x16x32_bf16 v[244:247], v[148:151], v[36:39], v[244:247]\n\t"
        "ds_read_b64_tr_b16 v[182:183], v21 offset:41984\n\t"
        "v_add_u32 v24, v24, %2\n\t"
        "s_waitcnt lgkmcnt(14)\n\t"
        "v_mfma_f32_16x16x32_bf16 v[232:235], v[152:155], v[32:35], v[232:235]\n\t"
        "ds_read_b64_tr_b16 v[184:185], v22 offset:40960\n\t"
        "global_load_dwordx4 v[64:67], v24, s[68:69] offset:0\n\t"
        "v_mfma_f32_16x16x32_bf16 v[248:251], v[152:155], v[36:39], v[248:251]\n\t"
        "ds_read_b64_tr_b16 v[186:187], v22 offset:41984\n\t"
        "ds_add_u32 %4, v31\n\t"
        "v_mfma_f32_16x16x32_bf16 v[236:239], v[156:159], v[32:35], v[236:239]\n\t"
        "ds_read_b64_tr_b16 v[188:189], v23 offset:40960\n\t"
        "global_load_dwordx4 v[68:71], v24, s[68:69] offset:1024\n\t"
        "v_mfma_f32_16x16x32_bf16 v[252:255], v[156:159], v[36:39], v[252:255]\n\t"
        "ds_read_b64_tr_b16 v[190:191], v23 offset:41984\n\t"
        "s_waitcnt vmcnt(6) lgkmcnt(15)\n\t"
        "v_mfma_f32_16x16x32_bf16 v[192:195], v[160:163], v[40:43], v[192:195]\n\t"
        "ds_read_b64_tr_b16 v[128:129], v16 offset:49152\n\t"
        "v_mfma_f32_16x16x32_bf16 v[208:211], v[160:163], v[44:47], v[208:211]\n\t"
        "ds_read_b64_tr_b16 v[130:131], v16 offset:50176\n\t"
        "v_mfma_f32_16x16x32_bf16 v[196:199], v[164:167], v[40:43], v[196:199]\n\t"
        "ds_read_b64_tr_b16 v[132:133], v17 offset:49152\n\t"
        "v_mfma_f32_16x16x32_bf16 v[212:215], v[164:167], v[44:47], v[212:215]\n\t"
        "ds_read_b64_tr_b16 v[134:135], v17 offset:50176\n\t"
        "s_waitcnt lgkmcnt(10)\n\t"
        "v_mfma_f32_16x16x32_bf16 v[200:203], v[168:171], v[40:43], v[200:203]\n\t"
        "ds_read_b64_tr_b16 v[136:137], v18 offset:49152\n\t"
        "s_movk_i32 s61, 47\n\t"
        "v_mfma_f32_16x16x32_bf16 v[216:219], v[168:171], v[44:47], v[216:219]\n\t"
        "ds_read_b64_tr_b16 v[138:139], v18 offset:50176\n\t"
        "v_cmp_ge_u32 vcc, s61, v121\n\t"
        "v_mfma_f32_16x16x32_bf16 v[204:207], v[172:175], v[40:43], v[204:207]\n\t"
        "ds_read_b64_tr_b16 v[140:141], v19 offset:49152\n\t"
        "s_getpc_b64 s[70:71]\n\t"
        "s_cbranch_vccnz .Lspin1_%=\n\t"
        "v_mfma_f32_16x16x32_bf16 v[220:223], v[172:175], v[44:47], v[220:223]\n\t"
        "ds_read_b64_tr_b16 v[142:143], v19 offset:50176\n\t"
        "s_movk_i32 s61, 49\n\t"
        "s_waitcnt lgkmcnt(13)\n\t"
        "v_mfma_f32_16x16x32_bf16 v[224:227], v[176:179], v[40:43], v[224:227]\n\t"
        "ds_read_b64_tr_b16 v[144:145], v20 offset:49152\n\t"
        "v_cmp_eq_u32 vcc, 0, v25\n\t"
        "v_mfma_f32_16x16x32_bf16 v[240:243], v[176:179], v[44:47], v[240:243]\n\t"
        "ds_read_b64_tr_b16 v[146:147], v20 offset:50176\n\t"
        "s_getpc_b64 s[70:71]\n\t"
        "s_cbranch_vccnz .Lspin5_%=\n\t"
        "v_mfma_f32_16x16x32_bf16 v[228:231], v[180:183], v[40:43], v[228:231]\n\t"
        "ds_read_b64_tr_b16 v[148:149], v21 offset:49152\n\t"
        "ds_read_b32 v121, v127 offset:4\n\t"
        "ds_read_b32 v26, v127 offset:456\n\t"
        "v_mfma_f32_16x16x32_bf16 v[244:247], v[180:183], v[44:47], v[244:247]\n\t"
        "ds_read_b64_tr_b16 v[150:151], v21 offset:50176\n\t"
        "v_add_u32 v25, v25, %2\n\t"
        "s_waitcnt lgkmcnt(14)\n\t"
        "v_mfma_f32_16x16x32_bf16 v[232:235], v[184:187], v[40:43], v[232:235]\n\t"
        "ds_read_b64_tr_b16 v[152:153], v22 offset:49152\n\t"
        "global_load_dwordx4 v[72:75], v25, s[68:69] offset:0\n\t"
        "v_mfma_f32_16x16x32_bf16 v[248:251], v[184:187], v[44:47], v[248:251]\n\t"
        "ds_read_b64_tr_b16 v[154:155], v22 offset:50176\n\t"
        "ds_add_u32 %4, v31\n\t"
        "v_mfma_f32_16x16x32_bf16 v[236:239], v[188:191], v[40:43], v[236:239]\n\t"
        "ds_read_b64_tr_b16 v[156:157], v23 offset:49152\n\t"
        "global_load_dwordx4 v[76:79], v25, s[68:69] offset:1024\n\t"
        "v_mfma_f32_16x16x32_bf16 v[252:255], v[188:191], v[44:47], v[252:255]\n\t"
        "ds_read_b64_tr_b16 v[158:159], v23 offset:50176\n\t"
        "s_waitcnt vmcnt(6) lgkmcnt(15)\n\t"
        "v_mfma_f32_16x16x32_bf16 v[192:195], v[128:131], v[48:51], v[192:195]\n\t"
        "ds_read_b64_tr_b16 v[160:161], v16 offset:57344\n\t"
        "v_mfma_f32_16x16x32_bf16 v[208:211], v[128:131], v[52:55], v[208:211]\n\t"
        "ds_read_b64_tr_b16 v[162:163], v16 offset:58368\n\t"
        "v_mfma_f32_16x16x32_bf16 v[196:199], v[132:135], v[48:51], v[196:199]\n\t"
        "ds_read_b64_tr_b16 v[164:165], v17 offset:57344\n\t"
        "v_mfma_f32_16x16x32_bf16 v[212:215], v[132:135], v[52:55], v[212:215]\n\t"
        "ds_read_b64_tr_b16 v[166:167], v17 offset:58368\n\t"
        "s_waitcnt lgkmcnt(10)\n\t"
        "v_mfma_f32_16x16x32_bf16 v[200:203], v[136:139], v[48:51], v[200:203]\n\t"
        "ds_read_b64_tr_b16 v[168:169], v18 offset:57344\n\t"
        "s_movk_i32 s61, 48\n\t"
        "v_mfma_f32_16x16x32_bf16 v[216:219], v[136:139], v[52:55], v[216:219]\n\t"
        "ds_read_b64_tr_b16 v[170:171], v18 offset:58368\n\t"
        "v_cmp_ge_u32 vcc, s61, v121\n\t"
        "v_mfma_f32_16x16x32_bf16 v[204:207], v[140:143], v[48:51], v[204:207]\n\t"
        "ds_read_b64_tr_b16 v[172:173], v19 offset:57344\n\t"
        "s_getpc_b64 s[70:71]\n\t"
        "s_cbranch_vccnz .Lspin1_%=\n\t"
        "v_mfma_f32_16x16x32_bf16 v[220:223], v[140:143], v[52:55], v[220:223]\n\t"
        "ds_read_b64_tr_b16 v[174:175], v19 offset:58368\n\t"
        "s_movk_i32 s61, 50\n\t"
        "s_waitcnt lgkmcnt(13)\n\t"
        "v_mfma_f32_16x16x32_bf16 v[224:227], v[144:147], v[48:51], v[224:227]\n\t"
        "ds_read_b64_tr_b16 v[176:177], v20 offset:57344\n\t"
        "v_cmp_eq_u32 vcc, 0, v26\n\t"
        "v_mfma_f32_16x16x32_bf16 v[240:243], v[144:147], v[52:55], v[240:243]\n\t"
        "ds_read_b64_tr_b16 v[178:179], v20 offset:58368\n\t"
        "s_getpc_b64 s[70:71]\n\t"
        "s_cbranch_vccnz .Lspin6_%=\n\t"
        "v_mfma_f32_16x16x32_bf16 v[228:231], v[148:151], v[48:51], v[228:231]\n\t"
        "ds_read_b64_tr_b16 v[180:181], v21 offset:57344\n\t"
        "ds_read_b32 v121, v127 offset:4\n\t"
        "ds_read_b32 v27, v127 offset:460\n\t"
        "v_mfma_f32_16x16x32_bf16 v[244:247], v[148:151], v[52:55], v[244:247]\n\t"
        "ds_read_b64_tr_b16 v[182:183], v21 offset:58368\n\t"
        "v_add_u32 v26, v26, %2\n\t"
        "s_waitcnt lgkmcnt(14)\n\t"
        "v_mfma_f32_16x16x32_bf16 v[232:235], v[152:155], v[48:51], v[232:235]\n\t"
        "ds_read_b64_tr_b16 v[184:185], v22 offset:57344\n\t"
        "global_load_dwordx4 v[80:83], v26, s[68:69] offset:0\n\t"
        "v_mfma_f32_16x16x32_bf16 v[248:251], v[152:155], v[52:55], v[248:251]\n\t"
        "ds_read_b64_tr_b16 v[186:187], v22 offset:58368\n\t"
        "ds_add_u32 %4, v31\n\t"
        "v_mfma_f32_16x16x32_bf16 v[236:239], v[156:159], v[48:51], v[236:239]\n\t"
        "ds_read_b64_tr_b16 v[188:189], v23 offset:57344\n\t"
        "global_load_dwordx4 v[84:87], v26, s[68:69] offset:1024\n\t"
        "v_mfma_f32_16x16x32_bf16 v[252:255], v[156:159], v[52:55], v[252:255]\n\t"
        "ds_read_b64_tr_b16 v[190:191], v23 offset:58368\n\t"
        "s_waitcnt vmcnt(6) lgkmcnt(15)\n\t"
        "v_mfma_f32_16x16x32_bf16 v[192:195], v[160:163], v[56:59], v[192:195]\n\t"
        "ds_read_b64_tr_b16 v[128:129], v8 offset:0\n\t"
        "v_mfma_f32_16x16x32_bf16 v[208:211], v[160:163], v[60:63], v[208:211]\n\t"
        "ds_read_b64_tr_b16 v[130:131], v8 offset:1024\n\t"
        "v_mfma_f32_16x16x32_bf16 v[196:199], v[164:167], v[56:59], v[196:199]\n\t"
        "ds_read_b64_tr_b16 v[132:133], v9 offset:0\n\t"
        "v_mfma_f32_16x16x32_bf16 v[212:215], v[164:167], v[60:63], v[212:215]\n\t"
        "ds_read_b64_tr_b16 v[134:135], v9 offset:1024\n\t"
        "s_waitcnt lgkmcnt(10)\n\t"
        "v_mfma_f32_16x16x32_bf16 v[200:203], v[168:171], v[56:59], v[200:203]\n\t"
        "ds_read_b64_tr_b16 v[136:137], v10 offset:0\n\t"
        "s_movk_i32 s61, 49\n\t"
        "v_mfma_f32_16x16x32_bf16 v[216:219], v[168:171], v[60:63], v[216:219]\n\t"
        "ds_read_b64_tr_b16 v[138:139], v10 offset:1024\n\t"
        "v_cmp_ge_u32 vcc, s61, v121\n\t"
        "v_mfma_f32_16x16x32_bf16 v[204:207], v[172:175], v[56:59], v[204:207]\n\t"
        "ds_read_b64_tr_b16 v[140:141], v11 offset:0\n\t"
        "s_getpc_b64 s[70:71]\n\t"
        "s_cbranch_vccnz .Lspin1_%=\n\t"
        "v_mfma_f32_16x16x32_bf16 v[220:223], v[172:175], v[60:63], v[220:223]\n\t"
        "ds_read_b64_tr_b16 v[142:143], v11 offset:1024\n\t"
        "s_movk_i32 s61, 51\n\t"
        "s_waitcnt lgkmcnt(13)\n\t"
        "v_mfma_f32_16x16x32_bf16 v[224:227], v[176:179], v[56:59], v[224:227]\n\t"
        "ds_read_b64_tr_b16 v[144:145], v12 offset:0\n\t"
        "v_cmp_eq_u32 vcc, 0, v27\n\t"
        "v_mfma_f32_16x16x32_bf16 v[240:243], v[176:179], v[60:63], v[240:243]\n\t"
        "ds_read_b64_tr_b16 v[146:147], v12 offset:1024\n\t"
        "s_getpc_b64 s[70:71]\n\t"
        "s_cbranch_vccnz .Lspin7_%=\n\t"
        "v_mfma_f32_16x16x32_bf16 v[228:231], v[180:183], v[56:59], v[228:231]\n\t"
        "ds_read_b64_tr_b16 v[148:149], v13 offset:0\n\t"
        "ds_read_b32 v121, v127 offset:4\n\t"
        "ds_read_b32 v24, v127 offset:464\n\t"
        "v_mfma_f32_16x16x32_bf16 v[244:247], v[180:183], v[60:63], v[244:247]\n\t"
        "ds_read_b64_tr_b16 v[150:151], v13 offset:1024\n\t"
        "v_add_u32 v27, v27, %2\n\t"
        "s_waitcnt lgkmcnt(14)\n\t"
        "v_mfma_f32_16x16x32_bf16 v[232:235], v[184:187], v[56:59], v[232:235]\n\t"
        "ds_read_b64_tr_b16 v[152:153], v14 offset:0\n\t"
        "global_load_dwordx4 v[88:91], v27, s[68:69] offset:0\n\t"
        "v_mfma_f32_16x16x32_bf16 v[248:251], v[184:187], v[60:63], v[248:251]\n\t"
        "ds_read_b64_tr_b16 v[154:155], v14 offset:1024\n\t"
        "ds_add_u32 %4, v31\n\t"
        "v_mfma_f32_16x16x32_bf16 v[236:239], v[188:191], v[56:59], v[236:239]\n\t"
        "ds_read_b64_tr_b16 v[156:157], v15 offset:0\n\t"
        "global_load_dwordx4 v[92:95], v27, s[68:69] offset:1024\n\t"
        "v_mfma_f32_16x16x32_bf16 v[252:255], v[188:191], v[60:63], v[252:255]\n\t"
        "ds_read_b64_tr_b16 v[158:159], v15 offset:1024\n\t"
        "s_waitcnt vmcnt(6) lgkmcnt(15)\n\t"
        "v_mfma_f32_16x16x32_bf16 v[192:195], v[128:131], v[64:67], v[192:195]\n\t"
        "ds_read_b64_tr_b16 v[160:161], v8 offset:8192\n\t"
        "v_mfma_f32_16x16x32_bf16 v[208:211], v[128:131], v[68:71], v[208:211]\n\t"
        "ds_read_b64_tr_b16 v[162:163], v8 offset:9216\n\t"
        "v_mfma_f32_16x16x32_bf16 v[196:199], v[132:135], v[64:67], v[196:199]\n\t"
        "ds_read_b64_tr_b16 v[164:165], v9 offset:8192\n\t"
        "v_mfma_f32_16x16x32_bf16 v[212:215], v[132:135], v[68:71], v[212:215]\n\t"
        "ds_read_b64_tr_b16 v[166:167], v9 offset:9216\n\t"
        "s_waitcnt lgkmcnt(10)\n\t"
        "v_mfma_f32_16x16x32_bf16 v[200:203], v[136:139], v[64:67], v[200:203]\n\t"
        "ds_read_b64_tr_b16 v[168:169], v10 offset:8192\n\t"
        "s_movk_i32 s61, 50\n\t"
        "v_mfma_f32_16x16x32_bf16 v[216:219], v[136:139], v[68:71], v[216:219]\n\t"
        "ds_read_b64_tr_b16 v[170:171], v10 offset:9216\n\t"
        "v_cmp_ge_u32 vcc, s61, v121\n\t"
        "v_mfma_f32_16x16x32_bf16 v[204:207], v[140:143], v[64:67], v[204:207]\n\t"
        "ds_read_b64_tr_b16 v[172:173], v11 offset:8192\n\t"
        "s_getpc_b64 s[70:71]\n\t"
        "s_cbranch_vccnz .Lspin1_%=\n\t"
        "v_mfma_f32_16x16x32_bf16 v[220:223], v[140:143], v[68:71], v[220:223]\n\t"
        "ds_read_b64_tr_b16 v[174:175], v11 offset:9216\n\t"
        "s_movk_i32 s61, 52\n\t"
        "s_waitcnt lgkmcnt(13)\n\t"
        "v_mfma_f32_16x16x32_bf16 v[224:227], v[144:147], v[64:67], v[224:227]\n\t"
        "ds_read_b64_tr_b16 v[176:177], v12 offset:8192\n\t"
        "v_cmp_eq_u32 vcc, 0, v24\n\t"
        "v_mfma_f32_16x16x32_bf16 v[240:243], v[144:147], v[68:71], v[240:243]\n\t"
        "ds_read_b64_tr_b16 v[178:179], v12 offset:9216\n\t"
        "s_getpc_b64 s[70:71]\n\t"
        "s_cbranch_vccnz .Lspin4_%=\n\t"
        "v_mfma_f32_16x16x32_bf16 v[228:231], v[148:151], v[64:67], v[228:231]\n\t"
        "ds_read_b64_tr_b16 v[180:181], v13 offset:8192\n\t"
        "ds_read_b32 v121, v127 offset:4\n\t"
        "ds_read_b32 v25, v127 offset:468\n\t"
        "v_mfma_f32_16x16x32_bf16 v[244:247], v[148:151], v[68:71], v[244:247]\n\t"
        "ds_read_b64_tr_b16 v[182:183], v13 offset:9216\n\t"
        "v_add_u32 v24, v24, %2\n\t"
        "global_load_dwordx4 v[96:99], v24, s[66:67] offset:0\n\t"
        "s_waitcnt lgkmcnt(14)\n\t"
        "v_mfma_f32_16x16x32_bf16 v[232:235], v[152:155], v[64:67], v[232:235]\n\t"
        "ds_read_b64_tr_b16 v[184:185], v14 offset:8192\n\t"
        "global_load_dwordx4 v[100:103], v24, s[66:67] offset:1024\n\t"
        "v_mfma_f32_16x16x32_bf16 v[248:251], v[152:155], v[68:71], v[248:251]\n\t"
        "ds_read_b64_tr_b16 v[186:187], v14 offset:9216\n\t"
        "global_load_dwordx4 v[104:107], v24, s[64:65] offset:0\n\t"
        "v_mfma_f32_16x16x32_bf16 v[236:239], v[156:159], v[64:67], v[236:239]\n\t"
        "ds_read_b64_tr_b16 v[188:189], v15 offset:8192\n\t"
        "global_load_dwordx4 v[108:111], v24, s[64:65] offset:1024\n\t"
        "v_mfma_f32_16x16x32_bf16 v[252:255], v[156:159], v[68:71], v[252:255]\n\t"
        "ds_read_b64_tr_b16 v[190:191], v15 offset:9216\n\t"
        "s_waitcnt vmcnt(8) lgkmcnt(14)\n\t"
        "v_mfma_f32_16x16x32_bf16 v[192:195], v[160:163], v[72:75], v[192:195]\n\t"
        "ds_read_b64_tr_b16 v[128:129], v8 offset:16384\n\t"
        "v_mfma_f32_16x16x32_bf16 v[208:211], v[160:163], v[76:79], v[208:211]\n\t"
        "ds_read_b64_tr_b16 v[130:131], v8 offset:17408\n\t"
        "v_mfma_f32_16x16x32_bf16 v[196:199], v[164:167], v[72:75], v[196:199]\n\t"
        "ds_read_b64_tr_b16 v[132:133], v9 offset:16384\n\t"
        "v_mfma_f32_16x16x32_bf16 v[212:215], v[164:167], v[76:79], v[212:215]\n\t"
        "ds_read_b64_tr_b16 v[134:135], v9 offset:17408\n\t"
        "s_waitcnt lgkmcnt(9)\n\t"
        "v_mfma_f32_16x16x32_bf16 v[200:203], v[168:171], v[72:75], v[200:203]\n\t"
        "ds_read_b64_tr_b16 v[136:137], v10 offset:16384\n\t"
        "s_movk_i32 s61, 51\n\t"
        "v_mfma_f32_16x16x32_bf16 v[216:219], v[168:171], v[76:79], v[216:219]\n\t"
        "ds_read_b64_tr_b16 v[138:139], v10 offset:17408\n\t"
        "v_cmp_ge_u32 vcc, s61, v121\n\t"
        "v_mfma_f32_16x16x32_bf16 v[204:207], v[172:175], v[72:75], v[204:207]\n\t"
        "ds_read_b64_tr_b16 v[140:141], v11 offset:16384\n\t"
        "s_getpc_b64 s[70:71]\n\t"
        "s_cbranch_vccnz .Lspin1_%=\n\t"
        "v_mfma_f32_16x16x32_bf16 v[220:223], v[172:175], v[76:79], v[220:223]\n\t"
        "ds_read_b64_tr_b16 v[142:143], v11 offset:17408\n\t"
        "s_movk_i32 s61, 53\n\t"
        "s_waitcnt lgkmcnt(12)\n\t"
        "v_mfma_f32_16x16x32_bf16 v[224:227], v[176:179], v[72:75], v[224:227]\n\t"
        "ds_read_b64_tr_b16 v[144:145], v12 offset:16384\n\t"
        "v_cmp_eq_u32 vcc, 0, v25\n\t"
        "v_mfma_f32_16x16x32_bf16 v[240:243], v[176:179], v[76:79], v[240:243]\n\t"
        "ds_read_b64_tr_b16 v[146:147], v12 offset:17408\n\t"
        "s_getpc_b64 s[70:71]\n\t"
        "s_cbranch_vccnz .Lspin5_%=\n\t"
        "v_mfma_f32_16x16x32_bf16 v[228:231], v[180:183], v[72:75], v[228:231]\n\t"
        "ds_read_b64_tr_b16 v[148:149], v13 offset:16384\n\t"
        "ds_read_b32 v121, v127 offset:4\n\t"
        "ds_read_b32 v26, v127 offset:472\n\t"
        "v_mfma_f32_16x16x32_bf16 v[244:247], v[180:183], v[76:79], v[244:247]\n\t"
        "ds_read_b64_tr_b16 v[150:151], v13 offset:17408\n\t"
        "v_add_u32 v25, v25, %2\n\t"
        "global_load_dwordx4 v[112:115], v25, s[66:67] offset:0\n\t"
        "s_waitcnt lgkmcnt(14)\n\t"
        "v_mfma_f32_16x16x32_bf16 v[232:235], v[184:187], v[72:75], v[232:235]\n\t"
        "ds_read_b64_tr_b16 v[152:153], v14 offset:16384\n\t"
        "global_load_dwordx4 v[116:119], v25, s[66:67] offset:1024\n\t"
        "v_mfma_f32_16x16x32_bf16 v[248:251], v[184:187], v[76:79], v[248:251]\n\t"
        "ds_read_b64_tr_b16 v[154:155], v14 offset:17408\n\t"
        "global_load_dwordx4 v[32:35], v25, s[64:65] offset:0\n\t"
        "v_mfma_f32_16x16x32_bf16 v[236:239], v[188:191], v[72:75], v[236:239]\n\t"
        "ds_read_b64_tr_b16 v[156:157], v15 offset:16384\n\t"
        "global_load_dwordx4 v[36:39], v25, s[64:65] offset:1024\n\t"
        "v_mfma_f32_16x16x32_bf16 v[252:255], v[188:191], v[76:79], v[252:255]\n\t"
        "ds_read_b64_tr_b16 v[158:159], v15 offset:17408\n\t"
        "s_waitcnt vmcnt(10) lgkmcnt(14)\n\t"
        "v_mfma_f32_16x16x32_bf16 v[192:195], v[128:131], v[80:83], v[192:195]\n\t"
        "ds_read_b64_tr_b16 v[160:161], v8 offset:24576\n\t"
        "v_mfma_f32_16x16x32_bf16 v[208:211], v[128:131], v[84:87], v[208:211]\n\t"
        "ds_read_b64_tr_b16 v[162:163], v8 offset:25600\n\t"
        "v_mfma_f32_16x16x32_bf16 v[196:199], v[132:135], v[80:83], v[196:199]\n\t"
        "ds_read_b64_tr_b16 v[164:165], v9 offset:24576\n\t"
        "v_mfma_f32_16x16x32_bf16 v[212:215], v[132:135], v[84:87], v[212:215]\n\t"
        "ds_read_b64_tr_b16 v[166:167], v9 offset:25600\n\t"
        "s_waitcnt lgkmcnt(9)\n\t"
        "v_mfma_f32_16x16x32_bf16 v[200:203], v[136:139], v[80:83], v[200:203]\n\t"
        "ds_read_b64_tr_b16 v[168:169], v10 offset:24576\n\t"
        "s_movk_i32 s61, 52\n\t"
        "v_mfma_f32_16x16x32_bf16 v[216:219], v[136:139], v[84:87], v[216:219]\n\t"
        "ds_read_b64_tr_b16 v[170:171], v10 offset:25600\n\t"
        "v_cmp_ge_u32 vcc, s61, v121\n\t"
        "v_mfma_f32_16x16x32_bf16 v[204:207], v[140:143], v[80:83], v[204:207]\n\t"
        "ds_read_b64_tr_b16 v[172:173], v11 offset:24576\n\t"
        "s_getpc_b64 s[70:71]\n\t"
        "s_cbranch_vccnz .Lspin1_%=\n\t"
        "v_mfma_f32_16x16x32_bf16 v[220:223], v[140:143], v[84:87], v[220:223]\n\t"
        "ds_read_b64_tr_b16 v[174:175], v11 offset:25600\n\t"
        "s_movk_i32 s61, 54\n\t"
        "s_waitcnt lgkmcnt(12)\n\t"
        "v_mfma_f32_16x16x32_bf16 v[224:227], v[144:147], v[80:83], v[224:227]\n\t"
        "ds_read_b64_tr_b16 v[176:177], v12 offset:24576\n\t"
        "v_cmp_eq_u32 vcc, 0, v26\n\t"
        "v_mfma_f32_16x16x32_bf16 v[240:243], v[144:147], v[84:87], v[240:243]\n\t"
        "ds_read_b64_tr_b16 v[178:179], v12 offset:25600\n\t"
        "s_getpc_b64 s[70:71]\n\t"
        "s_cbranch_vccnz .Lspin6_%=\n\t"
        "v_mfma_f32_16x16x32_bf16 v[228:231], v[148:151], v[80:83], v[228:231]\n\t"
        "ds_read_b64_tr_b16 v[180:181], v13 offset:24576\n\t"
        "ds_read_b32 v121, v127 offset:4\n\t"
        "ds_read_b32 v27, v127 offset:476\n\t"
        "v_mfma_f32_16x16x32_bf16 v[244:247], v[148:151], v[84:87], v[244:247]\n\t"
        "ds_read_b64_tr_b16 v[182:183], v13 offset:25600\n\t"
        "v_add_u32 v26, v26, %2\n\t"
        "global_load_dwordx4 v[40:43], v26, s[66:67] offset:0\n\t"
        "s_waitcnt lgkmcnt(14)\n\t"
        "v_mfma_f32_16x16x32_bf16 v[232:235], v[152:155], v[80:83], v[232:235]\n\t"
        "ds_read_b64_tr_b16 v[184:185], v14 offset:24576\n\t"
        "global_load_dwordx4 v[44:47], v26, s[66:67] offset:1024\n\t"
        "v_mfma_f32_16x16x32_bf16 v[248:251], v[152:155], v[84:87], v[248:251]\n\t"
        "ds_read_b64_tr_b16 v[186:187], v14 offset:25600\n\t"
        "global_load_dwordx4 v[48:51], v26, s[64:65] offset:0\n\t"
        "v_mfma_f32_16x16x32_bf16 v[236:239], v[156:159], v[80:83], v[236:239]\n\t"
        "ds_read_b64_tr_b16 v[188:189], v15 offset:24576\n\t"
        "global_load_dwordx4 v[52:55], v26, s[64:65] offset:1024\n\t"
        "v_mfma_f32_16x16x32_bf16 v[252:255], v[156:159], v[84:87], v[252:255]\n\t"
        "ds_read_b64_tr_b16 v[190:191], v15 offset:25600\n\t"
        "s_waitcnt vmcnt(12) lgkmcnt(14)\n\t"
        "v_mfma_f32_16x16x32_bf16 v[192:195], v[160:163], v[88:91], v[192:195]\n\t"
        "ds_read_b64_tr_b16 v[128:129], v8 offset:32768\n\t"
        "v_mfma_f32_16x16x32_bf16 v[208:211], v[160:163], v[92:95], v[208:211]\n\t"
        "ds_read_b64_tr_b16 v[130:131], v8 offset:33792\n\t"
        "v_mfma_f32_16x16x32_bf16 v[196:199], v[164:167], v[88:91], v[196:199]\n\t"
        "ds_read_b64_tr_b16 v[132:133], v9 offset:32768\n\t"
        "v_mfma_f32_16x16x32_bf16 v[212:215], v[164:167], v[92:95], v[212:215]\n\t"
        "ds_read_b64_tr_b16 v[134:135], v9 offset:33792\n\t"
        "s_waitcnt lgkmcnt(9)\n\t"
        "v_mfma_f32_16x16x32_bf16 v[200:203], v[168:171], v[88:91], v[200:203]\n\t"
        "ds_read_b64_tr_b16 v[136:137], v10 offset:32768\n\t"
        "s_movk_i32 s61, 53\n\t"
        "v_mfma_f32_16x16x32_bf16 v[216:219], v[168:171], v[92:95], v[216:219]\n\t"
        "ds_read_b64_tr_b16 v[138:139], v10 offset:33792\n\t"
        "v_cmp_ge_u32 vcc, s61, v121\n\t"
        "v_mfma_f32_16x16x32_bf16 v[204:207], v[172:175], v[88:91], v[204:207]\n\t"
        "ds_read_b64_tr_b16 v[140:141], v11 offset:32768\n\t"
        "s_getpc_b64 s[70:71]\n\t"
        "s_cbranch_vccnz .Lspin1_%=\n\t"
        "v_mfma_f32_16x16x32_bf16 v[220:223], v[172:175], v[92:95], v[220:223]\n\t"
        "ds_read_b64_tr_b16 v[142:143], v11 offset:33792\n\t"
        "s_movk_i32 s61, 55\n\t"
        "s_waitcnt lgkmcnt(12)\n\t"
        "v_mfma_f32_16x16x32_bf16 v[224:227], v[176:179], v[88:91], v[224:227]\n\t"
        "ds_read_b64_tr_b16 v[144:145], v12 offset:32768\n\t"
        "v_cmp_eq_u32 vcc, 0, v27\n\t"
        "v_mfma_f32_16x16x32_bf16 v[240:243], v[176:179], v[92:95], v[240:243]\n\t"
        "ds_read_b64_tr_b16 v[146:147], v12 offset:33792\n\t"
        "s_getpc_b64 s[70:71]\n\t"
        "s_cbranch_vccnz .Lspin7_%=\n\t"
        "v_mfma_f32_16x16x32_bf16 v[228:231], v[180:183], v[88:91], v[228:231]\n\t"
        "ds_read_b64_tr_b16 v[148:149], v13 offset:32768\n\t"
        "ds_read_b128 v[120:123], v127\n\t"
        "ds_read_b32 v24, v127 offset:480\n\t"
        "v_mfma_f32_16x16x32_bf16 v[244:247], v[180:183], v[92:95], v[244:247]\n\t"
        "ds_read_b64_tr_b16 v[150:151], v13 offset:33792\n\t"
        "v_add_u32 v27, v27, %2\n\t"
        "global_load_dwordx4 v[56:59], v27, s[66:67] offset:0\n\t"
        "s_waitcnt lgkmcnt(14)\n\t"
        "v_mfma_f32_16x16x32_bf16 v[232:235], v[184:187], v[88:91], v[232:235]\n\t"
        "ds_read_b64_tr_b16 v[152:153], v14 offset:32768\n\t"
        "global_load_dwordx4 v[60:63], v27, s[66:67] offset:1024\n\t"
        "v_mfma_f32_16x16x32_bf16 v[248:251], v[184:187], v[92:95], v[248:251]\n\t"
        "ds_read_b64_tr_b16 v[154:155], v14 offset:33792\n\t"
        "global_load_dwordx4 v[64:67], v27, s[64:65] offset:0\n\t"
        "v_mfma_f32_16x16x32_bf16 v[236:239], v[188:191], v[88:91], v[236:239]\n\t"
        "ds_read_b64_tr_b16 v[156:157], v15 offset:32768\n\t"
        "global_load_dwordx4 v[68:71], v27, s[64:65] offset:1024\n\t"
        "v_mfma_f32_16x16x32_bf16 v[252:255], v[188:191], v[92:95], v[252:255]\n\t"
        "ds_read_b64_tr_b16 v[158:159], v15 offset:33792\n\t"
        "s_waitcnt vmcnt(14) lgkmcnt(14)\n\t"
        "v_mfma_f32_16x16x32_bf16 v[192:195], v[128:131], v[96:99], v[192:195]\n\t"
        "ds_read_b64_tr_b16 v[160:161], v8 offset:40960\n\t"
        "v_mfma_f32_16x16x32_bf16 v[208:211], v[128:131], v[100:103], v[208:211]\n\t"
        "ds_read_b64_tr_b16 v[162:163], v8 offset:41984\n\t"
        "v_mfma_f32_16x16x32_bf16 v[196:199], v[132:135], v[96:99], v[196:199]\n\t"
        "ds_read_b64_tr_b16 v[164:165], v9 offset:40960\n\t"
        "v_mfma_f32_16x16x32_bf16 v[212:215], v[132:135], v[100:103], v[212:215]\n\t"
        "ds_read_b64_tr_b16 v[166:167], v9 offset:41984\n\t"
        "s_waitcnt lgkmcnt(9)\n\t"
        "v_mfma_f32_16x16x32_bf16 v[200:203], v[136:139], v[96:99], v[200:203]\n\t"
        "ds_read_b64_tr_b16 v[168:169], v10 offset:40960\n\t"
        "s_movk_i32 s61, 54\n\t"
        "v_cmp_ge_u32 vcc, s61, v121\n\t"
        "s_getpc_b64 s[70:71]\n\t"
        "s_cbranch_vccnz .Lspin1_%=\n\t"
        "v_mfma_f32_16x16x32_bf16 v[216:219], v[136:139], v[100:103], v[216:219]\n\t"
        "ds_read_b64_tr_b16 v[170:171], v10 offset:41984\n\t"
        "s_movk_i32 s61, 0\n\t"
        "v_cmp_ge_u32 vcc, s61, v122\n\t"
        "s_getpc_b64 s[70:71]\n\t"
        "s_cbranch_vccnz .Lspin2_%=\n\t"
        "v_mfma_f32_16x16x32_bf16 v[204:207], v[140:143], v[96:99], v[204:207]\n\t"
        "ds_read_b64_tr_b16 v[172:173], v11 offset:40960\n\t"
        "v_readfirstlane_b32 s60, v122\n\t"
        "s_and_b32 s60, s60, 2\n\t"
        "s_cbranch_scc1 .Lxs_%=\n\t"
        "global_store_dwordx4 %2, v[224:227], s[92:93] offset:-4096 sc0 sc1\n\t"
        "global_store_dwordx4 %2, v[228:231], s[92:93] offset:-3072 sc0 sc1\n\t"
        "global_store_dwordx4 %2, v[232:235], s[92:93] offset:-2048 sc0 sc1\n\t"
        "global_store_dwordx4 %2, v[236:239], s[92:93] offset:-1024 sc0 sc1\n\t"
        "global_store_dwordx4 %2, v[240:243], s[92:93] offset:0 sc0 sc1\n\t"
        "global_store_dwordx4 %2, v[244:247], s[92:93] offset:1024 sc0 sc1\n\t"
        "global_store_dwordx4 %2, v[248:251], s[92:93] offset:2048 sc0 sc1\n\t"
        "global_store_dwordx4 %2, v[252:255], s[92:93] offset:3072 sc0 sc1\n\t"
        "s_branch .Lxe_%=\n\t"
        ".Lxs_%=:\n\t"
        "global_store_dwordx4 %2, v[224:227], s[92:93] offset:-4096 sc1\n\t"
        "global_store_dwordx4 %2, v[228:231], s[92:93] offset:-3072 sc1\n\t"
        "global_store_dwordx4 %2, v[232:235], s[92:93] offset:-2048 sc1\n\t"
        "global_store_dwordx4 %2, v[236:239], s[92:93] offset:-1024 sc1\n\t"
        "global_store_dwordx4 %2, v[240:243], s[92:93] offset:0 sc1\n\t"
        "global_store_dwordx4 %2, v[244:247], s[92:93] offset:1024 sc1\n\t"
        "global_store_dwordx4 %2, v[248:251], s[92:93] offset:2048 sc1\n\t"
        "global_store_dwordx4 %2, v[252:255], s[92:93] offset:3072 sc1\n\t"
        ".Lxe_%=:\n\t"
        "v_mfma_f32_16x16x32_bf16 v[220:223], v[140:143], v[100:103], v[220:223]\n\t"
        "ds_read_b64_tr_b16 v[174:175], v11 offset:41984\n\t"
        "s_movk_i32 s61, 56\n\t"
        "s_waitcnt vmcnt(20) lgkmcnt(12)\n\t"
        "v_mfma_f32_16x16x32_bf16 v[192:195], v[144:147], v[104:107], v[192:195]\n\t"
        "ds_read_b64_tr_b16 v[176:177], v12 offset:40960\n\t"
        "v_cmp_eq_u32 vcc, 0, v24\n\t"
        "v_mfma_f32_16x16x32_bf16 v[208:211], v[144:147], v[108:111], v[208:211]\n\t"
        "ds_read_b64_tr_b16 v[178:179], v12 offset:41984\n\t"
        "s_getpc_b64 s[70:71]\n\t"
        "s_cbranch_vccnz .Lspin4_%=\n\t"
        "v_mfma_f32_16x16x32_bf16 v[196:199], v[148:151], v[104:107], v[196:199]\n\t"
        "ds_read_b64_tr_b16 v[180:181], v13 offset:40960\n\t"
        "ds_read_b128 v[120:123], v127\n\t"
        "v_mfma_f32_16x16x32_bf16 v[212:215], v[148:151], v[108:111], v[212:215]\n\t"
        "ds_read_b64_tr_b16 v[182:183], v13 offset:41984\n\t"
        "v_add_u32 v24, v24, %2\n\t"
        "global_load_dwordx4 v[72:75], v24, s[66:67] offset:0\n\t"
        "s_waitcnt lgkmcnt(13)\n\t"
        "v_mfma_f32_16x16x32_bf16 v[200:203], v[152:155], v[104:107], v[200:203]\n\t"
        "ds_read_b64_tr_b16 v[184:185], v14 offset:40960\n\t"
        "global_load_dwordx4 v[76:79], v24, s[66:67] offset:1024\n\t"
        "v_mfma_f32_16x16x32_bf16 v[216:219], v[152:155], v[108:111], v[216:219]\n\t"
        "ds_read_b64_tr_b16 v[186:187], v14 offset:41984\n\t"
        "global_load_dwordx4 v[80:83], v24, s[64:65] offset:0\n\t"
        "v_mfma_f32_16x16x32_bf16 v[204:207], v[156:159], v[104:107], v[204:207]\n\t"
        "ds_read_b64_tr_b16 v[188:189], v15 offset:40960\n\t"
        "global_load_dwordx4 v[84:87], v24, s[64:65] offset:1024\n\t"
        "v_mfma_f32_16x16x32_bf16 v[220:223], v[156:159], v[108:111], v[220:223]\n\t"
        "ds_read_b64_tr_b16 v[190:191], v15 offset:41984\n\t"
        "s_waitcnt vmcnt(22) lgkmcnt(13)\n\t"
        "v_mfma_f32_16x16x32_bf16 v[192:195], v[160:163], v[112:115], v[192:195]\n\t"
        "ds_read_b64_tr_b16 v[128:129], v8 offset:49152\n\t"
        "v_mfma_f32_16x16x32_bf16 v[208:211], v[160:163], v[116:119], v[208:211]\n\t"
        "ds_read_b64_tr_b16 v[130:131], v8 offset:50176\n\t"
        "v_mfma_f32_16x16x32_bf16 v[196:199], v[164:167], v[112:115], v[196:199]\n\t"
        "ds_read_b64_tr_b16 v[132:133], v9 offset:49152\n\t"
        "v_mfma_f32_16x16x32_bf16 v[212:215], v[164:167], v[116:119], v[212:215]\n\t"
        "ds_read_b64_tr_b16 v[134:135], v9 offset:50176\n\t"
        "s_waitcnt lgkmcnt(9)\n\t"
        "v_mfma_f32_16x16x32_bf16 v[200:203], v[168:171], v[112:115], v[200:203]\n\t"
        "ds_read_b64_tr_b16 v[136:137], v10 offset:49152\n\t"
        "s_movk_i32 s61, 55\n\t"
        "v_mfma_f32_16x16x32_bf16 v[216:219], v[168:171], v[116:119], v[216:219]\n\t"
        "ds_read_b64_tr_b16 v[138:139], v10 offset:50176\n\t"
        "v_cmp_ge_u32 vcc, s61, v121\n\t"
        "v_mfma_f32_16x16x32_bf16 v[204:207], v[172:175], v[112:115], v[204:207]\n\t"
        "ds_read_b64_tr_b16 v[140:141], v11 offset:49152\n\t"
        "s_getpc_b64 s[70:71]\n\t"
        "s_cbranch_vccnz .Lspin1_%=\n\t"
        "v_mfma_f32_16x16x32_bf16 v[220:223], v[172:175], v[116:119], v[220:223]\n\t"
        "ds_read_b64_tr_b16 v[142:143], v11 offset:50176\n\t"
        "ds_read_b32 v121, v127 offset:4\n\t"
        "ds_read_b32 v25, v127 offset:484\n\t"
        "ds_read_b32 v26, v127 offset:488\n\t"
        "s_waitcnt vmcnt(20) lgkmcnt(15)\n\t"
        "v_mfma_f32_16x16x32_bf16 v[192:195], v[176:179], v[32:35], v[192:195]\n\t"
        "ds_read_b64_tr_b16 v[144:145], v12 offset:49152\n\t"
        "v_mfma_f32_16x16x32_bf16 v[208:211], v[176:179], v[36:39], v[208:211]\n\t"
        "ds_read_b64_tr_b16 v[146:147], v12 offset:50176\n\t"
        "v_mfma_f32_16x16x32_bf16 v[196:199], v[180:183], v[32:35], v[196:199]\n\t"
        "ds_read_b64_tr_b16 v[148:149], v13 offset:49152\n\t"
        "s_waitcnt vmcnt(4)\n\t"
        "v_mfma_f32_16x16x32_bf16 v[212:215], v[180:183], v[36:39], v[212:215]\n\t"
        "ds_read_b64_tr_b16 v[150:151], v13 offset:50176\n\t"
        "v_mov_b32 v124, 0\n\t"
        "s_mov_b64 exec, 1\n\t"
        "global_store_dword v124, v123, s[96:97] sc0 sc1\n\t"
        "s_mov_b64 exec, -1\n\t"
        "s_waitcnt lgkmcnt(15)\n\t"
        "v_mfma_f32_16x16x32_bf16 v[200:203], v[184:187], v[32:35], v[200:203]\n\t"
        "ds_read_b64_tr_b16 v[152:153], v14 offset:49152\n\t"
        "v_mfma_f32_16x16x32_bf16 v[216:219], v[184:187], v[36:39], v[216:219]\n\t"
        "ds_read_b64_tr_b16 v[154:155], v14 offset:50176\n\t"
        "v_mfma_f32_16x16x32_bf16 v[204:207], v[188:191], v[32:35], v[204:207]\n\t"
        "ds_read_b64_tr_b16 v[156:157], v15 offset:49152\n\t"
        "v_mfma_f32_16x16x32_bf16 v[220:223], v[188:191], v[36:39], v[220:223]\n\t"
        "ds_read_b64_tr_b16 v[158:159], v15 offset:50176\n\t"
        "s_waitcnt vmcnt(19) lgkmcnt(15)\n\t"
        "v_mfma_f32_16x16x32_bf16 v[192:195], v[128:131], v[40:43], v[192:195]\n\t"
        "ds_read_b64_tr_b16 v[160:161], v8 offset:57344\n\t"
        "v_mfma_f32_16x16x32_bf16 v[208:211], v[128:131], v[44:47], v[208:211]\n\t"
        "ds_read_b64_tr_b16 v[162:163], v8 offset:58368\n\t"
        "v_mfma_f32_16x16x32_bf16 v[196:199], v[132:135], v[40:43], v[196:199]\n\t"
        "ds_read_b64_tr_b16 v[164:165], v9 offset:57344\n\t"
        "v_mfma_f32_16x16x32_bf16 v[212:215], v[132:135], v[44:47], v[212:215]\n\t"
        "ds_read_b64_tr_b16 v[166:167], v9 offset:58368\n\t"
        "s_waitcnt lgkmcnt(12)\n\t"
        "v_mfma_f32_16x16x32_bf16 v[200:203], v[136:139], v[40:43], v[200:203]\n\t"
        "ds_read_b64_tr_b16 v[168:169], v10 offset:57344\n\t"
        "s_movk_i32 s61, 56\n\t"
        "v_cmp_ge_u32 vcc, s61, v121\n\t"
        "v_mfma_f32_16x16x32_bf16 v[216:219], v[136:139], v[44:47], v[216:219]\n\t"
        "ds_read_b64_tr_b16 v[170:171], v10 offset:58368\n\t"
        "s_getpc_b64 s[70:71]\n\t"
        "s_cbranch_vccnz .Lspin1_%=\n\t"
        "s_movk_i32 s61, 57\n\t"
        "v_mfma_f32_16x16x32_bf16 v[204:207], v[140:143], v[40:43], v[204:207]\n\t"
        "ds_read_b64_tr_b16 v[172:173], v11 offset:57344\n\t"
        "v_cmp_eq_u32 vcc, 0, v25\n\t"
        "s_getpc_b64 s[70:71]\n\t"
        "s_cbranch_vccnz .Lspin5_%=\n\t"
        "v_mfma_f32_16x16x32_bf16 v[220:223], v[140:143], v[44:47], v[220:223]\n\t"
        "ds_read_b64_tr_b16 v[174:175], v11 offset:58368\n\t"
        "v_add_u32 v25, v25, %2\n\t"
        "global_load_dwordx4 v[88:91], v25, s[66:67] offset:0\n\t"
        "s_waitcnt vmcnt(18) lgkmcnt(12)\n\t"
        "v_mfma_f32_16x16x32_bf16 v[192:195], v[144:147], v[48:51], v[192:195]\n\t"
        "ds_read_b64_tr_b16 v[176:177], v12 offset:57344\n\t"
        "global_load_dwordx4 v[92:95], v25, s[66:67] offset:1024\n\t"
        "global_load_dwordx4 v[96:99], v25, s[64:65] offset:0\n\t"
        "v_mfma_f32_16x16x32_bf16 v[208:211], v[144:147], v[52:55], v[208:211]\n\t"
        "ds_read_b64_tr_b16 v[178:179], v12 offset:58368\n\t"
        "global_load_dwordx4 v[100:103], v25, s[64:65] offset:1024\n\t"
        "s_movk_i32 s61, 58\n\t"
        "v_mfma_f32_16x16x32_bf16 v[196:199], v[148:151], v[48:51], v[196:199]\n\t"
        "ds_read_b64_tr_b16 v[180:181], v13 offset:57344\n\t"
        "v_cmp_eq_u32 vcc, 0, v26\n\t"
        "s_getpc_b64 s[70:71]\n\t"
        "s_cbranch_vccnz .Lspin6_%=\n\t"
        "v_mfma_f32_16x16x32_bf16 v[212:215], v[148:151], v[52:55], v[212:215]\n\t"
        "ds_read_b64_tr_b16 v[182:183], v13 offset:58368\n\t"
        "ds_read_b32 v121, v127 offset:4\n\t"
        "ds_read_b32 v27, v127 offset:492\n\t"
        "v_add_u32 v26, v26, %2\n\t"
        "s_waitcnt lgkmcnt(14)\n\t"
        "v_mfma_f32_16x16x32_bf16 v[200:203], v[152:155], v[48:51], v[200:203]\n\t"
        "ds_read_b64_tr_b16 v[184:185], v14 offset:57344\n\t"
        "global_load_dwordx4 v[104:107], v26, s[66:67] offset:0\n\t"
        "global_load_dwordx4 v[108:111], v26, s[66:67] offset:1024\n\t"
        "v_mfma_f32_16x16x32_bf16 v[216:219], v[152:155], v[52:55], v[216:219]\n\t"
        "ds_read_b64_tr_b16 v[186:187], v14 offset:58368\n\t"
        "global_load_dwordx4 v[112:115], v26, s[64:65] offset:0\n\t"
        "v_mfma_f32_16x16x32_bf16 v[204:207], v[156:159], v[48:51], v[204:207]\n\t"
        "ds_read_b64_tr_b16 v[188:189], v15 offset:57344\n\t"
        "global_load_dwordx4 v[116:119], v26, s[64:65] offset:1024\n\t"
        "v_mfma_f32_16x16x32_bf16 v[220:223], v[156:159], v[52:55], v[220:223]\n\t"
        "ds_read_b64_tr_b16 v[190:191], v15 offset:58368\n\t"
        "s_waitcnt vmcnt(23) lgkmcnt(14)\n\t"
        "v_mfma_f32_16x16x32_bf16 v[192:195], v[160:163], v[56:59], v[192:195]\n\t"
        "ds_read_b64_tr_b16 v[128:129], v16 offset:0\n\t"
        "v_mfma_f32_16x16x32_bf16 v[208:211], v[160:163], v[60:63], v[208:211]\n\t"
        "ds_read_b64_tr_b16 v[130:131], v16 offset:1024\n\t"
        "v_mfma_f32_16x16x32_bf16 v[196:199], v[164:167], v[56:59], v[196:199]\n\t"
        "ds_read_b64_tr_b16 v[132:133], v17 offset:0\n\t"
        "v_mfma_f32_16x16x32_bf16 v[212:215], v[164:167], v[60:63], v[212:215]\n\t"
        "ds_read_b64_tr_b16 v[134:135], v17 offset:1024\n\t"
        "s_waitcnt lgkmcnt(8)\n\t"
        "v_mfma_f32_16x16x32_bf16 v[200:203], v[168:171], v[56:59], v[200:203]\n\t"
        "ds_read_b64_tr_b16 v[136:137], v18 offset:0\n\t"
        "s_movk_i32 s61, 57\n\t"
        "v_mfma_f32_16x16x32_bf16 v[216:219], v[168:171], v[60:63], v[216:219]\n\t"
        "ds_read_b64_tr_b16 v[138:139], v18 offset:1024\n\t"
        "v_cmp_ge_u32 vcc, s61, v121\n\t"
        "v_mfma_f32_16x16x32_bf16 v[204:207], v[172:175], v[56:59], v[204:207]\n\t"
        "ds_read_b64_tr_b16 v[140:141], v19 offset:0\n\t"
        "s_getpc_b64 s[70:71]\n\t"
        "s_cbranch_vccnz .Lspin1_%=\n\t"
        "v_mfma_f32_16x16x32_bf16 v[220:223], v[172:175], v[60:63], v[220:223]\n\t"
        "ds_read_b64_tr_b16 v[142:143], v19 offset:1024\n\t"
        "s_movk_i32 s61, 59\n\t"
        "s_waitcnt vmcnt(21) lgkmcnt(14)\n\t"
        "v_mfma_f32_16x16x32_bf16 v[192:195], v[176:179], v[64:67], v[192:195]\n\t"
        "ds_read_b64_tr_b16 v[144:145], v20 offset:0\n\t"
        "v_cmp_eq_u32 vcc, 0, v27\n\t"
        "v_mfma_f32_16x16x32_bf16 v[208:211], v[176:179], v[68:71], v[208:211]\n\t"
        "ds_read_b64_tr_b16 v[146:147], v20 offset:1024\n\t"
        "s_getpc_b64 s[70:71]\n\t"
        "s_cbranch_vccnz .Lspin7_%=\n\t"
        "v_mfma_f32_16x16x32_bf16 v[196:199], v[180:183], v[64:67], v[196:199]\n\t"
        "ds_read_b64_tr_b16 v[148:149], v21 offset:0\n\t"
        "ds_read_b32 v121, v127 offset:4\n\t"
        "ds_read_b32 v24, v127 offset:496\n\t"
        "v_mfma_f32_16x16x32_bf16 v[212:215], v[180:183], v[68:71], v[212:215]\n\t"
        "ds_read_b64_tr_b16 v[150:151], v21 offset:1024\n\t"
        "v_add_u32 v27, v27, %2\n\t"
        "global_load_dwordx4 v[32:35], v27, s[66:67] offset:0\n\t"
        "s_waitcnt lgkmcnt(14)\n\t"
        "v_mfma_f32_16x16x32_bf16 v[200:203], v[184:187], v[64:67], v[200:203]\n\t"
        "ds_read_b64_tr_b16 v[152:153], v22 offset:0\n\t"
        "global_load_dwordx4 v[36:39], v27, s[66:67] offset:1024\n\t"
        "v_mfma_f32_16x16x32_bf16 v[216:219], v[184:187], v[68:71], v[216:219]\n\t"
        "ds_read_b64_tr_b16 v[154:155], v22 offset:1024\n\t"
        "global_load_dwordx4 v[40:43], v27, s[64:65] offset:0\n\t"
        "v_mfma_f32_16x16x32_bf16 v[204:207], v[188:191], v[64:67], v[204:207]\n\t"
        "ds_read_b64_tr_b16 v[156:157], v23 offset:0\n\t"
        "global_load_dwordx4 v[44:47], v27, s[64:65] offset:1024\n\t"
        "v_mfma_f32_16x16x32_bf16 v[220:223], v[188:191], v[68:71], v[220:223]\n\t"
        "ds_read_b64_tr_b16 v[158:159], v23 offset:1024\n\t"
        "s_waitcnt vmcnt(15) lgkmcnt(14)\n\t"
        "v_mfma_f32_16x16x32_bf16 v[192:195], v[128:131], v[72:75], v[192:195]\n\t"
        "ds_read_b64_tr_b16 v[160:161], v16 offset:8192\n\t"
        "v_mfma_f32_16x16x32_bf16 v[208:211], v[128:131], v[76:79], v[208:211]\n\t"
        "ds_read_b64_tr_b16 v[162:163], v16 offset:9216\n\t"
        "v_mfma_f32_16x16x32_bf16 v[196:199], v[132:135], v[72:75], v[196:199]\n\t"
        "ds_read_b64_tr_b16 v[164:165], v17 offset:8192\n\t"
        "v_mfma_f32_16x16x32_bf16 v[212:215], v[132:135], v[76:79], v[212:215]\n\t"
        "ds_read_b64_tr_b16 v[166:167], v17 offset:9216\n\t"
        "s_waitcnt lgkmcnt(9)\n\t"
        "v_mfma_f32_16x16x32_bf16 v[200:203], v[136:139], v[72:75], v[200:203]\n\t"
        "ds_read_b64_tr_b16 v[168:169], v18 offset:8192\n\t"
        "s_movk_i32 s61, 58\n\t"
        "v_mfma_f32_16x16x32_bf16 v[216:219], v[136:139], v[76:79], v[216:219]\n\t"
        "ds_read_b64_tr_b16 v[170:171], v18 offset:9216\n\t"
        "v_cmp_ge_u32 vcc, s61, v121\n\t"
        "v_mfma_f32_16x16x32_bf16 v[204:207], v[140:143], v[72:75], v[204:207]\n\t"
        "ds_read_b64_tr_b16 v[172:173], v19 offset:8192\n\t"
        "s_getpc_b64 s[70:71]\n\t"
        "s_cbranch_vccnz .Lspin1_%=\n\t"
        "v_mfma_f32_16x16x32_bf16 v[220:223], v[140:143], v[76:79], v[220:223]\n\t"
        "ds_read_b64_tr_b16 v[174:175], v19 offset:9216\n\t"
        "s_movk_i32 s61, 60\n\t"
        "s_waitcnt vmcnt(13) lgkmcnt(12)\n\t"
        "v_mfma_f32_16x16x32_bf16 v[192:195], v[144:147], v[80:83], v[192:195]\n\t"
        "ds_read_b64_tr_b16 v[176:177], v20 offset:8192\n\t"
        "v_cmp_eq_u32 vcc, 0, v24\n\t"
        "v_mfma_f32_16x16x32_bf16 v[208:211], v[144:147], v[84:87], v[208:211]\n\t"
        "ds_read_b64_tr_b16 v[178:179], v20 offset:9216\n\t"
        "s_getpc_b64 s[70:71]\n\t"
        "s_cbranch_vccnz .Lspin4_%=\n\t"
        "v_mfma_f32_16x16x32_bf16 v[196:199], v[148:151], v[80:83], v[196:199]\n\t"
        "ds_read_b64_tr_b16 v[180:181], v21 offset:8192\n\t"
        "ds_read_b32 v121, v127 offset:4\n\t"
        "ds_read_b32 v25, v127 offset:500\n\t"
        "v_mfma_f32_16x16x32_bf16 v[212:215], v[148:151], v[84:87], v[212:215]\n\t"
        "ds_read_b64_tr_b16 v[182:183], v21 offset:9216\n\t"
        "v_add_u32 v24, v24, %2\n\t"
        "global_load_dwordx4 v[48:51], v24, s[66:67] offset:0\n\t"
        "s_waitcnt lgkmcnt(14)\n\t"
        "v_mfma_f32_16x16x32_bf16 v[200:203], v[152:155], v[80:83], v[200:203]\n\t"
        "ds_read_b64_tr_b16 v[184:185], v22 offset:8192\n\t"
        "global_load_dwordx4 v[52:55], v24, s[66:67] offset:1024\n\t"
        "v_mfma_f32_16x16x32_bf16 v[216:219], v[152:155], v[84:87], v[216:219]\n\t"
        "ds_read_b64_tr_b16 v[186:187], v22 offset:9216\n\t"
        "global_load_dwordx4 v[56:59], v24, s[64:65] offset:0\n\t"
        "v_mfma_f32_16x16x32_bf16 v[204:207], v[156:159], v[80:83], v[204:207]\n\t"
        "ds_read_b64_tr_b16 v[188:189], v23 offset:8192\n\t"
        "global_load_dwordx4 v[60:63], v24, s[64:65] offset:1024\n\t"
        "v_mfma_f32_16x16x32_bf16 v[220:223], v[156:159], v[84:87], v[220:223]\n\t"
        "ds_read_b64_tr_b16 v[190:191], v23 offset:9216\n\t"
        "s_waitcnt vmcnt(14) lgkmcnt(14)\n\t"
        "v_mfma_f32_16x16x32_bf16 v[192:195], v[160:163], v[88:91], v[192:195]\n\t"
        "ds_read_b64_tr_b16 v[128:129], v16 offset:16384\n\t"
        "v_mfma_f32_16x16x32_bf16 v[208:211], v[160:163], v[92:95], v[208:211]\n\t"
        "ds_read_b64_tr_b16 v[130:131], v16 offset:17408\n\t"
        "v_mfma_f32_16x16x32_bf16 v[196:199], v[164:167], v[88:91], v[196:199]\n\t"
        "ds_read_b64_tr_b16 v[132:133], v17 offset:16384\n\t"
        "v_mfma_f32_16x16x32_bf16 v[212:215], v[164:167], v[92:95], v[212:215]\n\t"
        "ds_read_b64_tr_b16 v[134:135], v17 offset:17408\n\t"
        "s_waitcnt lgkmcnt(9)\n\t"
        "v_mfma_f32_16x16x32_bf16 v[200:203], v[168:171], v[88:91], v[200:203]\n\t"
        "ds_read_b64_tr_b16 v[136:137], v18 offset:16384\n\t"
        "s_movk_i32 s61, 59\n\t"
        "v_mfma_f32_16x16x32_bf16 v[216:219], v[168:171], v[92:95], v[216:219]\n\t"
        "ds_read_b64_tr_b16 v[138:139], v18 offset:17408\n\t"
        "v_cmp_ge_u32 vcc, s61, v121\n\t"
        "v_mfma_f32_16x16x32_bf16 v[204:207], v[172:175], v[88:91], v[204:207]\n\t"
        "ds_read_b64_tr_b16 v[140:141], v19 offset:16384\n\t"
        "s_getpc_b64 s[70:71]\n\t"
        "s_cbranch_vccnz .Lspin1_%=\n\t"
        "v_mfma_f32_16x16x32_bf16 v[220:223], v[172:175], v[92:95], v[220:223]\n\t"
        "ds_read_b64_tr_b16 v[142:143], v19 offset:17408\n\t"
        "s_movk_i32 s61, 61\n\t"
        "s_waitcnt vmcnt(12) lgkmcnt(12)\n\t"
        "v_mfma_f32_16x16x32_bf16 v[192:195], v[176:179], v[96:99], v[192:195]\n\t"
        "ds_read_b64_tr_b16 v[144:145], v20 offset:16384\n\t"
        "v_cmp_eq_u32 vcc, 0, v25\n\t"
        "v_mfma_f32_16x16x32_bf16 v[208:211], v[176:179], v[100:103], v[208:211]\n\t"
        "ds_read_b64_tr_b16 v[146:147], v20 offset:17408\n\t"
        "s_getpc_b64 s[70:71]\n\t"
        "s_cbranch_vccnz .Lspin5_%=\n\t"
        "v_mfma_f32_16x16x32_bf16 v[196:199], v[180:183], v[96:99], v[196:199]\n\t"
        "ds_read_b64_tr_b16 v[148:149], v21 offset:16384\n\t"
        "ds_read_b32 v121, v127 offset:4\n\t"
        "ds_read_b32 v26, v127 offset:504\n\t"
        "v_mfma_f32_16x16x32_bf16 v[212:215], v[180:183], v[100:103], v[212:215]\n\t"
        "ds_read_b64_tr_b16 v[150:151], v21 offset:17408\n\t"
        "v_add_u32 v25, v25, %2\n\t"
        "global_load_dwordx4 v[64:67], v25, s[66:67] offset:0\n\t"
        "s_waitcnt lgkmcnt(14)\n\t"
        "v_mfma_f32_16x16x32_bf16 v[200:203], v[184:187], v[96:99], v[200:203]\n\t"
        "ds_read_b64_tr_b16 v[152:153], v22 offset:16384\n\t"
        "global_load_dwordx4 v[68:71], v25, s[66:67] offset:1024\n\t"
        "v_mfma_f32_16x16x32_bf16 v[216:219], v[184:187], v[100:103], v[216:219]\n\t"
        "ds_read_b64_tr_b16 v[154:155], v22 offset:17408\n\t"
        "global_load_dwordx4 v[72:75], v25, s[64:65] offset:0\n\t"
        "v_mfma_f32_16x16x32_bf16 v[204:207], v[188:191], v[96:99], v[204:207]\n\t"
        "ds_read_b64_tr_b16 v[156:157], v23 offset:16384\n\t"
        "global_load_dwordx4 v[76:79], v25, s[64:65] offset:1024\n\t"
        "v_mfma_f32_16x16x32_bf16 v[220:223], v[188:191], v[100:103], v[220:223]\n\t"
        "ds_read_b64_tr_b16 v[158:159], v23 offset:17408\n\t"
        "s_waitcnt vmcnt(14) lgkmcnt(14)\n\t"
        "v_mfma_f32_16x16x32_bf16 v[192:195], v[128:131], v[104:107], v[192:195]\n\t"
        "ds_read_b64_tr_b16 v[160:161], v16 offset:24576\n\t"
        "v_mfma_f32_16x16x32_bf16 v[208:211], v[128:131], v[108:111], v[208:211]\n\t"
        "ds_read_b64_tr_b16 v[162:163], v16 offset:25600\n\t"
        "v_mfma_f32_16x16x32_bf16 v[196:199], v[132:135], v[104:107], v[196:199]\n\t"
        "ds_read_b64_tr_b16 v[164:165], v17 offset:24576\n\t"
        "v_mfma_f32_16x16x32_bf16 v[212:215], v[132:135], v[108:111], v[212:215]\n\t"
        "ds_read_b64_tr_b16 v[166:167], v17 offset:25600\n\t"
        "s_waitcnt lgkmcnt(9)\n\t"
        "v_mfma_f32_16x16x32_bf16 v[200:203], v[136:139], v[104:107], v[200:203]\n\t"
        "ds_read_b64_tr_b16 v[168:169], v18 offset:24576\n\t"
        "s_movk_i32 s61, 60\n\t"
        "v_mfma_f32_16x16x32_bf16 v[216:219], v[136:139], v[108:111], v[216:219]\n\t"
        "ds_read_b64_tr_b16 v[170:171], v18 offset:25600\n\t"
        "v_cmp_ge_u32 vcc, s61, v121\n\t"
        "v_mfma_f32_16x16x32_bf16 v[204:207], v[140:143], v[104:107], v[204:207]\n\t"
        "ds_read_b64_tr_b16 v[172:173], v19 offset:24576\n\t"
        "s_getpc_b64 s[70:71]\n\t"
        "s_cbranch_vccnz .Lspin1_%=\n\t"
        "v_mfma_f32_16x16x32_bf16 v[220:223], v[140:143], v[108:111], v[220:223]\n\t"
        "ds_read_b64_tr_b16 v[174:175], v19 offset:25600\n\t"
        "s_movk_i32 s61, 62\n\t"
        "s_waitcnt vmcnt(12) lgkmcnt(12)\n\t"
        "v_mfma_f32_16x16x32_bf16 v[192:195], v[144:147], v[112:115], v[192:195]\n\t"
        "ds_read_b64_tr_b16 v[176:177], v20 offset:24576\n\t"
        "v_cmp_eq_u32 vcc, 0, v26\n\t"
        "v_mfma_f32_16x16x32_bf16 v[208:211], v[144:147], v[116:119], v[208:211]\n\t"
        "ds_read_b64_tr_b16 v[178:179], v20 offset:25600\n\t"
        "s_getpc_b64 s[70:71]\n\t"
        "s_cbranch_vccnz .Lspin6_%=\n\t"
        "v_mfma_f32_16x16x32_bf16 v[196:199], v[148:151], v[112:115], v[196:199]\n\t"
        "ds_read_b64_tr_b16 v[180:181], v21 offset:24576\n\t"
        "ds_read_b32 v121, v127 offset:4\n\t"
        "ds_read_b32 v27, v127 offset:508\n\t"
        "v_mfma_f32_16x16x32_bf16 v[212:215], v[148:151], v[116:119], v[212:215]\n\t"
        "ds_read_b64_tr_b16 v[182:183], v21 offset:25600\n\t"
        "v_add_u32 v26, v26, %2\n\t"
        "global_load_dwordx4 v[80:83], v26, s[66:67] offset:0\n\t"
        "s_waitcnt lgkmcnt(14)\n\t"
        "v_mfma_f32_16x16x32_bf16 v[200:203], v[152:155], v[112:115], v[200:203]\n\t"
        "ds_read_b64_tr_b16 v[184:185], v22 offset:24576\n\t"
        "global_load_dwordx4 v[84:87], v26, s[66:67] offset:1024\n\t"
        "v_mfma_f32_16x16x32_bf16 v[216:219], v[152:155], v[116:119], v[216:219]\n\t"
        "ds_read_b64_tr_b16 v[186:187], v22 offset:25600\n\t"
        "global_load_dwordx4 v[88:91], v26, s[64:65] offset:0\n\t"
        "v_mfma_f32_16x16x32_bf16 v[204:207], v[156:159], v[112:115], v[204:207]\n\t"
        "ds_read_b64_tr_b16 v[188:189], v23 offset:24576\n\t"
        "global_load_dwordx4 v[92:95], v26, s[64:65] offset:1024\n\t"
        "v_mfma_f32_16x16x32_bf16 v[220:223], v[156:159], v[116:119], v[220:223]\n\t"
        "ds_read_b64_tr_b16 v[190:191], v23 offset:25600\n\t"
        "s_waitcnt vmcnt(14) lgkmcnt(14)\n\t"
        "v_mfma_f32_16x16x32_bf16 v[192:195], v[160:163], v[32:35], v[192:195]\n\t"
        "ds_read_b64_tr_b16 v[128:129], v16 offset:32768\n\t"
        "v_mfma_f32_16x16x32_bf16 v[208:211], v[160:163], v[36:39], v[208:211]\n\t"
        "ds_read_b64_tr_b16 v[130:131], v16 offset:33792\n\t"
        "v_mfma_f32_16x16x32_bf16 v[196:199], v[164:167], v[32:35], v[196:199]\n\t"
        "ds_read_b64_tr_b16 v[132:133], v17 offset:32768\n\t"
        "v_mfma_f32_16x16x32_bf16 v[212:215], v[164:167], v[36:39], v[212:215]\n\t"
        "ds_read_b64_tr_b16 v[134:135], v17 offset:33792\n\t"
        "s_waitcnt lgkmcnt(9)\n\t"
        "v_mfma_f32_16x16x32_bf16 v[200:203], v[168:171], v[32:35], v[200:203]\n\t"
        "ds_read_b64_tr_b16 v[136:137], v18 offset:32768\n\t"
        "s_movk_i32 s61, 61\n\t"
        "v_mfma_f32_16x16x32_bf16 v[216:219], v[168:171], v[36:39], v[216:219]\n\t"
        "ds_read_b64_tr_b16 v[138:139], v18 offset:33792\n\t"
        "v_cmp_ge_u32 vcc, s61, v121\n\t"
        "v_mfma_f32_16x16x32_bf16 v[204:207], v[172:175], v[32:35], v[204:207]\n\t"
        "ds_read_b64_tr_b16 v[140:141], v19 offset:32768\n\t"
        "s_getpc_b64 s[70:71]\n\t"
        "s_cbranch_vccnz .Lspin1_%=\n\t"
        "v_mfma_f32_16x16x32_bf16 v[220:223], v[172:175], v[36:39], v[220:223]\n\t"
        "ds_read_b64_tr_b16 v[142:143], v19 offset:33792\n\t"
        "s_movk_i32 s61, 63\n\t"
        "s_waitcnt vmcnt(12) lgkmcnt(12)\n\t"
        "v_mfma_f32_16x16x32_bf16 v[192:195], v[176:179], v[40:43], v[192:195]\n\t"
        "ds_read_b64_tr_b16 v[144:145], v20 offset:32768\n\t"
        "v_cmp_eq_u32 vcc, 0, v27\n\t"
        "v_mfma_f32_16x16x32_bf16 v[208:211], v[176:179], v[44:47], v[208:211]\n\t"
        "ds_read_b64_tr_b16 v[146:147], v20 offset:33792\n\t"
        "s_getpc_b64 s[70:71]\n\t"
        "s_cbranch_vccnz .Lspin7_%=\n\t"
        "v_mfma_f32_16x16x32_bf16 v[196:199], v[180:183], v[40:43], v[196:199]\n\t"
        "ds_read_b64_tr_b16 v[148:149], v21 offset:32768\n\t"
        "ds_read_b32 v121, v127 offset:4\n\t"
        "v_mfma_f32_16x16x32_bf16 v[212:215], v[180:183], v[44:47], v[212:215]\n\t"
        "ds_read_b64_tr_b16 v[150:151], v21 offset:33792\n\t"
        "v_add_u32 v27, v27, %2\n\t"
        "global_load_dwordx4 v[96:99], v27, s[66:67] offset:0\n\t"
        "s_waitcnt lgkmcnt(13)\n\t"
        "v_mfma_f32_16x16x32_bf16 v[200:203], v[184:187], v[40:43], v[200:203]\n\t"
        "ds_read_b64_tr_b16 v[152:153], v22 offset:32768\n\t"
        "global_load_dwordx4 v[100:103], v27, s[66:67] offset:1024\n\t"
        "v_mfma_f32_16x16x32_bf16 v[216:219], v[184:187], v[44:47], v[216:219]\n\t"
        "ds_read_b64_tr_b16 v[154:155], v22 offset:33792\n\t"
        "global_load_dwordx4 v[104:107], v27, s[64:65] offset:0\n\t"
        "v_mfma_f32_16x16x32_bf16 v[204:207], v[188:191], v[40:43], v[204:207]\n\t"
        "ds_read_b64_tr_b16 v[156:157], v23 offset:32768\n\t"
        "global_load_dwordx4 v[108:111], v27, s[64:65] offset:1024\n\t"
        "v_mfma_f32_16x16x32_bf16 v[220:223], v[188:191], v[44:47], v[220:223]\n\t"
        "ds_read_b64_tr_b16 v[158:159], v23 offset:33792\n\t"
        "s_waitcnt vmcnt(14) lgkmcnt(13)\n\t"
        "v_mfma_f32_16x16x32_bf16 v[192:195], v[128:131], v[48:51], v[192:195]\n\t"
        "ds_read_b64_tr_b16 v[160:161], v16 offset:40960\n\t"
        "v_mfma_f32_16x16x32_bf16 v[208:211], v[128:131], v[52:55], v[208:211]\n\t"
        "ds_read_b64_tr_b16 v[162:163], v16 offset:41984\n\t"
        "v_mfma_f32_16x16x32_bf16 v[196:199], v[132:135], v[48:51], v[196:199]\n\t"
        "ds_read_b64_tr_b16 v[164:165], v17 offset:40960\n\t"
        "v_mfma_f32_16x16x32_bf16 v[212:215], v[132:135], v[52:55], v[212:215]\n\t"
        "ds_read_b64_tr_b16 v[166:167], v17 offset:41984\n\t"
        "s_waitcnt lgkmcnt(9)\n\t"
        "v_mfma_f32_16x16x32_bf16 v[200:203], v[136:139], v[48:51], v[200:203]\n\t"
        "ds_read_b64_tr_b16 v[168:169], v18 offset:40960\n\t"
        "s_movk_i32 s61, 62\n\t"
        "v_mfma_f32_16x16x32_bf16 v[216:219], v[136:139], v[52:55], v[216:219]\n\t"
        "ds_read_b64_tr_b16 v[170:171], v18 offset:41984\n\t"
        "v_cmp_ge_u32 vcc, s61, v121\n\t"
        "v_mfma_f32_16x16x32_bf16 v[204:207], v[140:143], v[48:51], v[204:207]\n\t"
        "ds_read_b64_tr_b16 v[172:173], v19 offset:40960\n\t"
        "s_getpc_b64 s[70:71]\n\t"
        "s_cbranch_vccnz .Lspin1_%=\n\t"
        "v_mfma_f32_16x16x32_bf16 v[220:223], v[140:143], v[52:55], v[220:223]\n\t"
        "ds_read_b64_tr_b16 v[174:175], v19 offset:41984\n\t"
        "ds_read_b128 v[120:123], v127\n\t"
        "s_waitcnt vmcnt(12) lgkmcnt(13)\n\t"
        "v_mfma_f32_16x16x32_bf16 v[192:195], v[144:147], v[56:59], v[192:195]\n\t"
        "ds_read_b64_tr_b16 v[176:177], v20 offset:40960\n\t"
        "v_mfma_f32_16x16x32_bf16 v[208:211], v[144:147], v[60:63], v[208:211]\n\t"
        "ds_read_b64_tr_b16 v[178:179], v20 offset:41984\n\t"
        "v_mfma_f32_16x16x32_bf16 v[196:199], v[148:151], v[56:59], v[196:199]\n\t"
        "ds_read_b64_tr_b16 v[180:181], v21 offset:40960\n\t"
        "v_mfma_f32_16x16x32_bf16 v[212:215], v[148:151], v[60:63], v[212:215]\n\t"
        "ds_read_b64_tr_b16 v[182:183], v21 offset:41984\n\t"
        "s_waitcnt lgkmcnt(13)\n\t"
        "v_mfma_f32_16x16x32_bf16 v[200:203], v[152:155], v[56:59], v[200:203]\n\t"
        "ds_read_b64_tr_b16 v[184:185], v22 offset:40960\n\t"
        "v_mfma_f32_16x16x32_bf16 v[216:219], v[152:155], v[60:63], v[216:219]\n\t"
        "ds_read_b64_tr_b16 v[186:187], v22 offset:41984\n\t"
        "v_mfma_f32_16x16x32_bf16 v[204:207], v[156:159], v[56:59], v[204:207]\n\t"
        "ds_read_b64_tr_b16 v[188:189], v23 offset:40960\n\t"
        "v_mfma_f32_16x16x32_bf16 v[220:223], v[156:159], v[60:63], v[220:223]\n\t"
        "ds_read_b64_tr_b16 v[190:191], v23 offset:41984\n\t"
        "s_waitcnt vmcnt(10) lgkmcnt(13)\n\t"
        "v_mfma_f32_16x16x32_bf16 v[192:195], v[160:163], v[64:67], v[192:195]\n\t"
        "ds_read_b64_tr_b16 v[128:129], v16 offset:49152\n\t"
        "v_mfma_f32_16x16x32_bf16 v[208:211], v[160:163], v[68:71], v[208:211]\n\t"
        "ds_read_b64_tr_b16 v[130:131], v16 offset:50176\n\t"
        "v_mfma_f32_16x16x32_bf16 v[196:199], v[164:167], v[64:67], v[196:199]\n\t"
        "ds_read_b64_tr_b16 v[132:133], v17 offset:49152\n\t"
        "v_mfma_f32_16x16x32_bf16 v[212:215], v[164:167], v[68:71], v[212:215]\n\t"
        "ds_read_b64_tr_b16 v[134:135], v17 offset:50176\n\t"
        "s_waitcnt lgkmcnt(12)\n\t"
        "v_mfma_f32_16x16x32_bf16 v[200:203], v[168:171], v[64:67], v[200:203]\n\t"
        "ds_read_b64_tr_b16 v[136:137], v18 offset:49152\n\t"
        "s_movk_i32 s61, 63\n\t"
        "v_cmp_ge_u32 vcc, s61, v121\n\t"
        "v_mfma_f32_16x16x32_bf16 v[216:219], v[168:171], v[68:71], v[216:219]\n\t"
        "ds_read_b64_tr_b16 v[138:139], v18 offset:50176\n\t"
        "s_getpc_b64 s[70:71]\n\t"
        "s_cbranch_vccnz .Lspin1_%=\n\t"
        "s_movk_i32 s61, 3\n\t"
        "v_mfma_f32_16x16x32_bf16 v[204:207], v[172:175], v[64:67], v[204:207]\n\t"
        "ds_read_b64_tr_b16 v[140:141], v19 offset:49152\n\t"
        "v_cmp_ge_u32 vcc, s61, v122\n\t"
        "s_getpc_b64 s[70:71]\n\t"
        "s_cbranch_vccnz .Lspin2_%=\n\t"
        "v_mfma_f32_16x16x32_bf16 v[220:223], v[172:175], v[68:71], v[220:223]\n\t"
        "ds_read_b64_tr_b16 v[142:143], v19 offset:50176\n\t"
        "ds_read_b32 v121, v127 offset:4\n\t"
        "s_waitcnt vmcnt(8) lgkmcnt(13)\n\t"
        "v_mfma_f32_16x16x32_bf16 v[192:195], v[176:179], v[72:75], v[192:195]\n\t"
        "ds_read_b64_tr_b16 v[144:145], v20 offset:49152\n\t"
        "global_load_dwordx4 v[224:227], %2, s[94:95] offset:-4096\n\t"
        "global_load_dwordx4 v[228:231], %2, s[94:95] offset:-3072\n\t"
        "v_mfma_f32_16x16x32_bf16 v[208:211], v[176:179], v[76:79], v[208:211]\n\t"
        "ds_read_b64_tr_b16 v[146:147], v20 offset:50176\n\t"
        "global_load_dwordx4 v[232:235], %2, s[94:95] offset:-2048\n\t"
        "v_mfma_f32_16x16x32_bf16 v[196:199], v[180:183], v[72:75], v[196:199]\n\t"
        "ds_read_b64_tr_b16 v[148:149], v21 offset:49152\n\t"
        "global_load_dwordx4 v[236:239], %2, s[94:95] offset:-1024\n\t"
        "v_mfma_f32_16x16x32_bf16 v[212:215], v[180:183], v[76:79], v[212:215]\n\t"
        "ds_read_b64_tr_b16 v[150:151], v21 offset:50176\n\t"
        "global_load_dwordx4 v[240:243], %2, s[94:95] offset:0\n\t"
        "s_waitcnt lgkmcnt(13)\n\t"
        "v_mfma_f32_16x16x32_bf16 v[200:203], v[184:187], v[72:75], v[200:203]\n\t"
        "ds_read_b64_tr_b16 v[152:153], v22 offset:49152\n\t"
        "global_load_dwordx4 v[244:247], %2, s[94:95] offset:1024\n\t"
        "v_mfma_f32_16x16x32_bf16 v[216:219], v[184:187], v[76:79], v[216:219]\n\t"
        "ds_read_b64_tr_b16 v[154:155], v22 offset:50176\n\t"
        "global_load_dwordx4 v[248:251], %2, s[94:95] offset:2048\n\t"
        "v_mfma_f32_16x16x32_bf16 v[204:207], v[188:191], v[72:75], v[204:207]\n\t"
        "ds_read_b64_tr_b16 v[156:157], v23 offset:49152\n\t"
        "global_load_dwordx4 v[252:255], %2, s[94:95] offset:3072\n\t"
        "v_mfma_f32_16x16x32_bf16 v[220:223], v[188:191], v[76:79], v[220:223]\n\t"
        "ds_read_b64_tr_b16 v[158:159], v23 offset:50176\n\t"
        "s_waitcnt vmcnt(14) lgkmcnt(13)\n\t"
        "v_mfma_f32_16x16x32_bf16 v[192:195], v[128:131], v[80:83], v[192:195]\n\t"
        "ds_read_b64_tr_b16 v[160:161], v16 offset:57344\n\t"
        "v_mfma_f32_16x16x32_bf16 v[208:211], v[128:131], v[84:87], v[208:211]\n\t"
        "ds_read_b64_tr_b16 v[162:163], v16 offset:58368\n\t"
        "v_mfma_f32_16x16x32_bf16 v[196:199], v[132:135], v[80:83], v[196:199]\n\t"
        "ds_read_b64_tr_b16 v[164:165], v17 offset:57344\n\t"
        "v_mfma_f32_16x16x32_bf16 v[212:215], v[132:135], v[84:87], v[212:215]\n\t"
        "ds_read_b64_tr_b16 v[166:167], v17 offset:58368\n\t"
        "s_waitcnt lgkmcnt(12)\n\t"
        "v_mfma_f32_16x16x32_bf16 v[200:203], v[136:139], v[80:83], v[200:203]\n\t"
        "ds_read_b64_tr_b16 v[176:177], v20 offset:57344\n\t"
        "v_mfma_f32_16x16x32_bf16 v[216:219], v[136:139], v[84:87], v[216:219]\n\t"
        "ds_read_b64_tr_b16 v[178:179], v20 offset:58368\n\t"
        "ds_read_b32 v121, v127 offset:4\n\t"
        "v_mfma_f32_16x16x32_bf16 v[204:207], v[140:143], v[80:83], v[204:207]\n\t"
        "ds_read_b64_tr_b16 v[180:181], v21 offset:57344\n\t"
        "ds_read_b32 v124, v127 offset:16\n\t"
        "v_mfma_f32_16x16x32_bf16 v[220:223], v[140:143], v[84:87], v[220:223]\n\t"
        "ds_read_b64_tr_b16 v[182:183], v21 offset:58368\n\t"
        "s_waitcnt vmcnt(12) lgkmcnt(14)\n\t"
        "v_mfma_f32_16x16x32_bf16 v[192:195], v[144:147], v[88:91], v[192:195]\n\t"
        "ds_read_b64_tr_b16 v[168:169], v18 offset:57344\n\t"
        "v_mfma_f32_16x16x32_bf16 v[208:211], v[144:147], v[92:95], v[208:211]\n\t"
        "ds_read_b64_tr_b16 v[170:171], v18 offset:58368\n\t"
        "v_mfma_f32_16x16x32_bf16 v[196:199], v[148:151], v[88:91], v[196:199]\n\t"
        "ds_read_b64_tr_b16 v[172:173], v19 offset:57344\n\t"
        "v_mfma_f32_16x16x32_bf16 v[212:215], v[148:151], v[92:95], v[212:215]\n\t"
        "ds_read_b64_tr_b16 v[174:175], v19 offset:58368\n\t"
        "s_waitcnt lgkmcnt(14)\n\t"
        "v_mfma_f32_16x16x32_bf16 v[200:203], v[152:155], v[88:91], v[200:203]\n\t"
        "ds_read_b64_tr_b16 v[184:185], v22 offset:57344\n\t"
        "v_mfma_f32_16x16x32_bf16 v[216:219], v[152:155], v[92:95], v[216:219]\n\t"
        "ds_read_b64_tr_b16 v[186:187], v22 offset:58368\n\t"
        "v_mfma_f32_16x16x32_bf16 v[204:207], v[156:159], v[88:91], v[204:207]\n\t"
        "ds_read_b64_tr_b16 v[188:189], v23 offset:57344\n\t"
        "v_mfma_f32_16x16x32_bf16 v[220:223], v[156:159], v[92:95], v[220:223]\n\t"
        "ds_read_b64_tr_b16 v[190:191], v23 offset:58368\n\t"
        "s_mov_b32 s74, 0xbfb8aa3b\n\t"
        "s_mov_b32 s75, 0xbfb8aa3b\n\t"
        "s_mov_b32 s76, 1.0\n\t"
        "s_mov_b32 s77, 1.0\n\t"
        "s_mov_b32 s78, 0x42ad1f97\n\t"
        "v_add_u32 v125, 0x20000, %5\n\t"
        "s_waitcnt lgkmcnt(9)\n\t"
        "v_cmp_ne_u32 vcc, 0, v124\n\t"
        "s_cbranch_vccnz .Lbad_%=\n\t"
        "s_waitcnt vmcnt(11)\n\t"
        "v_mfma_f32_16x16x32_bf16 v[192:195], v[160:163], v[96:99], v[192:195]\n\t"
        "v_mfma_f32_16x16x32_bf16 v[196:199], v[164:167], v[96:99], v[196:199]\n\t"
        "s_waitcnt vmcnt(9)\n\t"
        "v_mfma_f32_16x16x32_bf16 v[192:195], v[176:179], v[104:107], v[192:195]\n\t"
        "s_waitcnt lgkmcnt(8)\n\t"
        "v_mfma_f32_16x16x32_bf16 v[196:199], v[180:183], v[104:107], v[196:199]\n\t"
        "s_waitcnt lgkmcnt(6)\n\t"
        "v_mfma_f32_16x16x32_bf16 v[200:203], v[168:171], v[96:99], v[200:203]\n\t"
        "s_waitcnt lgkmcnt(4)\n\t"
        "v_mfma_f32_16x16x32_bf16 v[204:207], v[172:175], v[96:99], v[204:207]\n\t"
        "s_waitcnt lgkmcnt(2)\n\t"
        "v_mfma_f32_16x16x32_bf16 v[200:203], v[184:187], v[104:107], v[200:203]\n\t"
        "s_waitcnt lgkmcnt(0)\n\t"
        "v_mfma_f32_16x16x32_bf16 v[204:207], v[188:191], v[104:107], v[204:207]\n\t"
        "s_nop 3\n\t"
        "s_waitcnt vmcnt(6)\n\t"
        "v_pk_add_f32 v[192:193], v[192:193], v[224:225]\n\t"
        "v_pk_add_f32 v[194:195], v[194:195], v[226:227]\n\t"
        "v_pk_add_f32 v[196:197], v[196:197], v[228:229]\n\t"
        "v_pk_add_f32 v[198:199], v[198:199], v[230:231]\n\t"
        "v_pk_mul_f32 v[128:129], v[192:193], s[74:75]\n\t"
        "v_pk_mul_f32 v[130:131], v[194:195], s[74:75]\n\t"
        "v_pk_mul_f32 v[132:133], v[196:197], s[74:75]\n\t"
        "v_pk_mul_f32 v[134:135], v[198:199], s[74:75]\n\t"
        "v_med3_f32 v128, v128, s78, -s78\n\t"
        "v_med3_f32 v129, v129, s78, -s78\n\t"
        "v_med3_f32 v130, v130, s78, -s78\n\t"
        "v_med3_f32 v131, v131, s78, -s78\n\t"
        "v_med3_f32 v132, v132, s78, -s78\n\t"
        "v_med3_f32 v133, v133, s78, -s78\n\t"
        "v_med3_f32 v134, v134, s78, -s78\n\t"
        "v_med3_f32 v135, v135, s78, -s78\n\t"
        "v_mfma_f32_16x16x32_bf16 v[208:211], v[160:163], v[100:103], v[208:211]\n\t"
        "v_exp_f32 v128, v128\n\t"
        "v_exp_f32 v129, v129\n\t"
        "v_exp_f32 v130, v130\n\t"
        "v_exp_f32 v131, v131\n\t"
        "v_mfma_f32_16x16x32_bf16 v[212:215], v[164:167], v[100:103], v[212:215]\n\t"
        "v_exp_f32 v132, v132\n\t"
        "v_exp_f32 v133, v133\n\t"
        "v_exp_f32 v134, v134\n\t"
        "v_exp_f32 v135, v135\n\t"
        "v_pk_add_f32 v[128:129], v[128:129], s[76:77]\n\t"
        "v_pk_add_f32 v[130:131], v[130:131], s[76:77]\n\t"
        "v_pk_add_f32 v[132:133], v[132:133], s[76:77]\n\t"
        "v_pk_add_f32 v[134:135], v[134:135], s[76:77]\n\t"
        "v_mfma_f32_16x16x32_bf16 v[208:211], v[176:179], v[108:111], v[208:211]\n\t"
        "v_rcp_f32 v128, v128\n\t"
        "v_rcp_f32 v129, v129\n\t"
        "v_rcp_f32 v130, v130\n\t"
        "v_rcp_f32 v131, v131\n\t"
        "v_mfma_f32_16x16x32_bf16 v[212:215], v[180:183], v[108:111], v[212:215]\n\t"
        "v_rcp_f32 v132, v132\n\t"
        "v_rcp_f32 v133, v133\n\t"
        "v_rcp_f32 v134, v134\n\t"
        "v_rcp_f32 v135, v135\n\t"
        "v_pk_mul_f32 v[192:193], v[192:193], v[128:129]\n\t"
        "v_pk_mul_f32 v[194:195], v[194:195], v[130:131]\n\t"
        "v_pk_mul_f32 v[196:197], v[196:197], v[132:133]\n\t"
        "v_pk_mul_f32 v[198:199], v[198:199], v[134:135]\n\t"
        "v_cvt_pk_bf16_f32 v144, v192, v193\n\t"
        "v_cvt_pk_bf16_f32 v146, v196, v197\n\t"
        "v_cvt_pk_bf16_f32 v145, v194, v195\n\t"
        "v_cvt_pk_bf16_f32 v147, v198, v199\n\t"
        "s_waitcnt vmcnt(0)\n\t"
        "s_nop 0\n\t"
        "v_permlane16_swap_b32 v144, v146\n\t"
        "v_permlane16_swap_b32 v145, v147\n\t"
        "global_store_dwordx4 %5, v[144:147], s[56:57] offset:0 sc1\n\t"
        "v_pk_add_f32 v[200:201], v[200:201], v[232:233]\n\t"
        "v_pk_add_f32 v[202:203], v[202:203], v[234:235]\n\t"
        "v_pk_add_f32 v[204:205], v[204:205], v[236:237]\n\t"
        "v_pk_add_f32 v[206:207], v[206:207], v[238:239]\n\t"
        "v_pk_mul_f32 v[136:137], v[200:201], s[74:75]\n\t"
        "v_pk_mul_f32 v[138:139], v[202:203], s[74:75]\n\t"
        "v_pk_mul_f32 v[140:141], v[204:205], s[74:75]\n\t"
        "v_pk_mul_f32 v[142:143], v[206:207], s[74:75]\n\t"
        "v_med3_f32 v136, v136, s78, -s78\n\t"
        "v_med3_f32 v137, v137, s78, -s78\n\t"
        "v_med3_f32 v138, v138, s78, -s78\n\t"
        "v_med3_f32 v139, v139, s78, -s78\n\t"
        "v_med3_f32 v140, v140, s78, -s78\n\t"
        "v_med3_f32 v141, v141, s78, -s78\n\t"
        "v_med3_f32 v142, v142, s78, -s78\n\t"
        "v_med3_f32 v143, v143, s78, -s78\n\t"
        "v_mfma_f32_16x16x32_bf16 v[216:219], v[168:171], v[100:103], v[216:219]\n\t"
        "v_exp_f32 v136, v136\n\t"
        "v_exp_f32 v137, v137\n\t"
        "v_exp_f32 v138, v138\n\t"
        "v_exp_f32 v139, v139\n\t"
        "v_mfma_f32_16x16x32_bf16 v[220:223], v[172:175], v[100:103], v[220:223]\n\t"
        "v_exp_f32 v140, v140\n\t"
        "v_exp_f32 v141, v141\n\t"
        "v_exp_f32 v142, v142\n\t"
        "v_exp_f32 v143, v143\n\t"
        "v_pk_add_f32 v[136:137], v[136:137], s[76:77]\n\t"
        "v_pk_add_f32 v[138:139], v[138:139], s[76:77]\n\t"
        "v_pk_add_f32 v[140:141], v[140:141], s[76:77]\n\t"
        "v_pk_add_f32 v[142:143], v[142:143], s[76:77]\n\t"
        "v_mfma_f32_16x16x32_bf16 v[216:219], v[184:187], v[108:111], v[216:219]\n\t"
        "v_rcp_f32 v136, v136\n\t"
        "v_rcp_f32 v137, v137\n\t"
        "v_rcp_f32 v138, v138\n\t"
        "v_rcp_f32 v139, v139\n\t"
        "v_mfma_f32_16x16x32_bf16 v[220:223], v[188:191], v[108:111], v[220:223]\n\t"
        "v_rcp_f32 v140, v140\n\t"
        "v_rcp_f32 v141, v141\n\t"
        "v_rcp_f32 v142, v142\n\t"
        "v_rcp_f32 v143, v143\n\t"
        "v_pk_mul_f32 v[200:201], v[200:201], v[136:137]\n\t"
        "v_pk_mul_f32 v[202:203], v[202:203], v[138:139]\n\t"
        "v_pk_mul_f32 v[204:205], v[204:205], v[140:141]\n\t"
        "v_pk_mul_f32 v[206:207], v[206:207], v[142:143]\n\t"
        "v_cvt_pk_bf16_f32 v148, v200, v201\n\t"
        "v_cvt_pk_bf16_f32 v150, v204, v205\n\t"
        "v_cvt_pk_bf16_f32 v149, v202, v203\n\t"
        "v_cvt_pk_bf16_f32 v151, v206, v207\n\t"
        "s_nop 0\n\t"
        "v_permlane16_swap_b32 v148, v150\n\t"
        "v_permlane16_swap_b32 v149, v151\n\t"
        "global_store_dwordx4 %5, v[148:151], s[56:57] offset:64 sc1\n\t"
        "v_pk_add_f32 v[208:209], v[208:209], v[240:241]\n\t"
        "v_pk_add_f32 v[210:211], v[210:211], v[242:243]\n\t"
        "v_pk_add_f32 v[212:213], v[212:213], v[244:245]\n\t"
        "v_pk_add_f32 v[214:215], v[214:215], v[246:247]\n\t"
        "v_pk_mul_f32 v[128:129], v[208:209], s[74:75]\n\t"
        "v_pk_mul_f32 v[130:131], v[210:211], s[74:75]\n\t"
        "v_pk_mul_f32 v[132:133], v[212:213], s[74:75]\n\t"
        "v_pk_mul_f32 v[134:135], v[214:215], s[74:75]\n\t"
        "v_med3_f32 v128, v128, s78, -s78\n\t"
        "v_med3_f32 v129, v129, s78, -s78\n\t"
        "v_med3_f32 v130, v130, s78, -s78\n\t"
        "v_med3_f32 v131, v131, s78, -s78\n\t"
        "v_med3_f32 v132, v132, s78, -s78\n\t"
        "v_med3_f32 v133, v133, s78, -s78\n\t"
        "v_med3_f32 v134, v134, s78, -s78\n\t"
        "v_med3_f32 v135, v135, s78, -s78\n\t"
        "v_exp_f32 v128, v128\n\t"
        "v_exp_f32 v129, v129\n\t"
        "v_exp_f32 v130, v130\n\t"
        "v_exp_f32 v131, v131\n\t"
        "v_exp_f32 v132, v132\n\t"
        "v_exp_f32 v133, v133\n\t"
        "v_exp_f32 v134, v134\n\t"
        "v_exp_f32 v135, v135\n\t"
        "v_pk_add_f32 v[128:129], v[128:129], s[76:77]\n\t"
        "v_pk_add_f32 v[130:131], v[130:131], s[76:77]\n\t"
        "v_pk_add_f32 v[132:133], v[132:133], s[76:77]\n\t"
        "v_pk_add_f32 v[134:135], v[134:135], s[76:77]\n\t"
        "v_rcp_f32 v128, v128\n\t"
        "v_rcp_f32 v129, v129\n\t"
        "v_rcp_f32 v130, v130\n\t"
        "v_rcp_f32 v131, v131\n\t"
        "v_rcp_f32 v132, v132\n\t"
        "v_rcp_f32 v133, v133\n\t"
        "v_rcp_f32 v134, v134\n\t"
        "v_rcp_f32 v135, v135\n\t"
        "v_pk_mul_f32 v[208:209], v[208:209], v[128:129]\n\t"
        "v_pk_mul_f32 v[210:211], v[210:211], v[130:131]\n\t"
        "v_pk_mul_f32 v[212:213], v[212:213], v[132:133]\n\t"
        "v_pk_mul_f32 v[214:215], v[214:215], v[134:135]\n\t"
        "v_cvt_pk_bf16_f32 v152, v208, v209\n\t"
        "v_cvt_pk_bf16_f32 v154, v212, v213\n\t"
        "v_cvt_pk_bf16_f32 v153, v210, v211\n\t"
        "v_cvt_pk_bf16_f32 v155, v214, v215\n\t"
        "s_nop 0\n\t"
        "v_permlane16_swap_b32 v152, v154\n\t"
        "v_permlane16_swap_b32 v153, v155\n\t"
        "global_store_dwordx4 v125, v[152:155], s[56:57] offset:0 sc1\n\t"
        "v_pk_add_f32 v[216:217], v[216:217], v[248:249]\n\t"
        "v_pk_add_f32 v[218:219], v[218:219], v[250:251]\n\t"
        "v_pk_add_f32 v[220:221], v[220:221], v[252:253]\n\t"
        "v_pk_add_f32 v[222:223], v[222:223], v[254:255]\n\t"
        "v_pk_mul_f32 v[136:137], v[216:217], s[74:75]\n\t"
        "v_pk_mul_f32 v[138:139], v[218:219], s[74:75]\n\t"
        "v_pk_mul_f32 v[140:141], v[220:221], s[74:75]\n\t"
        "v_pk_mul_f32 v[142:143], v[222:223], s[74:75]\n\t"
        "v_med3_f32 v136, v136, s78, -s78\n\t"
        "v_med3_f32 v137, v137, s78, -s78\n\t"
        "v_med3_f32 v138, v138, s78, -s78\n\t"
        "v_med3_f32 v139, v139, s78, -s78\n\t"
        "v_med3_f32 v140, v140, s78, -s78\n\t"
        "v_med3_f32 v141, v141, s78, -s78\n\t"
        "v_med3_f32 v142, v142, s78, -s78\n\t"
        "v_med3_f32 v143, v143, s78, -s78\n\t"
        "v_exp_f32 v136, v136\n\t"
        "v_exp_f32 v137, v137\n\t"
        "v_exp_f32 v138, v138\n\t"
        "v_exp_f32 v139, v139\n\t"
        "v_exp_f32 v140, v140\n\t"
        "v_exp_f32 v141, v141\n\t"
        "v_exp_f32 v142, v142\n\t"
        "v_exp_f32 v143, v143\n\t"
        "v_pk_add_f32 v[136:137], v[136:137], s[76:77]\n\t"
        "v_pk_add_f32 v[138:139], v[138:139], s[76:77]\n\t"
        "v_pk_add_f32 v[140:141], v[140:141], s[76:77]\n\t"
        "v_pk_add_f32 v[142:143], v[142:143], s[76:77]\n\t"
        "v_rcp_f32 v136, v136\n\t"
        "v_rcp_f32 v137, v137\n\t"
        "v_rcp_f32 v138, v138\n\t"
        "v_rcp_f32 v139, v139\n\t"
        "v_rcp_f32 v140, v140\n\t"
        "v_rcp_f32 v141, v141\n\t"
        "v_rcp_f32 v142, v142\n\t"
        "v_rcp_f32 v143, v143\n\t"
        "v_pk_mul_f32 v[216:217], v[216:217], v[136:137]\n\t"
        "v_pk_mul_f32 v[218:219], v[218:219], v[138:139]\n\t"
        "v_pk_mul_f32 v[220:221], v[220:221], v[140:141]\n\t"
        "v_pk_mul_f32 v[222:223], v[222:223], v[142:143]\n\t"
        "v_cvt_pk_bf16_f32 v156, v216, v217\n\t"
        "v_cvt_pk_bf16_f32 v158, v220, v221\n\t"
        "v_cvt_pk_bf16_f32 v157, v218, v219\n\t"
        "v_cvt_pk_bf16_f32 v159, v222, v223\n\t"
        "s_nop 0\n\t"
        "v_permlane16_swap_b32 v156, v158\n\t"
        "v_permlane16_swap_b32 v157, v159\n\t"
        "global_store_dwordx4 v125, v[156:159], s[56:57] offset:64 sc1\n\t"
        "s_endpgm\n\t"
        ".Lspin1_%=:\n\t"
        ".Lspl1_%=:\n\t"
        "ds_read_b128 v[120:123], v127\n\t"
        "s_waitcnt lgkmcnt(0)\n\t"
        "v_cmp_ge_u32 vcc, s61, v121\n\t"
        "s_cbranch_vccz .Lret1_%=\n\t"
        "s_sleep 1\n\t"
        "s_branch .Lspl1_%=\n\t"
        ".Lret1_%=:\n\t"
        "s_setpc_b64 s[70:71]\n\t"
        ".Lspin2_%=:\n\t"
        ".Lspl2_%=:\n\t"
        "ds_read_b128 v[120:123], v127\n\t"
        "s_waitcnt lgkmcnt(0)\n\t"
        "v_cmp_ge_u32 vcc, s61, v122\n\t"
        "s_cbranch_vccz .Lret2_%=\n\t"
        "s_sleep 1\n\t"
        "s_branch .Lspl2_%=\n\t"
        ".Lret2_%=:\n\t"
        "s_setpc_b64 s[70:71]\n\t"
        ".Lspin4_%=:\n\t"
        "v_lshl_add_u32 v28, s61, 2, v127\n\t"
        ".Lspl4_%=:\n\t"
        "ds_read_b32 v24, v28 offset:256\n\t"
        "s_waitcnt lgkmcnt(0)\n\t"
        "v_cmp_eq_u32 vcc, 0, v24\n\t"
        "s_cbranch_vccz .Lret4_%=\n\t"
        "s_sleep 1\n\t"
        "s_branch .Lspl4_%=\n\t"
        ".Lret4_%=:\n\t"
        "s_setpc_b64 s[70:71]\n\t"
        ".Lspin5_%=:\n\t"
        "v_lshl_add_u32 v28, s61, 2, v127\n\t"
        ".Lspl5_%=:\n\t"
        "ds_read_b32 v25, v28 offset:256\n\t"
        "s_waitcnt lgkmcnt(0)\n\t"
        "v_cmp_eq_u32 vcc, 0, v25\n\t"
        "s_cbranch_vccz .Lret5_%=\n\t"
        "s_sleep 1\n\t"
        "s_branch .Lspl5_%=\n\t"
        ".Lret5_%=:\n\t"
        "s_setpc_b64 s[70:71]\n\t"
        ".Lspin6_%=:\n\t"
        "v_lshl_add_u32 v28, s61, 2, v127\n\t"
        ".Lspl6_%=:\n\t"
        "ds_read_b32 v26, v28 offset:256\n\t"
        "s_waitcnt lgkmcnt(0)\n\t"
        "v_cmp_eq_u32 vcc, 0, v26\n\t"
        "s_cbranch_vccz .Lret6_%=\n\t"
        "s_sleep 1\n\t"
        "s_branch .Lspl6_%=\n\t"
        ".Lret6_%=:\n\t"
        "s_setpc_b64 s[70:71]\n\t"
        ".Lspin7_%=:\n\t"
        "v_lshl_add_u32 v28, s61, 2, v127\n\t"
        ".Lspl7_%=:\n\t"
        "ds_read_b32 v27, v28 offset:256\n\t"
        "s_waitcnt lgkmcnt(0)\n\t"
        "v_cmp_eq_u32 vcc, 0, v27\n\t"
        "s_cbranch_vccz .Lret7_%=\n\t"
        "s_sleep 1\n\t"
        "s_branch .Lspl7_%=\n\t"
        ".Lret7_%=:\n\t"
        "s_setpc_b64 s[70:71]\n\t"
        ".Lbad_%=:\n\t"
        "s_waitcnt vmcnt(0) lgkmcnt(0)\n\t"
        "v_mov_b32 v144, 0x7fc07fc0\n\t"
        "v_mov_b32 v145, 0x7fc07fc0\n\t"
        "v_mov_b32 v146, 0x7fc07fc0\n\t"
        "v_mov_b32 v147, 0x7fc07fc0\n\t"
        "global_store_dwordx4 %5, v[144:147], s[56:57] offset:0 sc1\n\t"
        "global_store_dwordx4 %5, v[144:147], s[56:57] offset:64 sc1\n\t"
        "global_store_dwordx4 v125, v[144:147], s[56:57] offset:0 sc1\n\t"
        "global_store_dwordx4 v125, v[144:147], s[56:57] offset:64 sc1\n\t"
        "s_branch .Lend_%=\n\t"
        ".Lend_%=:\n\t"
        "s_nop 7\n\t"
        "s_nop 7\n\t"
        :  "=&" KA(ka)
        : "v"(vTr), "v"(voffA), "v"(vRow), "v"(vCons), "v"(vOut),
          "s"((mt * 8 + wv * 2) * 1024), "s"((unsigned)OFF_X + ((bid & XBM) * 4 + wv) * 8192 + 4096), "s"((unsigned)OFF_XF + bid * 128 + wv * 4),
          "s"(lds0 + LDS_CTRL), "s"(pos), "s"(par << 15)
        : "s51", "v8", "v9", "v10", "v11", "v12", "v13", "v14", "v15", "v16", "v17", "v18", "v19", "v20", "v21", "v22", "v23", "v24", "v25", "v26", "v27", "v28", "v29", "v30", "v31", "v32", "v33", "v34", "v35", "v36", "v37", "v38", "v39", "v40", "v41", "v42", "v43", "v44", "v45", "v46", "v47", "v48", "v49", "v50", "v51", "v52", "v53", "v54", "v55", "v56", "v57", "v58", "v59", "v60", "v61", "v62", "v63", "v64", "v65", "v66", "v67", "v68", "v69", "v70", "v71", "v72", "v73", "v74", "v75", "v76", "v77", "v78", "v79", "v80", "v81", "v82", "v83", "v84", "v85", "v86", "v87", "v88", "v89", "v90", "v91", "v92", "v93", "v94", "v95", "v96", "v97", "v98", "v99", "v100", "v101", "v102", "v103", "v104", "v105", "v106", "v107", "v108", "v109", "v110", "v111", "v112", "v113", "v114", "v115", "v116", "v117", "v118", "v119", "v120", "v121", "v122", "v123", "v124", "v125", "v126", "v127", "v128", "v129", "v130", "v131", "v132", "v133", "v134", "v135", "v136", "v137", "v138", "v139", "v140", "v141", "v142", "v143", "v144", "v145", "v146", "v147", "v148", "v149", "v150", "v151", "v152", "v153", "v154", "v155", "v156", "v157", "v158", "v159", "v160", "v161", "v162", "v163", "v164", "v165", "v166", "v167", "v168", "v169", "v170", "v171", "v172", "v173", "v174", "v175", "v176", "v177", "v178", "v179", "v180", "v181", "v182", "v183", "v184", "v185", "v186", "v187", "v188", "v189", "v190", "v191", "v192", "v193", "v194", "v195", "v196", "v197", "v198", "v199", "v200", "v201", "v202", "v203", "v204", "v205", "v206", "v207", "v208", "v209", "v210", "v211", "v212", "v213", "v214", "v215", "v216", "v217", "v218", "v219", "v220", "v221", "v222", "v223", "v224", "v225", "v226", "v227", "v228", "v229", "v230", "v231", "v232", "v233", "v234", "v235", "v236", "v237", "v238", "v239", "v240", "v241", "v242", "v243", "v244", "v245", "v246", "v247", "v248", "v249", "v250", "v251", "v252", "v253", "v254", "v255", "s60", "s61", "s62", "s63", "s64", "s65", "s66", "s67", "s68", "s69", "s70", "s71", "s72", "s73", "s74", "s75", "s76", "s77", "s78", "s79", "s80", "s81", "s82", "s83", "s84", "s85", "s86", "s87", "s88", "s89", "s90", "s91", "s92", "s93", "s94", "s95", "s96", "s97", "vcc", "scc", "memory");
    }
    ARGS();
    const int lane = get_lane();
    ENDW();
}

extern "C" void kernel_launch(int rank, int nranks, ncclComm_t comm,
                              hipStream_t stream,
                              void* const* d_in, const int* in_sizes, int n_in,
                              void* d_out, int out_size,
                              const int* in_dtypes, int out_dtype,
                              void* const* d_out_all,
                              void* const* d_ws_all, size_t ws_bytes) {
    (void)comm; (void)in_sizes; (void)n_in; (void)out_size; (void)in_dtypes; (void)out_dtype; (void)d_out_all;
    if (nranks != NR || (unsigned)rank >= (unsigned)NR || ws_bytes < WS_NEED) {
        fprintf(stderr, "unsupported configuration\n");
        return;
    }
    Params p;
    p.x = (const char*)d_in[0];
    p.wm = (const char*)d_in[1] - 4096;
    p.out = (char*)d_out;
    p.myws = (char*)d_ws_all[rank];
    for (int i = 0; i < NR - 1; ++i) p.peer[i] = (char*)d_ws_all[(rank + 1 + i) & 7];
    p.rank = rank;
    p.pad = 0;
    hipLaunchKernelGGL(a2a_gemm_silu, dim3(NWG), dim3(NTHR), 0, stream, p);
    HIP_CHECK(hipGetLastError());
}
```
